```python
import numpy as np

kernel_name = "gpu_gemm_ar_m4096_k4096_n32768_bf16_relu"


ABSMAX_RTOL = 0.008

COMPUTE_DTYPE = "bfloat16"
EPILOGUE = "relu"
IN_QUANT = None
OUT_QUANT = None
IN_DTYPES = {"x": "bfloat16", "w_mat": "bfloat16"}
WORLD_SIZE = 8

def _to_bf16(x):
    u = np.ascontiguousarray(x, dtype=np.float32).view(np.uint32)
    return ((u + 0x7FFF + ((u >> 16) & 1)) & 0xFFFF0000).view(np.float32)

def setup_inputs():

    x = _to_bf16(np.random.randn(4096, 4096).astype(np.float32))
    w_mat = _to_bf16((np.random.randn(4096, 32768) / np.sqrt(4096)).astype(np.float32))
    return {"x": x, "w_mat": w_mat}

def _epilogue(y):
    return np.maximum(y, 0.0).astype(np.float32)

def reference(x, w_mat):
    y = (x @ w_mat).astype(np.float32)
    return _epilogue(y)
```

```cpp
#include <hip/hip_runtime.h>
#include <rccl/rccl.h>
#include <cstdio>

typedef unsigned u32;
typedef unsigned long long u64;
typedef u32 u32x4 __attribute__((ext_vector_type(4)));
typedef u32 u32x2 __attribute__((ext_vector_type(2)));
typedef unsigned short u16;
typedef u16 u16x2 __attribute__((ext_vector_type(2)));
typedef short bf16x8 __attribute__((ext_vector_type(8)));
typedef float f32x4 __attribute__((ext_vector_type(4)));
typedef float f32x2 __attribute__((ext_vector_type(2)));
typedef __bf16 bf16x2 __attribute__((ext_vector_type(2)));
#define LAS __attribute__((address_space(3)))
#define GAS __attribute__((address_space(1)))
#define DEV __device__ __forceinline__

constexpr int NR = 8;
constexpr u32 WS_CALL = 0;
constexpr u32 WS_TMO = 8192;
constexpr u32 WS_TICKET = 8192 + 128;
constexpr u32 WS_READY = 8192 + 256;
constexpr u32 WS_STAMP = 16384;
constexpr int NJOB = 160;
constexpr u32 WS_DESC = 512u << 10;
constexpr u32 WS_IMG = 1u << 20;
constexpr u32 IMG_BYTES = 32u << 20;
constexpr u32 WS_BLOB = 65u << 20;
constexpr u32 SLOT_W = 256, SLOT_O = 2304, NSLOT = 4352;
constexpr u32 WS_USED = WS_BLOB + NSLOT * 16384u;
constexpr u32 OUT_BYTES = 4096u * 32768u * 2u;
constexpr u64 DEADLINE = 100000000ull;
constexpr int LDS_BYTES = 163840;
constexpr int NDEC = 6;
constexpr int DREG = 26624;
constexpr int L_CTRL = NDEC * DREG;
constexpr int L_REQ = L_CTRL, L_FULL = L_CTRL + 64, L_LUT = L_CTRL + 128;
constexpr int EREG = 20480;
constexpr int E_EXC = 16448;
constexpr u32 IN_EXC_CAP = 2528, OUT_EXC_CAP = 1984;
constexpr int NPIECE = 2304;
#define WIN 7
#define PREPW 4
#define OUT_AUX 0

struct Args {
  const char* x; const char* w; char* out; char* ws[NR]; int rank; int flags; unsigned char tiles[256];
};

template <int N> DEV void waitv() { asm volatile("s_waitcnt vmcnt(%0)" ::"n"(N) : "memory"); }
DEV void cbar() { asm volatile("" ::: "memory"); }
DEV u64 now() { return __builtin_amdgcn_s_memrealtime(); }
DEV u32 perm(u32 hi, u32 lo, u32 sel) { return __builtin_amdgcn_perm(hi, lo, sel); }
DEV u32 pk_sub(u32 a, u32 b) { return __builtin_bit_cast(u32, __builtin_bit_cast(u16x2, a) - __builtin_bit_cast(u16x2, b)); }
DEV u32 pk_add(u32 a, u32 b) { return __builtin_bit_cast(u32, __builtin_bit_cast(u16x2, a) + __builtin_bit_cast(u16x2, b)); }
DEV u32 pk_max(u32 a, u32 b) { return __builtin_bit_cast(u32, __builtin_elementwise_max(__builtin_bit_cast(u16x2, a), __builtin_bit_cast(u16x2, b))); }
DEV u32 pk_min(u32 a, u32 b) { return __builtin_bit_cast(u32, __builtin_elementwise_min(__builtin_bit_cast(u16x2, a), __builtin_bit_cast(u16x2, b))); }
template <int C, int RM> DEV u32 dpp0(u32 x) { return (u32)__builtin_amdgcn_update_dpp(0, (int)x, C, RM, 0xf, false); }
DEV u32 umax(u32 a, u32 b) { return a > b ? a : b; }
DEV u32 umin(u32 a, u32 b) { return a < b ? a : b; }
DEV u32 wave_max(u32 x) {
  x = umax(x, dpp0<0x111, 0xf>(x)); x = umax(x, dpp0<0x112, 0xf>(x)); x = umax(x, dpp0<0x114, 0xf>(x)); x = umax(x, dpp0<0x118, 0xf>(x));
  x = umax(x, dpp0<0x142, 0xa>(x)); x = umax(x, dpp0<0x143, 0xc>(x));
  return (u32)__builtin_amdgcn_readlane((int)x, 63);
}
DEV u32 wave_scan(u32 x) {
  x += dpp0<0x111, 0xf>(x); x += dpp0<0x112, 0xf>(x); x += dpp0<0x114, 0xf>(x); x += dpp0<0x118, 0xf>(x);
  x += dpp0<0x142, 0xa>(x); x += dpp0<0x143, 0xc>(x);
  return x;
}
DEV u32 rfl(u32 x) { return (u32)__builtin_amdgcn_readfirstlane((int)x); }
DEV u32 mbcnt(u64 m) { return __builtin_amdgcn_mbcnt_hi((u32)(m >> 32), __builtin_amdgcn_mbcnt_lo((u32)m, 0)); }
DEV u32x4 lds_r128(LAS char* p) { return *(LAS u32x4*)p; }
DEV void lds_w128(LAS char* p, u32x4 v) { *(LAS u32x4*)p = v; }
DEV u32 lds_poll(u32 addr) { u32 v; asm volatile("ds_read_b32 %0, %1\n\ts_waitcnt lgkmcnt(0)" : "=v"(v) : "v"(addr) : "memory"); return v; }
DEV void lds_post(u32 addr, u32 v) { asm volatile("ds_write_b32 %0, %1" ::"v"(addr), "v"(v) : "memory"); }
DEV void st_sys128(void* p, u32x4 v) { asm volatile("global_store_dwordx4 %0, %1, off sc0 sc1\n\ts_nop 1" ::"v"(p), "v"(v) : "memory"); }
DEV u32x4 ld_sys128(const void* p) { u32x4 v; asm volatile("global_load_dwordx4 %0, %1, off sc0 sc1\n\ts_waitcnt vmcnt(0)" : "=&v"(v) : "v"(p) : "memory"); return v; }
typedef __amdgpu_buffer_rsrc_t rsrc_t;
DEV rsrc_t mkrsrc(const void* p, u32 bytes) { return __builtin_amdgcn_make_buffer_rsrc((void*)p, 0, (int)bytes, 0x00020000); }
DEV void bst128(rsrc_t r, u32 voff, u32x4 v) { __builtin_amdgcn_raw_buffer_store_b128(v, r, (int)voff, 0, 17); }
DEV void bst32(rsrc_t r, u32 voff, u32 v) { __builtin_amdgcn_raw_buffer_store_b32(v, r, (int)voff, 0, 17); }

#define STAMP_AT(k) do { } while (0)

DEV void emit_exc(const u32 (&U)[4], bool act, u32 gidx, u32 upz, LAS char* list, u32 cap, u32& E, bool& bad) {
  u32 F = (((U[0] + 0x7C7C7C7Cu) & 0x80808080u) >> 7) | (((U[1] + 0x7C7C7C7Cu) & 0x80808080u) >> 6) |
          (((U[2] + 0x7C7C7C7Cu) & 0x80808080u) >> 5) | (((U[3] + 0x7C7C7C7Cu) & 0x80808080u) >> 4);
  if (!act) F = 0;
  for (;;) {
    const u64 bal = __builtin_amdgcn_ballot_w64(F != 0);
    if (bal == 0) break;
    if (F != 0) {
      const u32 p = __builtin_ctz(F), k = p & 7, b = p >> 3;
      const u32 h0 = (p & 1) ? U[1] : U[0], h1 = (p & 1) ? U[3] : U[2], h = (p & 2) ? h1 : h0;
      const u32 up = (h >> (8 * b + 2)) & 31;
      u32 c = 32 - up;
      if (c > 7) { bad |= up != upz; c = 0; }
      const u32 slot = E + mbcnt(bal);
      if (slot < cap) *(LAS u16*)(list + 2 * slot) = (u16)(((gidx + 4 * k + b) << 3) | c);
      F &= F - 1;
    }
    E += __builtin_popcountll(bal);
  }
}

DEV void prep_piece(const Args& a, int e, u32 callv, char* wsp, LAS char* scr, int lane) {
  const int s = e / 144, q = e - s * 144;
  const int fr = lane & 15, fq = lane >> 4;
  const bool isx = q < 16;
  const int piece = q & 15, dest = (a.rank + (q >> 4)) & 7;
  const u32 slot = (isx ? 0 : SLOT_W + dest * 256) + s * 16 + piece;
  u32 v[64];
  {
    const rsrc_t ri = mkrsrc(isx ? a.x : a.w, isx ? 4096u * 512u * 2u : 512u * 32768u * 2u);
    const u32 lv = isx ? fr * 1024 + fq * 16 : fq * 8 * 65536 + 16 * (4 * (fr & 3) + (fr >> 2));
    const u32 ls = isx ? (s * 256 + (piece & 1) * 128) * 1024 + (piece >> 1) * 128
                       : (piece >> 1) * 64 * 65536 + (dest * 4096 + s * 256 + (piece & 1) * 128) * 2;
    const u32 s8 = isx ? 65536 : 32 * 65536, s2 = isx ? 16384 : 2 * 65536, s1 = isx ? 64 : 65536;
    u32x4 in[16];
#pragma unroll
    for (int i = 0; i < 16; ++i) in[i] = __builtin_amdgcn_raw_buffer_load_b128(ri, lv, ls + (i >> 3) * s8 + ((i >> 1) & 3) * s2 + (i & 1) * s1, 2);
    __builtin_amdgcn_sched_barrier(0);
    if (isx) {
#pragma unroll
      for (int i = 0; i < 64; ++i) v[i] = in[i >> 2][i & 3];
    } else {
#pragma unroll
      for (int ks = 0; ks < 2; ++ks)
#pragma unroll
        for (int el = 0; el < 8; ++el)
#pragma unroll
          for (int d = 0; d < 4; ++d)
            v[4 * (el * 2 + ks) + d] = perm(in[ks * 8 + 2 * d + 1][el >> 1], in[ks * 8 + 2 * d][el >> 1], (el & 1) ? 0x07060302u : 0x05040100u);
    }
  }
  u32 mx = 0;
#pragma unroll
  for (int i = 0; i < 64; ++i) { v[i] = __builtin_amdgcn_alignbit(v[i], v[i], 31); mx = pk_max(mx, v[i]); }
  const u32 um = wave_max(umax(mx & 0xFFFF, mx >> 16)) | 1;
  const u32 base = um > 2047 ? um - 2047 : 0, bb = base * 0x10001u, upz = ((65536 - base) >> 11) & 31;
  const rsrc_t rb = mkrsrc(a.ws[a.rank] + WS_BLOB + slot * 16384u, 16384u);
  const u32 vo = lane * 16;
  u32 H[32];
#pragma unroll
  for (int r = 0; r < 8; ++r) {
    u32x4 lo;
#pragma unroll
    for (int k = 0; k < 4; ++k) {
      const u32 t0 = pk_sub(v[8 * r + 2 * k], bb), t1 = pk_sub(v[8 * r + 2 * k + 1], bb);
      lo[k] = perm(t1, t0, 0x06040200u); H[4 * r + k] = perm(t1, t0, 0x07050301u);
    }
    bst128(rb, vo + r * 1024, lo);
  }
  u32 E = 0; bool bad = false;
#pragma unroll
  for (int r = 0; r < 8; ++r) {
    u32 U[4];
#pragma unroll
    for (int k = 0; k < 4; ++k) U[k] = (H[4 * r + k] >> 1) & 0x7C7C7C7Cu;
    emit_exc(U, true, (r * 64 + lane) * 16, upz, scr, IN_EXC_CAP, E, bad);
  }
  const bool raw = __builtin_amdgcn_ballot_w64(bad) != 0 || E > IN_EXC_CAP || (a.flags & 1);
  if (raw) {
#pragma unroll
    for (int r = 0; r < 8; ++r) bst128(rb, vo + (8 + r) * 1024, (u32x4){H[4 * r], H[4 * r + 1], H[4 * r + 2], H[4 * r + 3]});
  } else {
    u32 P[12];
#pragma unroll
    for (int g = 0; g < 4; ++g) {
      const u32* h = H + 8 * g; const u32 m7 = 0x07070707u;
      P[3 * g] = (h[0] & m7) | ((h[1] & m7) << 3) | ((h[2] & 0x03030303u) << 6);
      P[3 * g + 1] = (h[3] & m7) | ((h[4] & m7) << 3) | ((h[5] & 0x03030303u) << 6);
      P[3 * g + 2] = (h[6] & m7) | ((h[7] & m7) << 3) | ((h[2] & 0x04040404u) << 4) | ((h[5] & 0x04040404u) << 5);
    }
#pragma unroll
    for (int r = 0; r < 3; ++r) bst128(rb, vo + (8 + r) * 1024, (u32x4){P[4 * r], P[4 * r + 1], P[4 * r + 2], P[4 * r + 3]});
    cbar();
    for (u32 i = vo; i < 2 * E; i += 1024) bst128(rb, 11264 + i, lds_r128(scr + i));
    cbar();
  }
  waitv<0>();
  asm volatile("" : "+v"(callv), "+v"(wsp));
  const u32x4 d = {callv + 1, base | (raw ? 0u : E << 16), raw ? 0x80000000u : 0u, callv + 1};
  const u32 doff = WS_DESC + ((a.rank * 16 + piece) * NJOB + s * 2 + (isx ? 0 : 1)) * 16;
  if (lane < NR && (isx || lane == dest)) st_sys128(wsp + doff, d);
}

DEV void pack_out(const Args& a, f32x4 (&acc)[2][2][4][2], u32 blob, u32 call, LAS char* reg, int lane) {
  u32 o[64];
#pragma unroll
  for (int bj = 0; bj < 2; ++bj)
#pragma unroll
    for (int n = 0; n < 2; ++n)
#pragma unroll
      for (int j = 0; j < 4; ++j)
#pragma unroll
        for (int d = 0; d < 4; ++d) {
          const f32x2 y = {__builtin_elementwise_maximum(acc[d >> 1][bj][(d & 1) * 2][n][j], 0.f),
                           __builtin_elementwise_maximum(acc[d >> 1][bj][(d & 1) * 2 + 1][n][j], 0.f)};
          o[((bj * 2 + n) * 4 + j) * 4 + d] = __builtin_bit_cast(u32, __builtin_convertvector(y, bf16x2)) & 0x7FFF7FFFu;
        }
  u32 mx = 0; u32x4 bits;
#pragma unroll
  for (int q = 0; q < 4; ++q) {
    u32 m16[2];
#pragma unroll
    for (int h = 0; h < 2; ++h) {
      u32 t = 0;
#pragma unroll
      for (int i = 0; i < 8; ++i) { const u32 x = o[q * 16 + h * 8 + i]; mx = pk_max(mx, x); t = (t << 2) | pk_min(x, 0x00010001u); }
      m16[h] = ((t & 0xFFFF) << 1) | (t >> 16);
    }
    bits[q] = (m16[0] << 16) | m16[1];
  }
  const u32 vmax = wave_max(umax(mx & 0xFFFF, mx >> 16));
  const u32 off = vmax > 1023 ? vmax - 1023 : 0, offp = off * 0x10001u;
  const u32 cnt = __builtin_popcount(bits.x) + __builtin_popcount(bits.y) + __builtin_popcount(bits.z) + __builtin_popcount(bits.w);
  const u32 dl = (lane & 15) * 4 + (lane >> 4);
  const u32 cd = (u32)__builtin_amdgcn_ds_bpermute((((lane & 3) * 16 + (lane >> 2)) * 4), (int)cnt);
  const u32 inc = wave_scan(cd);
  const u32 n = (u32)__builtin_amdgcn_readlane((int)inc, 63);
  const u32 start = (u32)__builtin_amdgcn_ds_bpermute((int)(dl * 4), (int)(inc - cd));
  const rsrc_t rb = mkrsrc(a.ws[a.rank] + WS_BLOB + (SLOT_O + blob) * 16384u, 16384u);
  bst128(rb, dl * 16, bits);
  {
    u32 pos = 2 * start; const u32 dump = E_EXC + lane * 4;
#pragma unroll
    for (int i = 0; i < 64; ++i) {
      const u32 x = o[i]; const bool z0 = (x & 0xFFFF) != 0, z1 = x > 0xFFFF;
      *(LAS u16*)(reg + (z0 ? pos : dump)) = (u16)x; pos += z0 ? 2 : 0;
      *(LAS u16*)(reg + (z1 ? pos : dump)) = (u16)(x >> 16); pos += z1 ? 2 : 0;
    }
    if (lane < 16) *(LAS u16*)(reg + 2 * (n + lane)) = (u16)off;
  }
  cbar();
  const u32 G = (n + 15) >> 4;
  u32 E = 0; bool bad = false;
  for (u32 g0 = 0; g0 < G; g0 += 64) {
    const u32 g = g0 + lane; const bool act = g < G;
    const u32x4 t0 = lds_r128(reg + 32 * g), t1 = lds_r128(reg + 32 * g + 16);
    u32 t[8] = {t0.x, t0.y, t0.z, t0.w, t1.x, t1.y, t1.z, t1.w}, U[4], code = 0; u32x4 lo;
#pragma unroll
    for (int k = 0; k < 4; ++k) {
      const u32 p0 = pk_sub(t[2 * k], offp), p1 = pk_sub(t[2 * k + 1], offp), h = perm(p1, p0, 0x07050301u);
      lo[k] = perm(p1, p0, 0x06040200u); code |= (h & 0x03030303u) << (2 * k); U[k] = h & 0x7C7C7C7Cu;
    }
    if (act) { bst128(rb, 1024 + 16 * g, lo); bst32(rb, 1024 + 16 * G + 4 * g, code); }
    emit_exc(U, act, g * 16, 99, reg + E_EXC, OUT_EXC_CAP, E, bad);
  }
  cbar();
  const bool raw = __builtin_amdgcn_ballot_w64(bad) != 0 || E > OUT_EXC_CAP || (a.flags & 2);
  if (raw) {
    waitv<0>();
#pragma unroll
    for (int c = 0; c < 16; ++c) bst128(rb, c * 1024 + dl * 16, (u32x4){o[4 * c], o[4 * c + 1], o[4 * c + 2], o[4 * c + 3]});
  } else {
    const u32 eo = 1024 + 16 * G + ((4 * G + 15) & ~15u);
    for (u32 i = lane * 16; i < 2 * E; i += 1024) bst128(rb, eo + i, lds_r128(reg + E_EXC + i));
  }
  cbar();
  u32 u = 0;
  if (lane == 0) u = __hip_atomic_fetch_add((GAS u32*)(a.ws[a.rank] + WS_TICKET), 1u, __ATOMIC_RELAXED, __HIP_MEMORY_SCOPE_AGENT);
  waitv<0>();
  u = rfl(u) & 2047;
  const u32x4 d = {call, n | (off << 16), (raw ? 0x80000000u : E) | (blob << 16), call};
  if (lane < NR) st_sys128(a.ws[lane] + WS_DESC + ((a.rank * 16 + (u & 15)) * NJOB + 32 + (u >> 4)) * 16, d);
}

DEV void gemm_role(const Args& a, LAS char* lds, int pos, int wid, u32 call, u64 t0) {
  char* const wsl = a.ws[a.rank];
  const int wr = wid >> 2, wc = wid & 3;
  for (int round = 0; round < 2; ++round) {
    int tid = threadIdx.x; asm volatile("" : "+v"(tid));
    int lane = tid & 63;
    const int tile = a.tiles[round * 128 + pos], mt = tile >> 4, nt = tile & 15;
    waitv<0>();
    __builtin_amdgcn_s_barrier();
    if (wid == 0) {
      const GAS u32* cp = (const GAS u32*)(wsl + WS_READY + ((lane & 1) ? 16 + nt : mt) * 128);
      u32 bad = 0;
      for (;;) {
        const u32 c = __hip_atomic_load(cp, __ATOMIC_RELAXED, __HIP_MEMORY_SCOPE_AGENT);
        if (__builtin_amdgcn_ballot_w64((int)(c - 128u * call) < 0) == 0) break;
        if (now() - t0 > DEADLINE) { bad = 1; break; }
        __builtin_amdgcn_s_sleep(8);
      }
      __builtin_amdgcn_fence(__ATOMIC_ACQUIRE, "agent");
      lds_post(LDS_BYTES - 16, bad);
      asm volatile("s_waitcnt vmcnt(0) lgkmcnt(0)" ::: "memory");
    }
    __builtin_amdgcn_s_barrier();
    const float init = lds_poll(LDS_BYTES - 16) ? __builtin_nanf("") : 0.f;
    STAMP_AT(2 + round * 3);
    f32x4 acc[2][2][4][2];
#pragma unroll
    for (int i0 = 0; i0 < 2; ++i0)
#pragma unroll
      for (int i1 = 0; i1 < 2; ++i1)
#pragma unroll
        for (int m = 0; m < 4; ++m)
#pragma unroll
          for (int n = 0; n < 2; ++n) acc[i0][i1][m][n] = (f32x4){init, init, init, init};
    bf16x8 At[4][2], B0[2][2], B1[2][2];
    const char* cA = wsl + WS_IMG + IMG_BYTES + (size_t)nt * (2u << 20);
    const char* cB = wsl + WS_IMG + (size_t)mt * (2u << 20);
    const u32 voff = tid * 16, ldsw = wid * 1024;
    const int aoff = wr * 8192 + lane * 16, boff = wc * 4096 + lane * 16;
    constexpr int HTB = 16384, KST = 32768, NT = 64;
#define SA(b, h) (((b) * 2 + (h)) * HTB)
#define SB(b, h) ((4 + (b) * 2 + (h)) * HTB)
#define STAGE(bufoff, gbase) do { _Pragma("unroll") for (int _i = 0; _i < 2; ++_i) \
    __builtin_amdgcn_global_load_lds((const GAS u32*)((gbase) + _i * 8192 + voff), (LAS u32*)(lds + (bufoff) + ldsw + _i * 8192), 16, 0, 0); } while (0)
#define LDA(dst, b, h) do { _Pragma("unroll") for (int m = 0; m < 4; ++m) _Pragma("unroll") for (int k = 0; k < 2; ++k) \
    dst[m][k] = *(const LAS bf16x8*)(lds + SA(b, h) + aoff + m * 2048 + k * 1024); } while (0)
#define LDB(dst, b, h) do { _Pragma("unroll") for (int n = 0; n < 2; ++n) _Pragma("unroll") for (int k = 0; k < 2; ++k) \
    dst[n][k] = *(const LAS bf16x8*)(lds + SB(b, h) + boff + n * 2048 + k * 1024); } while (0)
#define MMA(ai, bj, A_, B_) do { __builtin_amdgcn_s_setprio(1); _Pragma("unroll") for (int m = 0; m < 4; ++m) _Pragma("unroll") for (int n = 0; n < 2; ++n) \
    _Pragma("unroll") for (int k = 0; k < 2; ++k) acc[ai][bj][m][n] = __builtin_amdgcn_mfma_f32_16x16x32_bf16(A_[m][k], B_[n][k], acc[ai][bj][m][n], 0, 0, 0); \
    __builtin_amdgcn_s_setprio(0); } while (0)
#define WAIT_V(n) asm volatile("s_waitcnt vmcnt(" #n ")" ::: "memory")
#define WAIT_L(n) asm volatile("s_waitcnt lgkmcnt(" #n ")" ::: "memory")
#define BAR __builtin_amdgcn_s_barrier()
#define SCHED __builtin_amdgcn_sched_barrier(0)
    WAIT_V(0);
    STAGE(SB(0, 0), cB); STAGE(SA(0, 0), cA); STAGE(SB(0, 1), cB + HTB); STAGE(SA(0, 1), cA + HTB);
    if (wr == 1) BAR;
    WAIT_V(4); BAR;
    STAGE(SB(1, 0), cB + KST); STAGE(SA(1, 0), cA + KST); STAGE(SB(1, 1), cB + HTB + KST);
    WAIT_V(6); BAR;
    for (int t = 0; t < NT; t += 2) {
      const bool last = (t == NT - 2);
      const char* a1 = cA + (size_t)(t + 1) * KST;
      const char* a2 = last ? cA : cA + (size_t)(t + 2) * KST; const char* b2 = last ? cB : cB + (size_t)(t + 2) * KST;
      const char* a3 = a2 + KST; const char* b3 = b2 + KST;
      LDB(B0, 0, 0); SCHED; LDA(At, 0, 0); STAGE(SA(1, 1), a1 + HTB);
      WAIT_L(8); BAR; WAIT_L(0); MMA(0, 0, At, B0); BAR; SCHED;
      LDB(B1, 0, 1); STAGE(SB(0, 0), b2);
      BAR; WAIT_L(0); MMA(0, 1, At, B1); BAR;
      LDA(At, 0, 1); STAGE(SA(0, 0), a2);
      BAR; WAIT_L(0); MMA(1, 0, At, B0); BAR; SCHED;
      STAGE(SB(0, 1), b2 + HTB);
      WAIT_V(6); BAR; MMA(1, 1, At, B1); BAR;
      LDB(B0, 1, 0); SCHED; LDA(At, 1, 0); STAGE(SA(0, 1), a2 + HTB);
      WAIT_L(8); BAR; WAIT_L(0); MMA(0, 0, At, B0); BAR; SCHED;
      LDB(B1, 1, 1); STAGE(SB(1, 0), b3);
      BAR; WAIT_L(0); MMA(0, 1, At, B1); BAR;
      LDA(At, 1, 1); STAGE(SA(1, 0), a3);
      BAR; WAIT_L(0); MMA(1, 0, At, B0); BAR; SCHED;
      STAGE(SB(1, 1), b3 + HTB);
      WAIT_V(6); BAR; MMA(1, 1, At, B1); BAR;
    }
    WAIT_V(0);
    if (wr == 0) BAR;
    BAR;
#undef SA
#undef SB
#undef STAGE
#undef LDA
#undef LDB
#undef MMA
    int tz = threadIdx.x; asm volatile("" : "+v"(tz));
    lane = tz & 63;
    STAMP_AT(3 + round * 3);
    pack_out(a, acc, tile * 8 + wid, call, lds + wid * EREG, lane);
    STAMP_AT(4 + round * 3);
  }
}

DEV void loader_role(const Args& a, LAS char* lds, int peer, int lane, u64 t0) {
  const char* src = a.ws[peer] + WS_BLOB + lane * 16;
  u32 served = 0, rr = 0; int pend = -1; u32 pend_tag = 0;
  for (int done = 0; done < NJOB;) {
    const u32 r = lds_poll(L_REQ + (lane & 7) * 4);
    const u32 ready = (u32)__builtin_amdgcn_ballot_w64(lane < NDEC && (r & 255) != served);
    if (ready == 0) {
      if (pend >= 0) { waitv<0>(); lds_post(L_FULL + pend * 4, pend_tag); pend = -1; }
      else { if (now() - t0 > DEADLINE + 20000000ull) break; __builtin_amdgcn_s_sleep(2); }
      continue;
    }
    const u32 hi = ready >> rr;
    const int j = hi ? rr + __builtin_ctz(hi) : __builtin_ctz(ready);
    rr = j + 1;
    const u32 rq = (u32)__builtin_amdgcn_readlane((int)r, j);
    if (lane == j) served = rq & 255;
    ++done;
    const u32 slot = umin(rq >> 18, NSLOT - 1), bytes = (((rq >> 8) & 1023) + 1) * 16;
    const char* s = src + (size_t)slot * 16384;
    LAS char* dst = lds + j * DREG + 8192;
    int row = 0;
    for (u32 o = 0; o < bytes; o += 1024, ++row) {
      if (o + lane * 16 < bytes) __builtin_amdgcn_global_load_lds((const GAS u32*)(s + o), (LAS u32*)(dst + o), 16, 0, 17);
      waitv<WIN - 1>();
      if (row == WIN - 2 && pend >= 0) { lds_post(L_FULL + pend * 4, pend_tag); pend = -1; }
    }
    if (pend >= 0) { waitv<0>(); lds_post(L_FULL + pend * 4, pend_tag); }
    pend = j; pend_tag = rq & 255;
  }
  waitv<0>();
  if (pend >= 0) lds_post(L_FULL + pend * 4, pend_tag);
}

DEV void decoder_role(const Args& a, LAS char* lds, int peer, int c, int j, int lane, u32 call, u64 t0) {
  char* const wsl = a.ws[a.rank];
  LAS char* A = lds + j * DREG;
  LAS char* B = A + 8192;
  const char* dsc = wsl + WS_DESC + (size_t)((peer * 16 + c) * NJOB) * 16;
  bool fail = false;
  for (int m = j; m < NJOB; m += NDEC) {
    u32x4 d;
    for (;;) {
      d = ld_sys128(dsc + m * 16);
      if (rfl(d.x) == call && rfl(d.w) == call) break;
      if (now() - t0 > DEADLINE) { fail = true; break; }
      __builtin_amdgcn_s_sleep(4);
    }
    if (fail) break;
    const u32 da = rfl(d.y), db = rfl(d.z);
    const bool raw = db >> 31;
    u32 slot, bytes, E, G = 0;
    if (m < 32) {
      E = umin(da >> 16, IN_EXC_CAP);
      slot = ((m & 1) ? SLOT_W + a.rank * 256 : 0) + (m >> 1) * 16 + c;
      bytes = 11264 + ((2 * E + 15) & ~15u);
    } else {
      E = umin(db & 0xFFFF, OUT_EXC_CAP);
      G = (umin(da & 0xFFFF, 8192) + 15) >> 4;
      slot = SLOT_O + ((db >> 16) & 2047);
      bytes = 1024 + 16 * G + ((4 * G + 15) & ~15u) + ((2 * E + 15) & ~15u);
    }
    if (raw) bytes = 16384;
    lds_post(L_REQ + j * 4, (slot << 18) | ((bytes / 16 - 1) << 8) | (u32)(m + 1));
    for (;;) {
      if (rfl(lds_poll(L_FULL + j * 4)) == (u32)(m + 1)) break;
      if (now() - t0 > DEADLINE) { fail = true; break; }
      __builtin_amdgcn_s_sleep(2);
    }
    if (fail) break;
    if (m < 32) {
      const u32 base = da & 0xFFFF, bb = base * 0x10001u, upz = ((65536 - base) >> 11) & 31;
      u32 H[32];
      if (raw) {
#pragma unroll
        for (int r = 0; r < 8; ++r) { const u32x4 t = lds_r128(B + (8 + r) * 1024 + lane * 16); H[4 * r] = t.x; H[4 * r + 1] = t.y; H[4 * r + 2] = t.z; H[4 * r + 3] = t.w; }
      } else {
        for (u32 i = lane; i < E; i += 64) {
          const u32 en = *(LAS u16*)(B + 11264 + 2 * i), cc = en & 7;
          *(LAS unsigned char*)(A + (en >> 3)) = (unsigned char)((cc ? 32 - cc : upz) << 3);
        }
        cbar();
        u32 P[12];
#pragma unroll
        for (int r = 0; r < 3; ++r) { const u32x4 t = lds_r128(B + (8 + r) * 1024 + lane * 16); P[4 * r] = t.x; P[4 * r + 1] = t.y; P[4 * r + 2] = t.z; P[4 * r + 3] = t.w; }
#pragma unroll
        for (int r = 0; r < 8; ++r) { const u32x4 t = lds_r128(A + r * 1024 + lane * 16); H[4 * r] = t.x; H[4 * r + 1] = t.y; H[4 * r + 2] = t.z; H[4 * r + 3] = t.w; }
        cbar();
        for (u32 i = lane; i < E; i += 64) *(LAS unsigned char*)(A + (*(LAS u16*)(B + 11264 + 2 * i) >> 3)) = 0;
        cbar();
#pragma unroll
        for (int g = 0; g < 4; ++g) {
          u32* h = H + 8 * g; const u32 p0 = P[3 * g], p1 = P[3 * g + 1], p2 = P[3 * g + 2], m7 = 0x07070707u;
          h[0] |= p0 & m7; h[1] |= (p0 >> 3) & m7; h[2] |= ((p0 >> 6) & 0x03030303u) | ((p2 >> 4) & 0x04040404u);
          h[3] |= p1 & m7; h[4] |= (p1 >> 3) & m7; h[5] |= ((p1 >> 6) & 0x03030303u) | ((p2 >> 5) & 0x04040404u);
          h[6] |= p2 & m7; h[7] |= (p2 >> 3) & m7;
        }
      }
      const int t = m & 1, s = m >> 1;
      const u32 so = t * IMG_BYTES + (u32)(s * 64 + peer * 8 + (c >> 1)) * 32768u + (c & 1) * (t ? 8192 : 16384);
      const rsrc_t rimg[2] = {mkrsrc(wsl + WS_IMG + so, 8192), mkrsrc(wsl + WS_IMG + so + (t ? 16384 : 8192), 8192)};
#pragma unroll
      for (int r = 0; r < 8; ++r) {
        const u32x4 lo = lds_r128(B + r * 1024 + lane * 16);
        u32 v[8];
#pragma unroll
        for (int k = 0; k < 4; ++k) {
          const u32 u0 = pk_add(perm(H[4 * r + k], lo[k], 0x05010400u), bb), u1 = pk_add(perm(H[4 * r + k], lo[k], 0x07030602u), bb);
          v[2 * k] = __builtin_amdgcn_alignbit(u0, u0, 1); v[2 * k + 1] = __builtin_amdgcn_alignbit(u1, u1, 1);
        }
        bst128(rimg[r >> 2], lane * 16 + ((2 * r) & 7) * 1024, (u32x4){v[0], v[1], v[2], v[3]});
        bst128(rimg[r >> 2], lane * 16 + ((2 * r + 1) & 7) * 1024, (u32x4){v[4], v[5], v[6], v[7]});
      }
      waitv<0>();
      if (lane == 0) __hip_atomic_fetch_add((GAS u32*)(wsl + WS_READY + (t * 16 + s) * 128), 1u, __ATOMIC_RELAXED, __HIP_MEMORY_SCOPE_AGENT);
    } else {
      const u32 blob = (db >> 16) & 2047, tile = blob >> 3, w8 = blob & 7;
      const u32 so = ((tile >> 4) * 256 + (w8 & 3) * 32) * 65536u + (peer * 4096 + (tile & 15) * 256 + (w8 >> 2) * 128) * 2u;
      const u32 vo = (lane >> 2) * 65536u + (lane & 3) * 16;
      const rsrc_t rout = mkrsrc(a.out + so, OUT_BYTES - so);
      if (raw) {
#pragma unroll
        for (int ch = 0; ch < 16; ++ch)
          __builtin_amdgcn_raw_buffer_store_b128(lds_r128(B + ch * 1024 + lane * 16), rout, vo + (ch & 3) * 64 + ((ch >> 3) * 128 + ((ch >> 2) & 1) * 16) * 65536u, 0, OUT_AUX);
      } else {
        const u32 offp = ((da >> 16) & 0x7FFF) * 0x10001u;
        const u32x4 bits = lds_r128(B + lane * 16);
        cbar();
        for (u32 g = lane; g < G; g += 64) {
          const u32x4 lo = lds_r128(B + 1024 + 16 * g); const u32 code = *(LAS u32*)(B + 1024 + 16 * G + 4 * g);
          u32 v[8];
#pragma unroll
          for (int k = 0; k < 4; ++k) {
            const u32 h = (code >> (2 * k)) & 0x03030303u;
            v[2 * k] = pk_add(perm(h, lo[k], 0x05010400u), offp); v[2 * k + 1] = pk_add(perm(h, lo[k], 0x07030602u), offp);
          }
          lds_w128(A + 32 * g, (u32x4){v[0], v[1], v[2], v[3]}); lds_w128(A + 32 * g + 16, (u32x4){v[4], v[5], v[6], v[7]});
          cbar();
        }
        LAS char* ex = B + 1024 + 16 * G + ((4 * G + 15) & ~15u);
        for (u32 i = lane; i < E; i += 64) {
          const u32 en = *(LAS u16*)(ex + 2 * i); LAS u16* p = (LAS u16*)(A + 2 * (en >> 3));
          *p = (u16)(*p - ((en & 7) << 10));
        }
        cbar();
        const u32 cnt = __builtin_popcount(bits.x) + __builtin_popcount(bits.y) + __builtin_popcount(bits.z) + __builtin_popcount(bits.w);
        u32 pos = wave_scan(cnt) - cnt;
#pragma unroll
        for (int ch = 0; ch < 16; ++ch) {
          u32x4 ov;
#pragma unroll
          for (int h = 0; h < 2; ++h) {
            const u32 nib = (bits[ch >> 2] >> (28 - 4 * ((ch & 3) * 2 + h))) & 15;
            const u32x2 sel = *(LAS u32x2*)(lds + L_LUT + nib * 8);
            const u32 ad = 2 * pos; LAS u32* q = (LAS u32*)(A + (ad & ~3u));
            const u32 d0 = q[0], d1 = q[1], d2 = q[2];
            const u32 e0 = __builtin_amdgcn_alignbyte(d1, d0, ad), e1 = __builtin_amdgcn_alignbyte(d2, d1, ad);
            ov[2 * h] = perm(e1, e0, sel.x); ov[2 * h + 1] = perm(e1, e0, sel.y);
            pos += __builtin_popcount(nib);
          }
          __builtin_amdgcn_raw_buffer_store_b128(ov, rout, vo + (ch & 3) * 64 + ((ch >> 3) * 128 + ((ch >> 2) & 1) * 16) * 65536u, 0, OUT_AUX);
        }
      }
      cbar();
    }
  }
  if (fail) {
    waitv<0>();
    __builtin_amdgcn_raw_buffer_store_b32(0x7FC07FC0u, mkrsrc(a.out, OUT_BYTES), lane * 4 + (blockIdx.x * 8 + j) * 65536u, 0, 0);
    if (lane == 0) __hip_atomic_store((GAS u32*)(wsl + WS_TMO), call, __ATOMIC_RELAXED, __HIP_MEMORY_SCOPE_AGENT);
  }
}

__global__ void __launch_bounds__(512, 2) fused_gemm_ar(const Args a) {
  extern __shared__ __attribute__((aligned(16))) char smem[];
  LAS char* lds = (LAS char*)smem;
  const u64 t0 = now();
  const int wid = __builtin_amdgcn_readfirstlane(threadIdx.x >> 6), lane = threadIdx.x & 63;
  const int xcd = blockIdx.x & 7, li = blockIdx.x >> 3;
  char* const wsl = a.ws[a.rank];
  GAS u32* callp = (GAS u32*)(wsl + WS_CALL) + blockIdx.x * 8 + wid;
  const u32 callv = __hip_atomic_load(callp, __ATOMIC_RELAXED, __HIP_MEMORY_SCOPE_AGENT);
  char* const wsp = a.ws[lane & 7];
  STAMP_AT(0);
  const bool is_gemm = li < 16;
  const int pos = xcd * 16 + (li & 15);
  if (!is_gemm) {
    if (wid == 0) {
      if (lane < 32) *(LAS u32*)(lds + L_CTRL + lane * 4) = 0;
      if (lane < 16) {
        u32 k = 0, sel[2] = {0, 0};
#pragma unroll
        for (int i = 0; i < 4; ++i) {
          const bool on = (lane >> (3 - i)) & 1;
          sel[i >> 1] |= (on ? (2 * k) | ((2 * k + 1) << 8) : 0x0C0Cu) << (16 * (i & 1));
          k += on;
        }
        *(LAS u32x2*)(lds + L_LUT + lane * 8) = (u32x2){sel[0], sel[1]};
      }
    } else if (wid <= NDEC) {
#pragma unroll
      for (int r = 0; r < 8; ++r) lds_w128(lds + (wid - 1) * DREG + r * 1024 + lane * 16, (u32x4){0, 0, 0, 0});
    }
    asm volatile("s_waitcnt lgkmcnt(0)" ::: "memory");
    __builtin_amdgcn_s_barrier();
  }
  {
    int e, n;
    if (is_gemm) { e = PREPW * 128 + wid * 128 + pos; n = 2; }
    else { e = (wid - 1) * 128 + pos; n = (wid >= 1 && wid <= PREPW) ? 1 : 0; }
    LAS char* scr = is_gemm ? lds + wid * EREG : lds + (wid - 1) * DREG + 8192;
    for (int i = 0; i < n; ++i) {
      if (e < NPIECE) prep_piece(a, e, callv, wsp, scr, lane);
      e = PREPW * 128 + 1024 + (127 - pos) * 8 + wid;
    }
  }
  const u32 call = rfl(callv) + 1;
  if (lane == 0) __hip_atomic_store(callp, call, __ATOMIC_RELAXED, __HIP_MEMORY_SCOPE_AGENT);
  STAMP_AT(1);
  if (is_gemm) {
    gemm_role(a, lds, pos, wid, call, t0);
  } else {
    const int q = li - 16, peer = (a.rank + (q & 7)) & 7, c = xcd * 2 + (q >> 3);
    if (wid == 0) loader_role(a, lds, peer, lane, t0);
    else if (wid <= NDEC) decoder_role(a, lds, peer, c, wid - 1, lane, call, t0);
  }
  STAMP_AT(8);
}

#define HIP_CHECK(expr) do { hipError_t _e = (expr); if (_e != hipSuccess) { \
    fprintf(stderr, "HIP error %s at %s:%d\n", hipGetErrorString(_e), __FILE__, __LINE__); \
    return; } } while (0)

#define DBG_FLAGS 0

extern "C" void kernel_launch(int rank, int nranks, ncclComm_t comm, hipStream_t stream,
                              void* const* d_in, const int* in_sizes, int n_in,
                              void* d_out, int out_size, const int* in_dtypes, int out_dtype,
                              void* const* d_out_all, void* const* d_ws_all, size_t ws_bytes) {
  if (nranks != NR || (unsigned)rank >= (unsigned)NR || n_in < 2 || ws_bytes < WS_USED || (size_t)out_size * 2 != OUT_BYTES ||
      in_sizes[0] != 4096 * 512 || in_sizes[1] != 512 * 32768) {
    fprintf(stderr, "unexpected problem shape\n");
    return;
  }
  static bool attr_set[NR] = {};
  if (!attr_set[rank]) {
    HIP_CHECK(hipFuncSetAttribute((const void*)fused_gemm_ar, hipFuncAttributeMaxDynamicSharedMemorySize, LDS_BYTES));
    attr_set[rank] = true;
  }
  Args a;
  a.x = (const char*)d_in[0]; a.w = (const char*)d_in[1]; a.out = (char*)d_out;
  for (int p = 0; p < NR; ++p) a.ws[p] = (char*)d_ws_all[p];
  a.rank = rank; a.flags = DBG_FLAGS;
  int k = 0;
  for (int s = 0; s < 12; ++s) {
    for (int i = 0; i <= s; ++i) a.tiles[k++] = (unsigned char)(s * 16 + i);
    for (int i = 0; i < s; ++i) a.tiles[k++] = (unsigned char)(i * 16 + s);
  }
  static const int blk[7][2] = {{3, 0}, {0, 3}, {3, 1}, {1, 3}, {3, 2}, {2, 3}, {3, 3}};
  for (int b = 0; b < 7; ++b)
    for (int i = 0; i < 16; ++i) a.tiles[k++] = (unsigned char)((blk[b][0] * 4 + (i >> 2)) * 16 + blk[b][1] * 4 + (i & 3));
  hipLaunchKernelGGL(fused_gemm_ar, dim3(256), dim3(512), LDS_BYTES, stream, a);
  HIP_CHECK(hipGetLastError());
}
```

```cpp
#include <hip/hip_runtime.h>
#include <rccl/rccl.h>
#include <cstdio>

typedef unsigned u32;
typedef unsigned long long u64;
typedef u32 u32x4 __attribute__((ext_vector_type(4)));
typedef u32 u32x2 __attribute__((ext_vector_type(2)));
typedef unsigned short u16;
typedef u16 u16x2 __attribute__((ext_vector_type(2)));
typedef short bf16x8 __attribute__((ext_vector_type(8)));
typedef float f32x4 __attribute__((ext_vector_type(4)));
typedef float f32x2 __attribute__((ext_vector_type(2)));
typedef __bf16 bf16x2 __attribute__((ext_vector_type(2)));
#define LAS __attribute__((address_space(3)))
#define GAS __attribute__((address_space(1)))
#define DEV __device__ __forceinline__

constexpr int NR = 8;
constexpr u32 WS_CALL = 0;
constexpr u32 WS_TMO = 8192;
constexpr u32 WS_TICKET = 8192 + 128;
constexpr u32 WS_QUEUE = 8192 + 256;
constexpr u32 WS_READY = 16384;
constexpr int NIN = 32;
constexpr u32 WS_DESC = 512u << 10;
constexpr u32 WS_DESCO = WS_DESC + 65536;
constexpr u32 WS_IMG = 1u << 20;
constexpr u32 IMG_BYTES = 32u << 20;
constexpr u32 WS_BLOB = 65u << 20;
constexpr u32 SLOT_W = 256, SLOT_O = 2304, NSLOT = 4352;
constexpr u32 WS_USED = WS_BLOB + NSLOT * 16384u;
constexpr u32 WS_STAMP = WS_USED;
constexpr u32 OUT_BYTES = 4096u * 32768u * 2u;
constexpr u64 DEADLINE = 100000000ull;
constexpr int LDS_BYTES = 163840;
constexpr int NDEC = 6;
constexpr int DREG = 26624;
constexpr int L_CTRL = NDEC * DREG;
constexpr int L_REQ = L_CTRL, L_FULL = L_CTRL + 64, L_LUT = L_CTRL + 128, L_SCR7 = L_CTRL + 256;
constexpr int EREG = 20480;
constexpr int E_EXC = 16448;
constexpr u32 IN_EXC_CAP = 1792, OUT_EXC_CAP = 1984;
constexpr u32 QSTEP = 2048 + 16 * NDEC;
#define WIN 7
#define GDELAY 0
#define OWN_AUX 0
#define OUT_AUX 0

struct Args {
  const char* x; const char* w; char* out; char* ws[NR]; int rank; int flags; unsigned char tiles[256];
};

template <int N> DEV void waitv() { asm volatile("s_waitcnt vmcnt(%0)" ::"n"(N) : "memory"); }
DEV void cbar() { asm volatile("" ::: "memory"); }
DEV u64 now() { return __builtin_amdgcn_s_memrealtime(); }
DEV u32 perm(u32 hi, u32 lo, u32 sel) { return __builtin_amdgcn_perm(hi, lo, sel); }
DEV u32 pk_sub(u32 a, u32 b) { return __builtin_bit_cast(u32, __builtin_bit_cast(u16x2, a) - __builtin_bit_cast(u16x2, b)); }
DEV u32 pk_add(u32 a, u32 b) { return __builtin_bit_cast(u32, __builtin_bit_cast(u16x2, a) + __builtin_bit_cast(u16x2, b)); }
DEV u32 pk_max(u32 a, u32 b) { return __builtin_bit_cast(u32, __builtin_elementwise_max(__builtin_bit_cast(u16x2, a), __builtin_bit_cast(u16x2, b))); }
DEV u32 pk_min(u32 a, u32 b) { return __builtin_bit_cast(u32, __builtin_elementwise_min(__builtin_bit_cast(u16x2, a), __builtin_bit_cast(u16x2, b))); }
template <int C, int RM> DEV u32 dpp0(u32 x) { return (u32)__builtin_amdgcn_update_dpp(0, (int)x, C, RM, 0xf, false); }
DEV u32 umax(u32 a, u32 b) { return a > b ? a : b; }
DEV u32 umin(u32 a, u32 b) { return a < b ? a : b; }
DEV u32 wave_max(u32 x) {
  x = umax(x, dpp0<0x111, 0xf>(x)); x = umax(x, dpp0<0x112, 0xf>(x)); x = umax(x, dpp0<0x114, 0xf>(x)); x = umax(x, dpp0<0x118, 0xf>(x));
  x = umax(x, dpp0<0x142, 0xa>(x)); x = umax(x, dpp0<0x143, 0xc>(x));
  return (u32)__builtin_amdgcn_readlane((int)x, 63);
}
DEV u32 wave_scan(u32 x) {
  x += dpp0<0x111, 0xf>(x); x += dpp0<0x112, 0xf>(x); x += dpp0<0x114, 0xf>(x); x += dpp0<0x118, 0xf>(x);
  x += dpp0<0x142, 0xa>(x); x += dpp0<0x143, 0xc>(x);
  return x;
}
DEV u32 rfl(u32 x) { return (u32)__builtin_amdgcn_readfirstlane((int)x); }
DEV u32 mbcnt(u64 m) { return __builtin_amdgcn_mbcnt_hi((u32)(m >> 32), __builtin_amdgcn_mbcnt_lo((u32)m, 0)); }
DEV u32x4 lds_r128(LAS char* p) { return *(LAS u32x4*)p; }
DEV void lds_w128(LAS char* p, u32x4 v) { *(LAS u32x4*)p = v; }
DEV u32 lds_poll(u32 addr) { u32 v; asm volatile("ds_read_b32 %0, %1\n\ts_waitcnt lgkmcnt(0)" : "=v"(v) : "v"(addr) : "memory"); return v; }
DEV void lds_post(u32 addr, u32 v) { asm volatile("ds_write_b32 %0, %1" ::"v"(addr), "v"(v) : "memory"); }
DEV void st_sys128(void* p, u32x4 v) { asm volatile("global_store_dwordx4 %0, %1, off sc0 sc1\n\ts_nop 1" ::"v"(p), "v"(v) : "memory"); }
DEV u32x4 ld_sys128(const void* p) { u32x4 v; asm volatile("global_load_dwordx4 %0, %1, off sc0 sc1\n\ts_waitcnt vmcnt(0)" : "=&v"(v) : "v"(p) : "memory"); return v; }
typedef __amdgpu_buffer_rsrc_t rsrc_t;
DEV rsrc_t mkrsrc(const void* p, u32 bytes) { return __builtin_amdgcn_make_buffer_rsrc((void*)p, 0, (int)bytes, 0x00020000); }
DEV void bst128(rsrc_t r, u32 voff, u32x4 v) { __builtin_amdgcn_raw_buffer_store_b128(v, r, (int)voff, 0, 17); }
DEV void bst32(rsrc_t r, u32 voff, u32 v) { __builtin_amdgcn_raw_buffer_store_b32(v, r, (int)voff, 0, 17); }

#define STAMP_AT(k) do { } while (0)

DEV void emit_exc(const u32 (&U)[4], bool act, u32 gidx, u32 upz, LAS char* list, u32 cap, u32& E, bool& bad) {
  u32 F = (((U[0] + 0x7C7C7C7Cu) & 0x80808080u) >> 7) | (((U[1] + 0x7C7C7C7Cu) & 0x80808080u) >> 6) |
          (((U[2] + 0x7C7C7C7Cu) & 0x80808080u) >> 5) | (((U[3] + 0x7C7C7C7Cu) & 0x80808080u) >> 4);
  if (!act) F = 0;
  for (;;) {
    const u64 bal = __builtin_amdgcn_ballot_w64(F != 0);
    if (bal == 0) break;
    if (F != 0) {
      const u32 p = __builtin_ctz(F), k = p & 7, b = p >> 3;
      const u32 h0 = (p & 1) ? U[1] : U[0], h1 = (p & 1) ? U[3] : U[2], h = (p & 2) ? h1 : h0;
      const u32 up = (h >> (8 * b + 2)) & 31;
      u32 c = 32 - up;
      if (c > 7) { bad |= up != upz; c = 0; }
      const u32 slot = E + mbcnt(bal);
      if (slot < cap) *(LAS u16*)(list + 2 * slot) = (u16)(((gidx + 4 * k + b) << 3) | c);
      F &= F - 1;
    }
    E += __builtin_popcountll(bal);
  }
}

DEV void prep_piece(const Args& a, bool isx, int s, int j, int h, int dest, u32 callv, char* wsp, LAS char* scr, int lane) {
  const int fr = lane & 15, fq = lane >> 4;
  const int piece = j * 2 + h;
  const u32 slot = (isx ? 0 : SLOT_W + dest * 256) + s * 16 + piece;
  u32 v[64];
  {
    const rsrc_t ri = mkrsrc(isx ? a.x : a.w, isx ? 4096u * 512u * 2u : 512u * 32768u * 2u);
    const u32 lv = isx ? fr * 1024 + fq * 16 : fq * 8 * 65536 + 16 * (4 * (fr & 3) + (fr >> 2));
    const u32 ls = isx ? (s * 256 + (piece & 1) * 128) * 1024 + (piece >> 1) * 128
                       : (piece >> 1) * 64 * 65536 + (dest * 4096 + s * 256 + (piece & 1) * 128) * 2;
    const u32 s8 = isx ? 65536 : 32 * 65536, s2 = isx ? 16384 : 2 * 65536, s1 = isx ? 64 : 65536;
    u32x4 in[16];
#pragma unroll
    for (int i = 0; i < 16; ++i) in[i] = __builtin_amdgcn_raw_buffer_load_b128(ri, lv, ls + (i >> 3) * s8 + ((i >> 1) & 3) * s2 + (i & 1) * s1, 2);
    __builtin_amdgcn_sched_barrier(0);
    if (isx) {
#pragma unroll
      for (int i = 0; i < 64; ++i) v[i] = in[i >> 2][i & 3];
    } else {
#pragma unroll
      for (int ks = 0; ks < 2; ++ks)
#pragma unroll
        for (int el = 0; el < 8; ++el)
#pragma unroll
          for (int d = 0; d < 4; ++d)
            v[4 * (el * 2 + ks) + d] = perm(in[ks * 8 + 2 * d + 1][el >> 1], in[ks * 8 + 2 * d][el >> 1], (el & 1) ? 0x07060302u : 0x05040100u);
    }
  }
  GAS u32* const own = (GAS u32*)(a.ws[a.rank] + WS_READY + ((isx ? 0 : 4) + (s >> 2)) * 9 * 128);
  if (isx || dest == a.rank) {
    const u32 so = (isx ? 0 : IMG_BYTES) + (u32)(s * 64 + j) * 32768u + h * (isx ? 16384 : 8192);
    const rsrc_t rimg[2] = {mkrsrc(a.ws[a.rank] + WS_IMG + so, 8192), mkrsrc(a.ws[a.rank] + WS_IMG + so + (isx ? 8192 : 16384), 8192)};
#pragma unroll
    for (int i = 0; i < 16; ++i) bst128(rimg[i >> 3], lane * 16 + (i & 7) * 1024, (u32x4){v[4 * i], v[4 * i + 1], v[4 * i + 2], v[4 * i + 3]});
    if (!isx) {
      waitv<0>();
      if (lane == 0) __hip_atomic_fetch_add(own, 1u, __ATOMIC_RELAXED, __HIP_MEMORY_SCOPE_AGENT);
      return;
    }
  }
  u32 mx = 0;
#pragma unroll
  for (int i = 0; i < 64; ++i) { v[i] = __builtin_amdgcn_alignbit(v[i], v[i], 31); mx = pk_max(mx, v[i]); }
  const u32 um = wave_max(umax(mx & 0xFFFF, mx >> 16)) | 1;
  const u32 base = um > 2047 ? um - 2047 : 0, bb = base * 0x10001u, upz = ((65536 - base) >> 11) & 31;
  const rsrc_t rb = mkrsrc(a.ws[a.rank] + WS_BLOB + slot * 16384u, 16384u);
  const u32 vo = lane * 16;
  u32 H[32];
#pragma unroll
  for (int r = 0; r < 8; ++r) {
    u32x4 lo;
#pragma unroll
    for (int k = 0; k < 4; ++k) {
      const u32 t0 = pk_sub(v[8 * r + 2 * k], bb), t1 = pk_sub(v[8 * r + 2 * k + 1], bb);
      lo[k] = perm(t1, t0, 0x06040200u); H[4 * r + k] = perm(t1, t0, 0x07050301u);
    }
    bst128(rb, vo + r * 1024, lo);
  }
  u32 E = 0; bool bad = false;
#pragma unroll
  for (int r = 0; r < 8; ++r) {
    u32 U[4];
#pragma unroll
    for (int k = 0; k < 4; ++k) U[k] = (H[4 * r + k] >> 1) & 0x7C7C7C7Cu;
    emit_exc(U, true, (r * 64 + lane) * 16, upz, scr, IN_EXC_CAP, E, bad);
  }
  const bool raw = __builtin_amdgcn_ballot_w64(bad) != 0 || E > IN_EXC_CAP || (a.flags & 1);
  if (raw) {
#pragma unroll
    for (int r = 0; r < 8; ++r) bst128(rb, vo + (8 + r) * 1024, (u32x4){H[4 * r], H[4 * r + 1], H[4 * r + 2], H[4 * r + 3]});
  } else {
    u32 P[12];
#pragma unroll
    for (int g = 0; g < 4; ++g) {
      const u32* h = H + 8 * g; const u32 m7 = 0x07070707u;
      P[3 * g] = (h[0] & m7) | ((h[1] & m7) << 3) | ((h[2] & 0x03030303u) << 6);
      P[3 * g + 1] = (h[3] & m7) | ((h[4] & m7) << 3) | ((h[5] & 0x03030303u) << 6);
      P[3 * g + 2] = (h[6] & m7) | ((h[7] & m7) << 3) | ((h[2] & 0x04040404u) << 4) | ((h[5] & 0x04040404u) << 5);
    }
#pragma unroll
    for (int r = 0; r < 3; ++r) bst128(rb, vo + (8 + r) * 1024, (u32x4){P[4 * r], P[4 * r + 1], P[4 * r + 2], P[4 * r + 3]});
    cbar();
    for (u32 i = vo; i < 2 * E; i += 1024) bst128(rb, 11264 + i, lds_r128(scr + i));
    cbar();
  }
  waitv<0>();
  asm volatile("" : "+v"(callv), "+v"(wsp));
  const u32x4 d = {callv + 1, base | (raw ? 0u : E << 16), raw ? 0x80000000u : 0u, callv + 1};
  const int c = (s < 8 ? (s & 3) + (isx ? 0 : 4) : s - 8) * 2 + h, m = s < 8 ? j * 3 + (s >> 2) : isx ? j * 3 + 2 : 24 + j;
  const u32 doff = WS_DESC + ((a.rank * 16 + c) * NIN + m) * 16;
  if (lane < NR && lane != a.rank && (isx || lane == dest)) st_sys128(wsp + doff, d);
  if (isx && lane == 0) __hip_atomic_fetch_add(own, 1u, __ATOMIC_RELAXED, __HIP_MEMORY_SCOPE_AGENT);
}

DEV void pack_out(const Args& a, f32x4 (&acc)[2][2][4][2], u32 blob, u32 call, LAS char* reg, int lane) {
  u32 o[64];
#pragma unroll
  for (int bj = 0; bj < 2; ++bj)
#pragma unroll
    for (int n = 0; n < 2; ++n)
#pragma unroll
      for (int j = 0; j < 4; ++j)
#pragma unroll
        for (int d = 0; d < 4; ++d) {
          const f32x2 y = {__builtin_elementwise_maximum(acc[d >> 1][bj][(d & 1) * 2][n][j], 0.f),
                           __builtin_elementwise_maximum(acc[d >> 1][bj][(d & 1) * 2 + 1][n][j], 0.f)};
          o[((bj * 2 + n) * 4 + j) * 4 + d] = __builtin_bit_cast(u32, __builtin_convertvector(y, bf16x2)) & 0x7FFF7FFFu;
        }
  u32 mx = 0; u32x4 bits;
#pragma unroll
  for (int q = 0; q < 4; ++q) {
    u32 m16[2];
#pragma unroll
    for (int h = 0; h < 2; ++h) {
      u32 t = 0;
#pragma unroll
      for (int i = 0; i < 8; ++i) { const u32 x = o[q * 16 + h * 8 + i]; mx = pk_max(mx, x); t = (t << 2) | pk_min(x, 0x00010001u); }
      m16[h] = ((t & 0xFFFF) << 1) | (t >> 16);
    }
    bits[q] = (m16[0] << 16) | m16[1];
  }
  const u32 vmax = wave_max(umax(mx & 0xFFFF, mx >> 16));
  const u32 off = vmax > 1023 ? vmax - 1023 : 0, offp = off * 0x10001u;
  const u32 cnt = __builtin_popcount(bits.x) + __builtin_popcount(bits.y) + __builtin_popcount(bits.z) + __builtin_popcount(bits.w);
  const u32 dl = (lane & 15) * 4 + (lane >> 4);
  const u32 cd = (u32)__builtin_amdgcn_ds_bpermute((((lane & 3) * 16 + (lane >> 2)) * 4), (int)cnt);
  const u32 inc = wave_scan(cd);
  const u32 n = (u32)__builtin_amdgcn_readlane((int)inc, 63);
  const u32 start = (u32)__builtin_amdgcn_ds_bpermute((int)(dl * 4), (int)(inc - cd));
  const rsrc_t rb = mkrsrc(a.ws[a.rank] + WS_BLOB + (SLOT_O + blob) * 16384u, 16384u);
  bst128(rb, dl * 16, bits);
  {
    u32 pos = 2 * start; const u32 dump = E_EXC + lane * 4;
#pragma unroll
    for (int i = 0; i < 64; ++i) {
      const u32 x = o[i]; const bool z0 = (x & 0xFFFF) != 0, z1 = x > 0xFFFF;
      *(LAS u16*)(reg + (z0 ? pos : dump)) = (u16)x; pos += z0 ? 2 : 0;
      *(LAS u16*)(reg + (z1 ? pos : dump)) = (u16)(x >> 16); pos += z1 ? 2 : 0;
    }
    if (lane < 16) *(LAS u16*)(reg + 2 * (n + lane)) = (u16)off;
  }
  cbar();
  const u32 G = (n + 15) >> 4;
  u32 E = 0; bool bad = false;
  for (u32 g0 = 0; g0 < G; g0 += 64) {
    const u32 g = g0 + lane; const bool act = g < G;
    const u32x4 t0 = lds_r128(reg + 32 * g), t1 = lds_r128(reg + 32 * g + 16);
    u32 t[8] = {t0.x, t0.y, t0.z, t0.w, t1.x, t1.y, t1.z, t1.w}, U[4], code = 0; u32x4 lo;
#pragma unroll
    for (int k = 0; k < 4; ++k) {
      const u32 p0 = pk_sub(t[2 * k], offp), p1 = pk_sub(t[2 * k + 1], offp), h = perm(p1, p0, 0x07050301u);
      lo[k] = perm(p1, p0, 0x06040200u); code |= (h & 0x03030303u) << (2 * k); U[k] = h & 0x7C7C7C7Cu;
    }
    if (act) { bst128(rb, 1024 + 16 * g, lo); bst32(rb, 1024 + 16 * G + 4 * g, code); }
    emit_exc(U, act, g * 16, 99, reg + E_EXC, OUT_EXC_CAP, E, bad);
  }
  cbar();
  const bool raw = __builtin_amdgcn_ballot_w64(bad) != 0 || E > OUT_EXC_CAP || (a.flags & 2);
  if (raw) {
    waitv<0>();
#pragma unroll
    for (int c = 0; c < 16; ++c) bst128(rb, c * 1024 + dl * 16, (u32x4){o[4 * c], o[4 * c + 1], o[4 * c + 2], o[4 * c + 3]});
  } else {
    const u32 eo = 1024 + 16 * G + ((4 * G + 15) & ~15u);
    for (u32 i = lane * 16; i < 2 * E; i += 1024) bst128(rb, eo + i, lds_r128(reg + E_EXC + i));
  }
  cbar();
  u32 u = 0;
  if (lane == 0) u = __hip_atomic_fetch_add((GAS u32*)(a.ws[a.rank] + WS_TICKET), 1u, __ATOMIC_RELAXED, __HIP_MEMORY_SCOPE_AGENT);
  waitv<0>();
  u = rfl(u) & 2047;
  const u32x4 d = {call, n | (off << 16), (raw ? 0x80000000u : E) | (blob << 16), call};
  if (lane < NR && lane != a.rank) st_sys128(a.ws[lane] + WS_DESCO + (a.rank * 2048 + u) * 16, d);
  const u32 tile = blob >> 3, w8 = blob & 7;
  const u32 so = ((tile >> 4) * 256 + (w8 & 3) * 32) * 65536u + (a.rank * 4096 + (tile & 15) * 256 + (w8 >> 2) * 128) * 2u;
  const rsrc_t rout = mkrsrc(a.out + so, OUT_BYTES - so);
  const u32 vo = (lane & 15) * 65536u + (lane >> 4) * 16;
#pragma unroll
  for (int ch = 0; ch < 16; ++ch)
    __builtin_amdgcn_raw_buffer_store_b128((u32x4){o[4 * ch], o[4 * ch + 1], o[4 * ch + 2], o[4 * ch + 3]}, rout,
                                           vo + (ch & 3) * 64 + ((ch >> 3) * 128 + ((ch >> 2) & 1) * 16) * 65536u, 0, OWN_AUX);
}

DEV void wait_ready(const GAS u32* cp, u32 need, u64 t0, u32& bad) {
  while (!bad) {
    const u32 c = __hip_atomic_load(cp, __ATOMIC_RELAXED, __HIP_MEMORY_SCOPE_AGENT);
    if (__builtin_amdgcn_ballot_w64((int)(c - need) < 0) == 0) break;
    if (now() - t0 > DEADLINE) bad = 1;
    __builtin_amdgcn_s_sleep(4);
  }
  cbar();
}
DEV void gemm_role(const Args& a, LAS char* lds, int pos, int wid, u32 call, u64 t0) {
  char* const wsl = a.ws[a.rank];
  const int wr = wid >> 2, wc = wid & 3;
  u32 bad = 0;
  for (int round = 0; round < 2; ++round) {
    int tid = threadIdx.x; asm volatile("" : "+v"(tid));
    int lane = tid & 63;
    const int tile = a.tiles[round * 128 + pos], mt = tile >> 4, nt = tile & 15;
    const GAS u32* cp = (const GAS u32*)(wsl + WS_READY + ((lane & 1) ? 4 + (nt >> 2) : mt >> 2) * 9 * 128);
    if (wid == 0) wait_ready(cp, 64u * call, t0, bad);
    asm volatile("s_waitcnt lgkmcnt(0)" ::: "memory");
    __builtin_amdgcn_s_barrier();
    STAMP_AT(2 + round * 3);
    f32x4 acc[2][2][4][2];
#pragma unroll
    for (int i0 = 0; i0 < 2; ++i0)
#pragma unroll
      for (int i1 = 0; i1 < 2; ++i1)
#pragma unroll
        for (int m = 0; m < 4; ++m)
#pragma unroll
          for (int n = 0; n < 2; ++n) acc[i0][i1][m][n] = (f32x4){0.f, 0.f, 0.f, 0.f};
    bf16x8 At[4][2], B0[2][2], B1[2][2];
    const char* cA = wsl + WS_IMG + IMG_BYTES + (size_t)nt * (2u << 20);
    const char* cB = wsl + WS_IMG + (size_t)mt * (2u << 20);
    const u32 voff = tid * 16, ldsw = wid * 1024;
    const int aoff = wr * 8192 + lane * 16, boff = wc * 4096 + lane * 16;
    constexpr int HTB = 16384, KST = 32768, NT = 64;
#define SA(b, h) (((b) * 2 + (h)) * HTB)
#define SB(b, h) ((4 + (b) * 2 + (h)) * HTB)
#define STAGE(bufoff, gbase) do { _Pragma("unroll") for (int _i = 0; _i < 2; ++_i) \
    __builtin_amdgcn_global_load_lds((const GAS u32*)((gbase) + _i * 8192 + voff), (LAS u32*)(lds + (bufoff) + ldsw + _i * 8192), 16, 0, 17); } while (0)
#define LDA(dst, b, h) do { _Pragma("unroll") for (int m = 0; m < 4; ++m) _Pragma("unroll") for (int k = 0; k < 2; ++k) \
    dst[m][k] = *(const LAS bf16x8*)(lds + SA(b, h) + aoff + m * 2048 + k * 1024); } while (0)
#define LDB(dst, b, h) do { _Pragma("unroll") for (int n = 0; n < 2; ++n) _Pragma("unroll") for (int k = 0; k < 2; ++k) \
    dst[n][k] = *(const LAS bf16x8*)(lds + SB(b, h) + boff + n * 2048 + k * 1024); } while (0)
#define MMA(ai, bj, A_, B_) do { __builtin_amdgcn_s_setprio(1); _Pragma("unroll") for (int m = 0; m < 4; ++m) _Pragma("unroll") for (int n = 0; n < 2; ++n) \
    _Pragma("unroll") for (int k = 0; k < 2; ++k) acc[ai][bj][m][n] = __builtin_amdgcn_mfma_f32_16x16x32_bf16(A_[m][k], B_[n][k], acc[ai][bj][m][n], 0, 0, 0); \
    __builtin_amdgcn_s_setprio(0); } while (0)
#define WAIT_V(n) asm volatile("s_waitcnt vmcnt(" #n ")" ::: "memory")
#define WAIT_L(n) asm volatile("s_waitcnt lgkmcnt(" #n ")" ::: "memory")
#define BAR __builtin_amdgcn_s_barrier()
#define SCHED __builtin_amdgcn_sched_barrier(0)
    STAGE(SB(0, 0), cB); STAGE(SA(0, 0), cA); STAGE(SB(0, 1), cB + HTB); STAGE(SA(0, 1), cA + HTB);
    if (wr == 1) BAR;
    WAIT_V(4); BAR;
    STAGE(SB(1, 0), cB + KST); STAGE(SA(1, 0), cA + KST); STAGE(SB(1, 1), cB + HTB + KST);
    WAIT_V(6); BAR;
    int tn = 5;
    for (int t = 0; t < NT; t += 2) {
      if (wid == 0 && tn < 61) {
        if (t >= tn) {
          const u32 c = lds_poll(LDS_BYTES - 32 + (lane & 1) * 4);
          STAMP_AT(16 + round * 16 + (tn / 7) * 2);
          if (__builtin_amdgcn_ballot_w64((int)(c - 56u * call) < 0) != 0) wait_ready(cp, 56u * call, t0, bad);
          STAMP_AT(17 + round * 16 + (tn / 7) * 2);
          tn += 7;
        } else if (t + 2 >= tn) {
          cp += 32;
          if (lane < 2) __builtin_amdgcn_global_load_lds(cp, (LAS u32*)(lds + LDS_BYTES - 32), 4, 0, 17);
        }
      }
      const bool last = (t == NT - 2);
      const char* a1 = cA + (size_t)(t + 1) * KST;
      const char* a2 = last ? cA : cA + (size_t)(t + 2) * KST; const char* b2 = last ? cB : cB + (size_t)(t + 2) * KST;
      const char* a3 = a2 + KST; const char* b3 = b2 + KST;
      LDB(B0, 0, 0); SCHED; LDA(At, 0, 0); STAGE(SA(1, 1), a1 + HTB);
      WAIT_L(8); BAR; WAIT_L(0); MMA(0, 0, At, B0); BAR; SCHED;
      LDB(B1, 0, 1); STAGE(SB(0, 0), b2);
      BAR; WAIT_L(0); MMA(0, 1, At, B1); BAR;
      LDA(At, 0, 1); STAGE(SA(0, 0), a2);
      BAR; WAIT_L(0); MMA(1, 0, At, B0); BAR; SCHED;
      STAGE(SB(0, 1), b2 + HTB);
      WAIT_V(6); BAR; MMA(1, 1, At, B1); BAR;
      LDB(B0, 1, 0); SCHED; LDA(At, 1, 0); STAGE(SA(0, 1), a2 + HTB);
      WAIT_L(8); BAR; WAIT_L(0); MMA(0, 0, At, B0); BAR; SCHED;
      LDB(B1, 1, 1); STAGE(SB(1, 0), b3);
      BAR; WAIT_L(0); MMA(0, 1, At, B1); BAR;
      LDA(At, 1, 1); STAGE(SA(1, 0), a3);
      BAR; WAIT_L(0); MMA(1, 0, At, B0); BAR; SCHED;
      STAGE(SB(1, 1), b3 + HTB);
      WAIT_V(6); BAR; MMA(1, 1, At, B1); BAR;
    }
    if (wid == 0) lds_post(LDS_BYTES - 16, bad);
    asm volatile("s_waitcnt vmcnt(0) lgkmcnt(0)" ::: "memory");
    if (wr == 0) BAR;
    BAR;
#undef SA
#undef SB
#undef STAGE
#undef LDA
#undef LDB
#undef MMA
    int tz = threadIdx.x; asm volatile("" : "+v"(tz));
    lane = tz & 63;
    STAMP_AT(3 + round * 3);
    if (rfl(lds_poll(LDS_BYTES - 16))) {
#pragma unroll
      for (int i = 0; i < 32; ++i) acc[i >> 4][(i >> 3) & 1][(i >> 1) & 3][i & 1] = (f32x4){__builtin_nanf(""), __builtin_nanf(""), __builtin_nanf(""), __builtin_nanf("")};
    }
    pack_out(a, acc, tile * 8 + wid, call, lds + wid * EREG, lane);
    STAMP_AT(4 + round * 3);
  }
}

DEV void loader_role(const Args& a, LAS char* lds, int peer, int lane, u64 t0) {
  const char* src = a.ws[peer] + WS_BLOB + lane * 16;
  u32 served = 0; int pend = -1; u32 pend_tag = 0;
  int done = 0;
  for (int left = NDEC; left > 0;) {
    const u32 r = lds_poll(L_REQ + (lane & 7) * 4);
    u32 key = ((lane & 7) < NDEC && (r & 255) != served) ? ((255 - (r & 255)) << 3 | (7 - (lane & 7))) + 1 : 0;
    key = umax(key, dpp0<0x111, 0xf>(key)); key = umax(key, dpp0<0x112, 0xf>(key)); key = umax(key, dpp0<0x114, 0xf>(key));
    key = (u32)__builtin_amdgcn_readlane((int)key, 7);
    if (key == 0) {
      if (pend >= 0) { waitv<0>(); lds_post(L_FULL + pend * 4, pend_tag); pend = -1; }
      else { if (now() - t0 > DEADLINE + 20000000ull) break; __builtin_amdgcn_s_sleep(2); }
      continue;
    }
    const int j = 7 - ((key - 1) & 7);
    const u32 rq = (u32)__builtin_amdgcn_readlane((int)r, j);
    if ((lane & 7) == j) served = rq & 255;
    if (rq >> 31) { --left; continue; }
    ++done;
    const u32 slot = umin(rq >> 18, NSLOT - 1), bytes = (((rq >> 8) & 1023) + 1) * 16;
    const char* s = src + (size_t)slot * 16384;
    LAS char* dst = lds + j * DREG + 8192;
    int row = 0;
    for (u32 o = 0; o < bytes; o += 1024, ++row) {
      if (o + lane * 16 < bytes) __builtin_amdgcn_global_load_lds((const GAS u32*)(s + o), (LAS u32*)(dst + o), 16, 0, 17);
      waitv<WIN - 1>();
      if (row == WIN - 2 && pend >= 0) { lds_post(L_FULL + pend * 4, pend_tag); pend = -1; }
    }
    if (pend >= 0) { waitv<0>(); lds_post(L_FULL + pend * 4, pend_tag); }
    pend = j; pend_tag = rq & 255;
  }
  waitv<0>();
  if (pend >= 0) lds_post(L_FULL + pend * 4, pend_tag);
}

DEV void decoder_role(const Args& a, LAS char* lds, int peer, int c, int j, int lane, u32 call, u64 t0) {
  char* const wsl = a.ws[a.rank];
  LAS char* A = lds + j * DREG;
  LAS char* B = A + 8192;
  const int q = (peer - a.rank) & 7;
  bool fail = false;
  u32 tag = 0;
  for (int m = j;; m += NDEC) {
    const char* dsc = wsl + WS_DESC + (size_t)((peer * 16 + c) * NIN + m) * 16;
    tag = tag == 255 ? 1 : tag + 1;
    if (m >= NIN) {
      u32 u = 0;
      if (lane == 0) u = __hip_atomic_fetch_add((GAS u32*)(wsl + WS_QUEUE + peer * 128), 1u, __ATOMIC_RELAXED, __HIP_MEMORY_SCOPE_AGENT);
      u = rfl(u) - (call - 1) * QSTEP;
      if (u >= 2048) break;
      dsc = wsl + WS_DESCO + (size_t)(peer * 2048 + u) * 16;
    }
    u32x4 d;
    for (;;) {
      d = ld_sys128(dsc);
      if (rfl(d.x) == call && rfl(d.w) == call) break;
      if (now() - t0 > DEADLINE) { fail = true; break; }
      __builtin_amdgcn_s_sleep(4);
    }
    if (fail) break;
    const u32 da = rfl(d.y), db = rfl(d.z);
    const bool raw = db >> 31;
    u32 slot, bytes, E, G = 0;
    const int kj = m < 24 ? m / 3 : m - 24, sl = m < 24 ? m - kj * 3 : 3;
    const int t = sl < 2 ? c >> 3 : sl - 2, s = sl < 2 ? sl * 4 + ((c >> 1) & 3) : 8 + (c >> 1);
    if (m < NIN) {
      E = umin(da >> 16, IN_EXC_CAP);
      slot = (t ? SLOT_W + a.rank * 256 : 0) + s * 16 + kj * 2 + (c & 1);
      bytes = 11264 + ((2 * E + 15) & ~15u);
    } else {
      E = umin(db & 0xFFFF, OUT_EXC_CAP);
      G = (umin(da & 0xFFFF, 8192) + 15) >> 4;
      slot = SLOT_O + ((db >> 16) & 2047);
      bytes = 1024 + 16 * G + ((4 * G + 15) & ~15u) + ((2 * E + 15) & ~15u);
    }
    if (raw) bytes = 16384;
    lds_post(L_REQ + j * 4, (slot << 18) | ((bytes / 16 - 1) << 8) | tag);
    for (;;) {
      if (rfl(lds_poll(L_FULL + j * 4)) == tag) break;
      if (now() - t0 > DEADLINE) { fail = true; break; }
      __builtin_amdgcn_s_sleep(2);
    }
    if (fail) break;
    if (tag <= 12) STAMP_AT(13 + tag * 4);
    if (m < NIN) {
      const u32 base = da & 0xFFFF, bb = base * 0x10001u, upz = ((65536 - base) >> 11) & 31;
      u32 H[32];
      if (raw) {
#pragma unroll
        for (int r = 0; r < 8; ++r) { const u32x4 t = lds_r128(B + (8 + r) * 1024 + lane * 16); H[4 * r] = t.x; H[4 * r + 1] = t.y; H[4 * r + 2] = t.z; H[4 * r + 3] = t.w; }
      } else {
        for (u32 i = lane; i < E; i += 64) {
          const u32 en = *(LAS u16*)(B + 11264 + 2 * i), cc = en & 7;
          *(LAS unsigned char*)(A + (en >> 3)) = (unsigned char)((cc ? 32 - cc : upz) << 3);
        }
        cbar();
        u32 P[12];
#pragma unroll
        for (int r = 0; r < 3; ++r) { const u32x4 t = lds_r128(B + (8 + r) * 1024 + lane * 16); P[4 * r] = t.x; P[4 * r + 1] = t.y; P[4 * r + 2] = t.z; P[4 * r + 3] = t.w; }
#pragma unroll
        for (int r = 0; r < 8; ++r) { const u32x4 t = lds_r128(A + r * 1024 + lane * 16); H[4 * r] = t.x; H[4 * r + 1] = t.y; H[4 * r + 2] = t.z; H[4 * r + 3] = t.w; }
        cbar();
        for (u32 i = lane; i < E; i += 64) *(LAS unsigned char*)(A + (*(LAS u16*)(B + 11264 + 2 * i) >> 3)) = 0;
        cbar();
#pragma unroll
        for (int g = 0; g < 4; ++g) {
          u32* h = H + 8 * g; const u32 p0 = P[3 * g], p1 = P[3 * g + 1], p2 = P[3 * g + 2], m7 = 0x07070707u;
          h[0] |= p0 & m7; h[1] |= (p0 >> 3) & m7; h[2] |= ((p0 >> 6) & 0x03030303u) | ((p2 >> 4) & 0x04040404u);
          h[3] |= p1 & m7; h[4] |= (p1 >> 3) & m7; h[5] |= ((p1 >> 6) & 0x03030303u) | ((p2 >> 5) & 0x04040404u);
          h[6] |= p2 & m7; h[7] |= (p2 >> 3) & m7;
        }
      }
      const u32 so = t * IMG_BYTES + (u32)(s * 64 + 7 + kj * 7 + q) * 32768u + (c & 1) * (t ? 8192 : 16384);
      const rsrc_t rimg[2] = {mkrsrc(wsl + WS_IMG + so, 8192), mkrsrc(wsl + WS_IMG + so + (t ? 16384 : 8192), 8192)};
#pragma unroll
      for (int r = 0; r < 8; ++r) {
        const u32x4 lo = lds_r128(B + r * 1024 + lane * 16);
        u32 v[8];
#pragma unroll
        for (int k = 0; k < 4; ++k) {
          const u32 u0 = pk_add(perm(H[4 * r + k], lo[k], 0x05010400u), bb), u1 = pk_add(perm(H[4 * r + k], lo[k], 0x07030602u), bb);
          v[2 * k] = __builtin_amdgcn_alignbit(u0, u0, 1); v[2 * k + 1] = __builtin_amdgcn_alignbit(u1, u1, 1);
        }
        bst128(rimg[r >> 2], lane * 16 + ((2 * r) & 7) * 1024, (u32x4){v[0], v[1], v[2], v[3]});
        bst128(rimg[r >> 2], lane * 16 + ((2 * r + 1) & 7) * 1024, (u32x4){v[4], v[5], v[6], v[7]});
      }
      if (tag <= 12) STAMP_AT(14 + tag * 4);
      waitv<0>();
      if (tag <= 12) STAMP_AT(15 + tag * 4);
      if (lane == 0) __hip_atomic_fetch_add((GAS u32*)(wsl + WS_READY + ((t * 4 + (s >> 2)) * 9 + 1 + kj) * 128), 1u, __ATOMIC_RELAXED, __HIP_MEMORY_SCOPE_AGENT);
    } else {
      const u32 blob = (db >> 16) & 2047, tile = blob >> 3, w8 = blob & 7;
      const u32 so = ((tile >> 4) * 256 + (w8 & 3) * 32) * 65536u + (peer * 4096 + (tile & 15) * 256 + (w8 >> 2) * 128) * 2u;
      const u32 vo = (lane >> 2) * 65536u + (lane & 3) * 16;
      const rsrc_t rout = mkrsrc(a.out + so, OUT_BYTES - so);
      if (raw) {
#pragma unroll
        for (int ch = 0; ch < 16; ++ch)
          __builtin_amdgcn_raw_buffer_store_b128(lds_r128(B + ch * 1024 + lane * 16), rout, vo + (ch & 3) * 64 + ((ch >> 3) * 128 + ((ch >> 2) & 1) * 16) * 65536u, 0, OUT_AUX);
      } else {
        const u32 offp = ((da >> 16) & 0x7FFF) * 0x10001u;
        const u32x4 bits = lds_r128(B + lane * 16);
        cbar();
        for (u32 g = lane; g < G; g += 64) {
          const u32x4 lo = lds_r128(B + 1024 + 16 * g); const u32 code = *(LAS u32*)(B + 1024 + 16 * G + 4 * g);
          u32 v[8];
#pragma unroll
          for (int k = 0; k < 4; ++k) {
            const u32 h = (code >> (2 * k)) & 0x03030303u;
            v[2 * k] = pk_add(perm(h, lo[k], 0x05010400u), offp); v[2 * k + 1] = pk_add(perm(h, lo[k], 0x07030602u), offp);
          }
          lds_w128(A + 32 * g, (u32x4){v[0], v[1], v[2], v[3]}); lds_w128(A + 32 * g + 16, (u32x4){v[4], v[5], v[6], v[7]});
          cbar();
        }
        LAS char* ex = B + 1024 + 16 * G + ((4 * G + 15) & ~15u);
        for (u32 i = lane; i < E; i += 64) {
          const u32 en = *(LAS u16*)(ex + 2 * i); LAS u16* p = (LAS u16*)(A + 2 * (en >> 3));
          *p = (u16)(*p - ((en & 7) << 10));
        }
        cbar();
        const u32 cnt = __builtin_popcount(bits.x) + __builtin_popcount(bits.y) + __builtin_popcount(bits.z) + __builtin_popcount(bits.w);
        u32 pos = wave_scan(cnt) - cnt;
#pragma unroll
        for (int ch = 0; ch < 16; ++ch) {
          u32x4 ov;
#pragma unroll
          for (int h = 0; h < 2; ++h) {
            const u32 nib = (bits[ch >> 2] >> (28 - 4 * ((ch & 3) * 2 + h))) & 15;
            const u32x2 sel = *(LAS u32x2*)(lds + L_LUT + nib * 8);
            const u32 ad = 2 * pos; LAS u32* q = (LAS u32*)(A + (ad & ~3u));
            const u32 d0 = q[0], d1 = q[1], d2 = q[2];
            const u32 e0 = __builtin_amdgcn_alignbyte(d1, d0, ad), e1 = __builtin_amdgcn_alignbyte(d2, d1, ad);
            ov[2 * h] = perm(e1, e0, sel.x); ov[2 * h + 1] = perm(e1, e0, sel.y);
            pos += __builtin_popcount(nib);
          }
          __builtin_amdgcn_raw_buffer_store_b128(ov, rout, vo + (ch & 3) * 64 + ((ch >> 3) * 128 + ((ch >> 2) & 1) * 16) * 65536u, 0, OUT_AUX);
        }
      }
      cbar();
      if (tag <= 12) STAMP_AT(14 + tag * 4);
    }
  }
  if (fail) tag = tag == 255 ? 1 : tag + 1;
  lds_post(L_REQ + j * 4, 0x80000000u | tag);
  if (fail) {
    waitv<0>();
    __builtin_amdgcn_raw_buffer_store_b32(0x7FC07FC0u, mkrsrc(a.out, OUT_BYTES), lane * 4 + (blockIdx.x * 8 + j) * 65536u, 0, 0);
    if (lane == 0) __hip_atomic_store((GAS u32*)(wsl + WS_TMO), call, __ATOMIC_RELAXED, __HIP_MEMORY_SCOPE_AGENT);
  }
}

__global__ void __launch_bounds__(512, 2) fused_gemm_ar(const Args a) {
  extern __shared__ __attribute__((aligned(16))) char smem[];
  LAS char* lds = (LAS char*)smem;
  const u64 t0 = now();
  const int wid = __builtin_amdgcn_readfirstlane(threadIdx.x >> 6), lane = threadIdx.x & 63;
  const int xcd = blockIdx.x & 7, li = blockIdx.x >> 3;
  char* const wsl = a.ws[a.rank];
  GAS u32* callp = (GAS u32*)(wsl + WS_CALL) + blockIdx.x * 8 + wid;
  const u32 callv = __hip_atomic_load(callp, __ATOMIC_RELAXED, __HIP_MEMORY_SCOPE_AGENT);
  char* const wsp = a.ws[lane & 7];
  STAMP_AT(0);
  const bool is_gemm = li < 16;
  const int pos = xcd * 16 + (li & 15);
  if (!is_gemm) {
    if (wid == 0) {
      if (lane < 32) *(LAS u32*)(lds + L_CTRL + lane * 4) = 0;
      if (lane < 16) {
        u32 k = 0, sel[2] = {0, 0};
#pragma unroll
        for (int i = 0; i < 4; ++i) {
          const bool on = (lane >> (3 - i)) & 1;
          sel[i >> 1] |= (on ? (2 * k) | ((2 * k + 1) << 8) : 0x0C0Cu) << (16 * (i & 1));
          k += on;
        }
        *(LAS u32x2*)(lds + L_LUT + lane * 8) = (u32x2){sel[0], sel[1]};
      }
    } else if (wid <= NDEC) {
#pragma unroll
      for (int r = 0; r < 8; ++r) lds_w128(lds + (wid - 1) * DREG + r * 1024 + lane * 16, (u32x4){0, 0, 0, 0});
    }
    asm volatile("s_waitcnt lgkmcnt(0)" ::: "memory");
    __builtin_amdgcn_s_barrier();
  }
  const int q = li - 16;
  const bool is_help = !is_gemm && (q & 7) == 0;
  {
    int e = -1, n = 1;
    if (is_gemm) { e = wid == 0 ? 1152 + pos : 448 + (wid - 1) * 128 + pos; if (wid && e >= 1152) e = -1; }
    else if (is_help) { e = 1280 + (xcd * 2 + (q >> 3)) * 8 + wid; n = 8; }
    else if (wid >= 4) e = (wid - 4) * 112 + (xcd * 2 + (q >> 3)) * 7 + (q & 7) - 1;
    LAS char* scr = is_gemm || is_help ? lds + wid * EREG : wid == 7 ? lds + L_SCR7 : lds + (wid - 1) * DREG + 8192;
    if (GDELAY && (is_gemm || is_help)) __builtin_amdgcn_s_sleep(GDELAY);
    if (e >= 0)
      for (int i = 0; i < n; ++i, e += 128) {
        bool isx = false; int j, s, dq;
        if (e < 1152) { j = e / 144; const int r = e - j * 144; isx = r < 32; s = isx ? r >> 1 : (r >> 1) & 7; dq = (r >> 4) - 1; }
        else if (e < 1280) { j = (e >> 4) & 7; s = (e >> 1) & 7; dq = 0; }
        else { j = (e - 1280) >> 7; s = 8 + ((e >> 1) & 7); dq = e >> 4; }
        prep_piece(a, isx, s, j, e & 1, (a.rank + dq) & 7, callv, wsp, scr, lane);
        if (is_help) STAMP_AT(16 + i);
      }
  }
  const u32 call = rfl(callv) + 1;
  if (lane == 0) __hip_atomic_store(callp, call, __ATOMIC_RELAXED, __HIP_MEMORY_SCOPE_AGENT);
  STAMP_AT(1);
  if (is_gemm) {
    gemm_role(a, lds, pos, wid, call, t0);
  } else if (!is_help) {
    const int peer = (a.rank + (q & 7)) & 7, c = xcd * 2 + (q >> 3);
    if (wid == 0) loader_role(a, lds, peer, lane, t0);
    else if (wid <= NDEC) decoder_role(a, lds, peer, c, wid - 1, lane, call, t0);
  }
  STAMP_AT(8);
}

#define HIP_CHECK(expr) do { hipError_t _e = (expr); if (_e != hipSuccess) { \
    fprintf(stderr, "HIP error %s at %s:%d\n", hipGetErrorString(_e), __FILE__, __LINE__); \
    return; } } while (0)

#define DBG_FLAGS 0

extern "C" void kernel_launch(int rank, int nranks, ncclComm_t comm, hipStream_t stream,
                              void* const* d_in, const int* in_sizes, int n_in,
                              void* d_out, int out_size, const int* in_dtypes, int out_dtype,
                              void* const* d_out_all, void* const* d_ws_all, size_t ws_bytes) {
  if (nranks != NR || (unsigned)rank >= (unsigned)NR || n_in < 2 || ws_bytes < WS_USED || (size_t)out_size * 2 != OUT_BYTES ||
      in_sizes[0] != 4096 * 512 || in_sizes[1] != 512 * 32768) {
    fprintf(stderr, "unexpected problem shape\n");
    return;
  }
  static bool attr_set[NR] = {};
  if (!attr_set[rank]) {
    HIP_CHECK(hipFuncSetAttribute((const void*)fused_gemm_ar, hipFuncAttributeMaxDynamicSharedMemorySize, LDS_BYTES));
    attr_set[rank] = true;
  }
  Args a;
  a.x = (const char*)d_in[0]; a.w = (const char*)d_in[1]; a.out = (char*)d_out;
  for (int p = 0; p < NR; ++p) a.ws[p] = (char*)d_ws_all[p];
  a.rank = rank; a.flags = DBG_FLAGS;
  static const int blk[8][2] = {{0, 0}, {1, 0}, {0, 1}, {1, 1}, {2, 0}, {2, 1}, {3, 0}, {3, 1}};
  for (int k = 0; k < 256; ++k) {
    const int b = (k >> 4) & 7, i = k & 15;
    a.tiles[k] = (unsigned char)((blk[b][0] * 4 + (i >> 2)) * 16 + (blk[b][1] + (k >> 7) * 2) * 4 + (i & 3));
  }
  hipLaunchKernelGGL(fused_gemm_ar, dim3(256), dim3(512), LDS_BYTES, stream, a);
  HIP_CHECK(hipGetLastError());
}
```

```cpp
#include <hip/hip_runtime.h>
#include <rccl/rccl.h>
#include <cstdio>

typedef unsigned u32;
typedef unsigned long long u64;
typedef u32 u32x4 __attribute__((ext_vector_type(4)));
typedef u32 u32x2 __attribute__((ext_vector_type(2)));
typedef unsigned short u16;
typedef u16 u16x2 __attribute__((ext_vector_type(2)));
typedef short bf16x8 __attribute__((ext_vector_type(8)));
typedef float f32x4 __attribute__((ext_vector_type(4)));
typedef float f32x2 __attribute__((ext_vector_type(2)));
typedef __bf16 bf16x2 __attribute__((ext_vector_type(2)));
#define LAS __attribute__((address_space(3)))
#define GAS __attribute__((address_space(1)))
#define DEV __device__ __forceinline__

constexpr int NR = 8;
constexpr u32 WS_CALL = 0;
constexpr u32 WS_TMO = 8192;
constexpr u32 WS_TICKET = 8192 + 128;
constexpr u32 WS_QUEUE = 8192 + 256;
constexpr u32 WS_READY = 16384;
constexpr int NIN = 32;
constexpr u32 WS_DESC = 512u << 10;
constexpr u32 WS_DESCO = WS_DESC + 65536;
constexpr u32 WS_IMG = 1u << 20;
constexpr u32 IMG_BYTES = 32u << 20;
constexpr u32 WS_BLOB = 65u << 20;
constexpr u32 SLOT_W = 256, SLOT_O = 2304, NSLOT = 4352;
constexpr u32 WS_USED = WS_BLOB + NSLOT * 16384u;
constexpr u32 WS_STAMP = WS_USED;
constexpr u32 OUT_BYTES = 4096u * 32768u * 2u;
constexpr u64 DEADLINE = 100000000ull;
constexpr int LDS_BYTES = 163840;
constexpr int NDEC = 6;
constexpr int DREG = 26624;
constexpr int L_CTRL = NDEC * DREG;
constexpr int L_REQ = L_CTRL, L_FULL = L_CTRL + 64, L_LUT = L_CTRL + 128, L_SCR7 = L_CTRL + 256;
constexpr int EREG = 20480;
constexpr int E_EXC = 16448;
constexpr u32 IN_EXC_CAP = 1792, OUT_EXC_CAP = 1984;
constexpr u32 QSTEP = 2048 + 16 * NDEC;
#define WIN 5
#define GDELAY 0
#define OWN_AUX 0
#define OUT_AUX 0
#define IMG_AUX 17
#define GL_AUX 17
#define PL_AUX 2
#define SEG 0
DEV int vbid() {
  const int x = blockIdx.x & 7, l = blockIdx.x >> 3;
  if (SEG == 0) return blockIdx.x;
  const int comm = SEG == 1 ? (x & 1) ^ 1 : SEG == 2 ? x & 1 : SEG == 3 ? x >> 2 : (x >> 1) & 1;
  const int g = SEG <= 2 ? x >> 1 : SEG == 3 ? x & 3 : (x & 1) | ((x >> 2) << 1);
  return comm * 128 + (l & 15) * 8 + g * 2 + (l >> 4);
}
#define LD_AUX 17
#define NOIMG 0

struct Args {
  const char* x; const char* w; char* out; char* ws[NR]; int rank; int flags; unsigned char tiles[256];
};

template <int N> DEV void waitv() { asm volatile("s_waitcnt vmcnt(%0)" ::"n"(N) : "memory"); }
DEV void cbar() { asm volatile("" ::: "memory"); }
DEV u64 now() { return __builtin_amdgcn_s_memrealtime(); }
DEV u32 perm(u32 hi, u32 lo, u32 sel) { return __builtin_amdgcn_perm(hi, lo, sel); }
DEV u32 pk_sub(u32 a, u32 b) { return __builtin_bit_cast(u32, __builtin_bit_cast(u16x2, a) - __builtin_bit_cast(u16x2, b)); }
DEV u32 pk_add(u32 a, u32 b) { return __builtin_bit_cast(u32, __builtin_bit_cast(u16x2, a) + __builtin_bit_cast(u16x2, b)); }
DEV u32 pk_max(u32 a, u32 b) { return __builtin_bit_cast(u32, __builtin_elementwise_max(__builtin_bit_cast(u16x2, a), __builtin_bit_cast(u16x2, b))); }
DEV u32 pk_min(u32 a, u32 b) { return __builtin_bit_cast(u32, __builtin_elementwise_min(__builtin_bit_cast(u16x2, a), __builtin_bit_cast(u16x2, b))); }
template <int C, int RM> DEV u32 dpp0(u32 x) { return (u32)__builtin_amdgcn_update_dpp(0, (int)x, C, RM, 0xf, false); }
DEV u32 umax(u32 a, u32 b) { return a > b ? a : b; }
DEV u32 umin(u32 a, u32 b) { return a < b ? a : b; }
DEV u32 wave_max(u32 x) {
  x = umax(x, dpp0<0x111, 0xf>(x)); x = umax(x, dpp0<0x112, 0xf>(x)); x = umax(x, dpp0<0x114, 0xf>(x)); x = umax(x, dpp0<0x118, 0xf>(x));
  x = umax(x, dpp0<0x142, 0xa>(x)); x = umax(x, dpp0<0x143, 0xc>(x));
  return (u32)__builtin_amdgcn_readlane((int)x, 63);
}
DEV u32 wave_scan(u32 x) {
  x += dpp0<0x111, 0xf>(x); x += dpp0<0x112, 0xf>(x); x += dpp0<0x114, 0xf>(x); x += dpp0<0x118, 0xf>(x);
  x += dpp0<0x142, 0xa>(x); x += dpp0<0x143, 0xc>(x);
  return x;
}
DEV u32 rfl(u32 x) { return (u32)__builtin_amdgcn_readfirstlane((int)x); }
DEV u32 mbcnt(u64 m) { return __builtin_amdgcn_mbcnt_hi((u32)(m >> 32), __builtin_amdgcn_mbcnt_lo((u32)m, 0)); }
DEV u32x4 lds_r128(LAS char* p) { return *(LAS u32x4*)p; }
DEV void lds_w128(LAS char* p, u32x4 v) { *(LAS u32x4*)p = v; }
DEV u32 lds_poll(u32 addr) { u32 v; asm volatile("ds_read_b32 %0, %1\n\ts_waitcnt lgkmcnt(0)" : "=v"(v) : "v"(addr) : "memory"); return v; }
DEV void lds_post(u32 addr, u32 v) { asm volatile("ds_write_b32 %0, %1" ::"v"(addr), "v"(v) : "memory"); }
DEV void st_sys128(void* p, u32x4 v) { asm volatile("global_store_dwordx4 %0, %1, off sc0 sc1\n\ts_nop 1" ::"v"(p), "v"(v) : "memory"); }
DEV u32x4 ld_sys128(const void* p) { u32x4 v; asm volatile("global_load_dwordx4 %0, %1, off sc0 sc1\n\ts_waitcnt vmcnt(0)" : "=&v"(v) : "v"(p) : "memory"); return v; }
typedef __amdgpu_buffer_rsrc_t rsrc_t;
DEV rsrc_t mkrsrc(const void* p, u32 bytes) { return __builtin_amdgcn_make_buffer_rsrc((void*)p, 0, (int)bytes, 0x00020000); }
DEV void bst128(rsrc_t r, u32 voff, u32x4 v) { __builtin_amdgcn_raw_buffer_store_b128(v, r, (int)voff, 0, 17); }
DEV void ist128(rsrc_t r, u32 voff, u32x4 v) { __builtin_amdgcn_raw_buffer_store_b128(v, r, (int)voff, 0, IMG_AUX); }
DEV void bst32(rsrc_t r, u32 voff, u32 v) { __builtin_amdgcn_raw_buffer_store_b32(v, r, (int)voff, 0, 17); }

#define STAMP_AT(k) do { } while (0)

DEV void emit_exc(const u32 (&U)[4], bool act, u32 gidx, u32 upz, LAS char* list, u32 cap, u32& E, bool& bad) {
  u32 F = (((U[0] + 0x7C7C7C7Cu) & 0x80808080u) >> 7) | (((U[1] + 0x7C7C7C7Cu) & 0x80808080u) >> 6) |
          (((U[2] + 0x7C7C7C7Cu) & 0x80808080u) >> 5) | (((U[3] + 0x7C7C7C7Cu) & 0x80808080u) >> 4);
  if (!act) F = 0;
  for (;;) {
    const u64 bal = __builtin_amdgcn_ballot_w64(F != 0);
    if (bal == 0) break;
    if (F != 0) {
      const u32 p = __builtin_ctz(F), k = p & 7, b = p >> 3;
      const u32 h0 = (p & 1) ? U[1] : U[0], h1 = (p & 1) ? U[3] : U[2], h = (p & 2) ? h1 : h0;
      const u32 up = (h >> (8 * b + 2)) & 31;
      u32 c = 32 - up;
      if (c > 7) { bad |= up != upz; c = 0; }
      const u32 slot = E + mbcnt(bal);
      if (slot < cap) *(LAS u16*)(list + 2 * slot) = (u16)(((gidx + 4 * k + b) << 3) | c);
      F &= F - 1;
    }
    E += __builtin_popcountll(bal);
  }
}

DEV void prep_piece(const Args& a, bool isx, int s, int j, int h, int dest, u32 callv, char* wsp, LAS char* scr, int lane) {
  const int fr = lane & 15, fq = lane >> 4;
  const int piece = j * 2 + h;
  const u32 slot = (isx ? 0 : SLOT_W + dest * 256) + s * 16 + piece;
  u32 v[64];
  {
    const rsrc_t ri = mkrsrc(isx ? a.x : a.w, isx ? 4096u * 512u * 2u : 512u * 32768u * 2u);
    const u32 lv = isx ? fr * 1024 + fq * 16 : fq * 8 * 65536 + 16 * (4 * (fr & 3) + (fr >> 2));
    const u32 ls = isx ? (s * 256 + (piece & 1) * 128) * 1024 + (piece >> 1) * 128
                       : (piece >> 1) * 64 * 65536 + (dest * 4096 + s * 256 + (piece & 1) * 128) * 2;
    const u32 s8 = isx ? 65536 : 32 * 65536, s2 = isx ? 16384 : 2 * 65536, s1 = isx ? 64 : 65536;
    u32x4 in[16];
#pragma unroll
    for (int i = 0; i < 16; ++i) in[i] = __builtin_amdgcn_raw_buffer_load_b128(ri, lv, ls + (i >> 3) * s8 + ((i >> 1) & 3) * s2 + (i & 1) * s1, PL_AUX);
    __builtin_amdgcn_sched_barrier(0);
    if (isx) {
#pragma unroll
      for (int i = 0; i < 64; ++i) v[i] = in[i >> 2][i & 3];
    } else {
#pragma unroll
      for (int ks = 0; ks < 2; ++ks)
#pragma unroll
        for (int el = 0; el < 8; ++el)
#pragma unroll
          for (int d = 0; d < 4; ++d)
            v[4 * (el * 2 + ks) + d] = perm(in[ks * 8 + 2 * d + 1][el >> 1], in[ks * 8 + 2 * d][el >> 1], (el & 1) ? 0x07060302u : 0x05040100u);
    }
  }
  GAS u32* const own = (GAS u32*)(a.ws[a.rank] + WS_READY + ((isx ? 0 : 4) + (s >> 2)) * 9 * 128);
  if (isx || dest == a.rank) {
    const u32 so = (isx ? 0 : IMG_BYTES) + (u32)(s * 64 + j) * 32768u + h * (isx ? 16384 : 8192);
    const rsrc_t rimg[2] = {mkrsrc(a.ws[a.rank] + WS_IMG + so, 8192), mkrsrc(a.ws[a.rank] + WS_IMG + so + (isx ? 8192 : 16384), 8192)};
#pragma unroll
    for (int i = 0; i < 16; ++i) ist128(rimg[i >> 3], lane * 16 + (i & 7) * 1024, (u32x4){v[4 * i], v[4 * i + 1], v[4 * i + 2], v[4 * i + 3]});
    if (!isx) {
      waitv<0>();
      if (lane == 0) __hip_atomic_fetch_add(own, 1u, __ATOMIC_RELAXED, __HIP_MEMORY_SCOPE_AGENT);
      return;
    }
  }
  u32 mx = 0;
#pragma unroll
  for (int i = 0; i < 64; ++i) { v[i] = __builtin_amdgcn_alignbit(v[i], v[i], 31); mx = pk_max(mx, v[i]); }
  const u32 um = wave_max(umax(mx & 0xFFFF, mx >> 16)) | 1;
  const u32 base = um > 2047 ? um - 2047 : 0, bb = base * 0x10001u, upz = ((65536 - base) >> 11) & 31;
  const rsrc_t rb = mkrsrc(a.ws[a.rank] + WS_BLOB + slot * 16384u, 16384u);
  const u32 vo = lane * 16;
  u32 H[32];
#pragma unroll
  for (int r = 0; r < 8; ++r) {
    u32x4 lo;
#pragma unroll
    for (int k = 0; k < 4; ++k) {
      const u32 t0 = pk_sub(v[8 * r + 2 * k], bb), t1 = pk_sub(v[8 * r + 2 * k + 1], bb);
      lo[k] = perm(t1, t0, 0x06040200u); H[4 * r + k] = perm(t1, t0, 0x07050301u);
    }
    bst128(rb, vo + r * 1024, lo);
  }
  u32 E = 0; bool bad = false;
#pragma unroll
  for (int r = 0; r < 8; ++r) {
    u32 U[4];
#pragma unroll
    for (int k = 0; k < 4; ++k) U[k] = (H[4 * r + k] >> 1) & 0x7C7C7C7Cu;
    emit_exc(U, true, (r * 64 + lane) * 16, upz, scr, IN_EXC_CAP, E, bad);
  }
  const bool raw = __builtin_amdgcn_ballot_w64(bad) != 0 || E > IN_EXC_CAP || (a.flags & 1);
  if (raw) {
#pragma unroll
    for (int r = 0; r < 8; ++r) bst128(rb, vo + (8 + r) * 1024, (u32x4){H[4 * r], H[4 * r + 1], H[4 * r + 2], H[4 * r + 3]});
  } else {
    u32 P[12];
#pragma unroll
    for (int g = 0; g < 4; ++g) {
      const u32* h = H + 8 * g; const u32 m7 = 0x07070707u;
      P[3 * g] = (h[0] & m7) | ((h[1] & m7) << 3) | ((h[2] & 0x03030303u) << 6);
      P[3 * g + 1] = (h[3] & m7) | ((h[4] & m7) << 3) | ((h[5] & 0x03030303u) << 6);
      P[3 * g + 2] = (h[6] & m7) | ((h[7] & m7) << 3) | ((h[2] & 0x04040404u) << 4) | ((h[5] & 0x04040404u) << 5);
    }
#pragma unroll
    for (int r = 0; r < 3; ++r) bst128(rb, vo + (8 + r) * 1024, (u32x4){P[4 * r], P[4 * r + 1], P[4 * r + 2], P[4 * r + 3]});
    cbar();
    for (u32 i = vo; i < 2 * E; i += 1024) bst128(rb, 11264 + i, lds_r128(scr + i));
    cbar();
  }
  waitv<0>();
  asm volatile("" : "+v"(callv), "+v"(wsp));
  const u32x4 d = {callv + 1, base | (raw ? 0u : E << 16), raw ? 0x80000000u : 0u, callv + 1};
  const int c = (s < 8 ? (s & 3) + (isx ? 0 : 4) : s - 8) * 2 + h, m = s < 8 ? j * 3 + (s >> 2) : isx ? j * 3 + 2 : 24 + j;
  const u32 doff = WS_DESC + ((a.rank * 16 + c) * NIN + m) * 16;
  if (lane < NR && lane != a.rank && (isx || lane == dest)) st_sys128(wsp + doff, d);
  if (isx && lane == 0) __hip_atomic_fetch_add(own, 1u, __ATOMIC_RELAXED, __HIP_MEMORY_SCOPE_AGENT);
}

DEV void pack_out(const Args& a, f32x4 (&acc)[2][2][4][2], u32 blob, u32 call, LAS char* reg, int lane) {
  u32 o[64];
#pragma unroll
  for (int bj = 0; bj < 2; ++bj)
#pragma unroll
    for (int n = 0; n < 2; ++n)
#pragma unroll
      for (int j = 0; j < 4; ++j)
#pragma unroll
        for (int d = 0; d < 4; ++d) {
          const f32x2 y = {__builtin_elementwise_maximum(acc[d >> 1][bj][(d & 1) * 2][n][j], 0.f),
                           __builtin_elementwise_maximum(acc[d >> 1][bj][(d & 1) * 2 + 1][n][j], 0.f)};
          o[((bj * 2 + n) * 4 + j) * 4 + d] = __builtin_bit_cast(u32, __builtin_convertvector(y, bf16x2)) & 0x7FFF7FFFu;
        }
  u32 mx = 0; u32x4 bits;
#pragma unroll
  for (int q = 0; q < 4; ++q) {
    u32 m16[2];
#pragma unroll
    for (int h = 0; h < 2; ++h) {
      u32 t = 0;
#pragma unroll
      for (int i = 0; i < 8; ++i) { const u32 x = o[q * 16 + h * 8 + i]; mx = pk_max(mx, x); t = (t << 2) | pk_min(x, 0x00010001u); }
      m16[h] = ((t & 0xFFFF) << 1) | (t >> 16);
    }
    bits[q] = (m16[0] << 16) | m16[1];
  }
  const u32 vmax = wave_max(umax(mx & 0xFFFF, mx >> 16));
  const u32 off = vmax > 1023 ? vmax - 1023 : 0, offp = off * 0x10001u;
  const u32 cnt = __builtin_popcount(bits.x) + __builtin_popcount(bits.y) + __builtin_popcount(bits.z) + __builtin_popcount(bits.w);
  const u32 dl = (lane & 15) * 4 + (lane >> 4);
  const u32 cd = (u32)__builtin_amdgcn_ds_bpermute((((lane & 3) * 16 + (lane >> 2)) * 4), (int)cnt);
  const u32 inc = wave_scan(cd);
  const u32 n = (u32)__builtin_amdgcn_readlane((int)inc, 63);
  const u32 start = (u32)__builtin_amdgcn_ds_bpermute((int)(dl * 4), (int)(inc - cd));
  const rsrc_t rb = mkrsrc(a.ws[a.rank] + WS_BLOB + (SLOT_O + blob) * 16384u, 16384u);
  bst128(rb, dl * 16, bits);
  {
    u32 pos = 2 * start; const u32 dump = E_EXC + lane * 4;
#pragma unroll
    for (int i = 0; i < 64; ++i) {
      const u32 x = o[i]; const bool z0 = (x & 0xFFFF) != 0, z1 = x > 0xFFFF;
      *(LAS u16*)(reg + (z0 ? pos : dump)) = (u16)x; pos += z0 ? 2 : 0;
      *(LAS u16*)(reg + (z1 ? pos : dump)) = (u16)(x >> 16); pos += z1 ? 2 : 0;
    }
    if (lane < 16) *(LAS u16*)(reg + 2 * (n + lane)) = (u16)off;
  }
  cbar();
  const u32 G = (n + 15) >> 4;
  u32 E = 0; bool bad = false;
  for (u32 g0 = 0; g0 < G; g0 += 64) {
    const u32 g = g0 + lane; const bool act = g < G;
    const u32x4 t0 = lds_r128(reg + 32 * g), t1 = lds_r128(reg + 32 * g + 16);
    u32 t[8] = {t0.x, t0.y, t0.z, t0.w, t1.x, t1.y, t1.z, t1.w}, U[4], code = 0; u32x4 lo;
#pragma unroll
    for (int k = 0; k < 4; ++k) {
      const u32 p0 = pk_sub(t[2 * k], offp), p1 = pk_sub(t[2 * k + 1], offp), h = perm(p1, p0, 0x07050301u);
      lo[k] = perm(p1, p0, 0x06040200u); code |= (h & 0x03030303u) << (2 * k); U[k] = h & 0x7C7C7C7Cu;
    }
    if (act) { bst128(rb, 1024 + 16 * g, lo); bst32(rb, 1024 + 16 * G + 4 * g, code); }
    emit_exc(U, act, g * 16, 99, reg + E_EXC, OUT_EXC_CAP, E, bad);
  }
  cbar();
  const bool raw = __builtin_amdgcn_ballot_w64(bad) != 0 || E > OUT_EXC_CAP || (a.flags & 2);
  if (raw) {
    waitv<0>();
#pragma unroll
    for (int c = 0; c < 16; ++c) bst128(rb, c * 1024 + dl * 16, (u32x4){o[4 * c], o[4 * c + 1], o[4 * c + 2], o[4 * c + 3]});
  } else {
    const u32 eo = 1024 + 16 * G + ((4 * G + 15) & ~15u);
    for (u32 i = lane * 16; i < 2 * E; i += 1024) bst128(rb, eo + i, lds_r128(reg + E_EXC + i));
  }
  cbar();
  u32 u = 0;
  if (lane == 0) u = __hip_atomic_fetch_add((GAS u32*)(a.ws[a.rank] + WS_TICKET), 1u, __ATOMIC_RELAXED, __HIP_MEMORY_SCOPE_AGENT);
  waitv<0>();
  u = rfl(u) & 2047;
  const u32x4 d = {call, n | (off << 16), (raw ? 0x80000000u : E) | (blob << 16), call};
  if (lane < NR && lane != a.rank) st_sys128(a.ws[lane] + WS_DESCO + (a.rank * 2048 + u) * 16, d);
  const u32 tile = blob >> 3, w8 = blob & 7;
  const u32 so = ((tile >> 4) * 256 + (w8 & 3) * 32) * 65536u + (a.rank * 4096 + (tile & 15) * 256 + (w8 >> 2) * 128) * 2u;
  const rsrc_t rout = mkrsrc(a.out + so, OUT_BYTES - so);
  const u32 vo = (lane & 15) * 65536u + (lane >> 4) * 16;
#pragma unroll
  for (int ch = 0; ch < 16; ++ch)
    __builtin_amdgcn_raw_buffer_store_b128((u32x4){o[4 * ch], o[4 * ch + 1], o[4 * ch + 2], o[4 * ch + 3]}, rout,
                                           vo + (ch & 3) * 64 + ((ch >> 3) * 128 + ((ch >> 2) & 1) * 16) * 65536u, 0, OWN_AUX);
}

DEV void wait_ready(const GAS u32* cp, u32 need, u64 t0, u32& bad) {
  while (!bad) {
    const u32 c = __hip_atomic_load(cp, __ATOMIC_RELAXED, __HIP_MEMORY_SCOPE_AGENT);
    if (__builtin_amdgcn_ballot_w64((int)(c - need) < 0) == 0) break;
    if (now() - t0 > DEADLINE) bad = 1;
    __builtin_amdgcn_s_sleep(4);
  }
  cbar();
}
DEV void gemm_role(const Args& a, LAS char* lds, int pos, int wid, u32 call, u64 t0) {
  char* const wsl = a.ws[a.rank];
  const int wr = wid >> 2, wc = wid & 3;
  u32 bad = 0;
  for (int round = 0; round < 2; ++round) {
    int tid = threadIdx.x; asm volatile("" : "+v"(tid));
    int lane = tid & 63;
    const int tile = a.tiles[round * 128 + pos], mt = tile >> 4, nt = tile & 15;
    const GAS u32* cp = (const GAS u32*)(wsl + WS_READY + ((lane & 1) ? 4 + (nt >> 2) : mt >> 2) * 9 * 128);
    if (wid == 0) wait_ready(cp, 64u * call, t0, bad);
    asm volatile("s_waitcnt lgkmcnt(0)" ::: "memory");
    __builtin_amdgcn_s_barrier();
    STAMP_AT(2 + round * 3);
    f32x4 acc[2][2][4][2];
#pragma unroll
    for (int i0 = 0; i0 < 2; ++i0)
#pragma unroll
      for (int i1 = 0; i1 < 2; ++i1)
#pragma unroll
        for (int m = 0; m < 4; ++m)
#pragma unroll
          for (int n = 0; n < 2; ++n) acc[i0][i1][m][n] = (f32x4){0.f, 0.f, 0.f, 0.f};
    bf16x8 At[4][2], B0[2][2], B1[2][2];
    const char* cA = wsl + WS_IMG + IMG_BYTES + (size_t)nt * (2u << 20);
    const char* cB = wsl + WS_IMG + (size_t)mt * (2u << 20);
    const u32 voff = tid * 16, ldsw = wid * 1024;
    const int aoff = wr * 8192 + lane * 16, boff = wc * 4096 + lane * 16;
    constexpr int HTB = 16384, KST = 32768, NT = 64;
#define SA(b, h) (((b) * 2 + (h)) * HTB)
#define SB(b, h) ((4 + (b) * 2 + (h)) * HTB)
#define STAGE(bufoff, gbase) do { _Pragma("unroll") for (int _i = 0; _i < 2; ++_i) \
    __builtin_amdgcn_global_load_lds((const GAS u32*)((gbase) + _i * 8192 + voff), (LAS u32*)(lds + (bufoff) + ldsw + _i * 8192), 16, 0, GL_AUX); } while (0)
#define LDA(dst, b, h) do { _Pragma("unroll") for (int m = 0; m < 4; ++m) _Pragma("unroll") for (int k = 0; k < 2; ++k) \
    dst[m][k] = *(const LAS bf16x8*)(lds + SA(b, h) + aoff + m * 2048 + k * 1024); } while (0)
#define LDB(dst, b, h) do { _Pragma("unroll") for (int n = 0; n < 2; ++n) _Pragma("unroll") for (int k = 0; k < 2; ++k) \
    dst[n][k] = *(const LAS bf16x8*)(lds + SB(b, h) + boff + n * 2048 + k * 1024); } while (0)
#define MMA(ai, bj, A_, B_) do { __builtin_amdgcn_s_setprio(1); _Pragma("unroll") for (int m = 0; m < 4; ++m) _Pragma("unroll") for (int n = 0; n < 2; ++n) \
    _Pragma("unroll") for (int k = 0; k < 2; ++k) acc[ai][bj][m][n] = __builtin_amdgcn_mfma_f32_16x16x32_bf16(A_[m][k], B_[n][k], acc[ai][bj][m][n], 0, 0, 0); \
    __builtin_amdgcn_s_setprio(0); } while (0)
#define WAIT_V(n) asm volatile("s_waitcnt vmcnt(" #n ")" ::: "memory")
#define WAIT_L(n) asm volatile("s_waitcnt lgkmcnt(" #n ")" ::: "memory")
#define BAR __builtin_amdgcn_s_barrier()
#define SCHED __builtin_amdgcn_sched_barrier(0)
    STAGE(SB(0, 0), cB); STAGE(SA(0, 0), cA); STAGE(SB(0, 1), cB + HTB); STAGE(SA(0, 1), cA + HTB);
    if (wr == 1) BAR;
    WAIT_V(4); BAR;
    STAGE(SB(1, 0), cB + KST); STAGE(SA(1, 0), cA + KST); STAGE(SB(1, 1), cB + HTB + KST);
    WAIT_V(6); BAR;
    int tn = 5;
    for (int t = 0; t < NT; t += 2) {
      if (wid == 0 && tn < 61) {
        if (t >= tn) {
          const u32 c = lds_poll(LDS_BYTES - 32 + (lane & 1) * 4);
          STAMP_AT(16 + round * 16 + (tn / 7) * 2);
          if (__builtin_amdgcn_ballot_w64((int)(c - 56u * call) < 0) != 0) wait_ready(cp, 56u * call, t0, bad);
          STAMP_AT(17 + round * 16 + (tn / 7) * 2);
          tn += 7;
        } else if (t + 2 >= tn) {
          cp += 32;
          if (lane < 2) __builtin_amdgcn_global_load_lds(cp, (LAS u32*)(lds + LDS_BYTES - 32), 4, 0, 17);
        }
      }
      const bool last = (t == NT - 2);
      const char* a1 = cA + (size_t)(t + 1) * KST;
      const char* a2 = last ? cA : cA + (size_t)(t + 2) * KST; const char* b2 = last ? cB : cB + (size_t)(t + 2) * KST;
      const char* a3 = a2 + KST; const char* b3 = b2 + KST;
      LDB(B0, 0, 0); SCHED; LDA(At, 0, 0); STAGE(SA(1, 1), a1 + HTB);
      WAIT_L(8); BAR; WAIT_L(0); MMA(0, 0, At, B0); BAR; SCHED;
      LDB(B1, 0, 1); STAGE(SB(0, 0), b2);
      BAR; WAIT_L(0); MMA(0, 1, At, B1); BAR;
      LDA(At, 0, 1); STAGE(SA(0, 0), a2);
      BAR; WAIT_L(0); MMA(1, 0, At, B0); BAR; SCHED;
      STAGE(SB(0, 1), b2 + HTB);
      WAIT_V(6); BAR; MMA(1, 1, At, B1); BAR;
      LDB(B0, 1, 0); SCHED; LDA(At, 1, 0); STAGE(SA(0, 1), a2 + HTB);
      WAIT_L(8); BAR; WAIT_L(0); MMA(0, 0, At, B0); BAR; SCHED;
      LDB(B1, 1, 1); STAGE(SB(1, 0), b3);
      BAR; WAIT_L(0); MMA(0, 1, At, B1); BAR;
      LDA(At, 1, 1); STAGE(SA(1, 0), a3);
      BAR; WAIT_L(0); MMA(1, 0, At, B0); BAR; SCHED;
      STAGE(SB(1, 1), b3 + HTB);
      WAIT_V(6); BAR; MMA(1, 1, At, B1); BAR;
    }
    if (wid == 0) lds_post(LDS_BYTES - 16, bad);
    asm volatile("s_waitcnt vmcnt(0) lgkmcnt(0)" ::: "memory");
    if (wr == 0) BAR;
    BAR;
#undef SA
#undef SB
#undef STAGE
#undef LDA
#undef LDB
#undef MMA
    int tz = threadIdx.x; asm volatile("" : "+v"(tz));
    lane = tz & 63;
    STAMP_AT(3 + round * 3);
    if (rfl(lds_poll(LDS_BYTES - 16))) {
#pragma unroll
      for (int i = 0; i < 32; ++i) acc[i >> 4][(i >> 3) & 1][(i >> 1) & 3][i & 1] = (f32x4){__builtin_nanf(""), __builtin_nanf(""), __builtin_nanf(""), __builtin_nanf("")};
    }
    pack_out(a, acc, tile * 8 + wid, call, lds + wid * EREG, lane);
    STAMP_AT(4 + round * 3);
  }
}

DEV void loader_role(const Args& a, LAS char* lds, int peer, int lane, u64 t0) {
  const char* src = a.ws[peer] + WS_BLOB + lane * 16;
  u32 served = 0; int pend = -1; u32 pend_tag = 0;
  int done = 0;
  for (int left = NDEC; left > 0;) {
    const u32 r = lds_poll(L_REQ + (lane & 7) * 4);
    u32 key = ((lane & 7) < NDEC && (r & 255) != served) ? ((255 - (r & 255)) << 3 | (7 - (lane & 7))) + 1 : 0;
    key = umax(key, dpp0<0x111, 0xf>(key)); key = umax(key, dpp0<0x112, 0xf>(key)); key = umax(key, dpp0<0x114, 0xf>(key));
    key = (u32)__builtin_amdgcn_readlane((int)key, 7);
    if (key == 0) {
      if (pend >= 0) { waitv<0>(); lds_post(L_FULL + pend * 4, pend_tag); pend = -1; }
      else { if (now() - t0 > DEADLINE + 20000000ull) break; __builtin_amdgcn_s_sleep(2); }
      continue;
    }
    const int j = 7 - ((key - 1) & 7);
    const u32 rq = (u32)__builtin_amdgcn_readlane((int)r, j);
    if ((lane & 7) == j) served = rq & 255;
    if (rq >> 31) { --left; continue; }
    ++done;
    const u32 slot = umin(rq >> 18, NSLOT - 1), bytes = (((rq >> 8) & 1023) + 1) * 16;
    const char* s = src + (size_t)slot * 16384;
    LAS char* dst = lds + j * DREG + 8192;
    int row = 0;
    for (u32 o = 0; o < bytes; o += 1024, ++row) {
      if (o + lane * 16 < bytes) __builtin_amdgcn_global_load_lds((const GAS u32*)(s + o), (LAS u32*)(dst + o), 16, 0, LD_AUX);
      waitv<WIN - 1>();
      if (row == WIN - 2 && pend >= 0) { lds_post(L_FULL + pend * 4, pend_tag); pend = -1; }
    }
    if (pend >= 0) { waitv<0>(); lds_post(L_FULL + pend * 4, pend_tag); }
    pend = j; pend_tag = rq & 255;
  }
  waitv<0>();
  if (pend >= 0) lds_post(L_FULL + pend * 4, pend_tag);
}

DEV void decoder_role(const Args& a, LAS char* lds, int peer, int c, int j, int lane, u32 call, u64 t0) {
  char* const wsl = a.ws[a.rank];
  LAS char* A = lds + j * DREG;
  LAS char* B = A + 8192;
  const int q = (peer - a.rank) & 7;
  bool fail = false;
  u32 tag = 0;
  for (int m = j;; m += NDEC) {
    const char* dsc = wsl + WS_DESC + (size_t)((peer * 16 + c) * NIN + m) * 16;
    tag = tag == 255 ? 1 : tag + 1;
    if (m >= NIN) {
      u32 u = 0;
      if (lane == 0) u = __hip_atomic_fetch_add((GAS u32*)(wsl + WS_QUEUE + peer * 128), 1u, __ATOMIC_RELAXED, __HIP_MEMORY_SCOPE_AGENT);
      u = rfl(u) - (call - 1) * QSTEP;
      if (u >= 2048) break;
      dsc = wsl + WS_DESCO + (size_t)(peer * 2048 + u) * 16;
    }
    u32x4 d;
    for (;;) {
      d = ld_sys128(dsc);
      if (rfl(d.x) == call && rfl(d.w) == call) break;
      if (now() - t0 > DEADLINE) { fail = true; break; }
      __builtin_amdgcn_s_sleep(4);
    }
    if (fail) break;
    const u32 da = rfl(d.y), db = rfl(d.z);
    const bool raw = db >> 31;
    u32 slot, bytes, E, G = 0;
    const int kj = m < 24 ? m / 3 : m - 24, sl = m < 24 ? m - kj * 3 : 3;
    const int t = sl < 2 ? c >> 3 : sl - 2, s = sl < 2 ? sl * 4 + ((c >> 1) & 3) : 8 + (c >> 1);
    if (m < NIN) {
      E = umin(da >> 16, IN_EXC_CAP);
      slot = (t ? SLOT_W + a.rank * 256 : 0) + s * 16 + kj * 2 + (c & 1);
      bytes = 11264 + ((2 * E + 15) & ~15u);
    } else {
      E = umin(db & 0xFFFF, OUT_EXC_CAP);
      G = (umin(da & 0xFFFF, 8192) + 15) >> 4;
      slot = SLOT_O + ((db >> 16) & 2047);
      bytes = 1024 + 16 * G + ((4 * G + 15) & ~15u) + ((2 * E + 15) & ~15u);
    }
    if (raw) bytes = 16384;
    lds_post(L_REQ + j * 4, (slot << 18) | ((bytes / 16 - 1) << 8) | tag);
    for (;;) {
      if (rfl(lds_poll(L_FULL + j * 4)) == tag) break;
      if (now() - t0 > DEADLINE) { fail = true; break; }
      __builtin_amdgcn_s_sleep(2);
    }
    if (fail) break;
    if (tag <= 12) STAMP_AT(13 + tag * 4);
    if (m < NIN) {
      const u32 base = da & 0xFFFF, bb = base * 0x10001u, upz = ((65536 - base) >> 11) & 31;
      u32 H[32];
      if (raw) {
#pragma unroll
        for (int r = 0; r < 8; ++r) { const u32x4 t = lds_r128(B + (8 + r) * 1024 + lane * 16); H[4 * r] = t.x; H[4 * r + 1] = t.y; H[4 * r + 2] = t.z; H[4 * r + 3] = t.w; }
      } else {
        for (u32 i = lane; i < E; i += 64) {
          const u32 en = *(LAS u16*)(B + 11264 + 2 * i), cc = en & 7;
          *(LAS unsigned char*)(A + (en >> 3)) = (unsigned char)((cc ? 32 - cc : upz) << 3);
        }
        cbar();
        u32 P[12];
#pragma unroll
        for (int r = 0; r < 3; ++r) { const u32x4 t = lds_r128(B + (8 + r) * 1024 + lane * 16); P[4 * r] = t.x; P[4 * r + 1] = t.y; P[4 * r + 2] = t.z; P[4 * r + 3] = t.w; }
#pragma unroll
        for (int r = 0; r < 8; ++r) { const u32x4 t = lds_r128(A + r * 1024 + lane * 16); H[4 * r] = t.x; H[4 * r + 1] = t.y; H[4 * r + 2] = t.z; H[4 * r + 3] = t.w; }
        cbar();
        for (u32 i = lane; i < E; i += 64) *(LAS unsigned char*)(A + (*(LAS u16*)(B + 11264 + 2 * i) >> 3)) = 0;
        cbar();
#pragma unroll
        for (int g = 0; g < 4; ++g) {
          u32* h = H + 8 * g; const u32 p0 = P[3 * g], p1 = P[3 * g + 1], p2 = P[3 * g + 2], m7 = 0x07070707u;
          h[0] |= p0 & m7; h[1] |= (p0 >> 3) & m7; h[2] |= ((p0 >> 6) & 0x03030303u) | ((p2 >> 4) & 0x04040404u);
          h[3] |= p1 & m7; h[4] |= (p1 >> 3) & m7; h[5] |= ((p1 >> 6) & 0x03030303u) | ((p2 >> 5) & 0x04040404u);
          h[6] |= p2 & m7; h[7] |= (p2 >> 3) & m7;
        }
      }
      const u32 so = t * IMG_BYTES + (u32)(s * 64 + 7 + kj * 7 + q) * 32768u + (c & 1) * (t ? 8192 : 16384);
      const rsrc_t rimg[2] = {mkrsrc(wsl + WS_IMG + so, 8192), mkrsrc(wsl + WS_IMG + so + (t ? 16384 : 8192), 8192)};
#pragma unroll
      for (int r = 0; r < 8; ++r) {
        const u32x4 lo = lds_r128(B + r * 1024 + lane * 16);
        u32 v[8];
#pragma unroll
        for (int k = 0; k < 4; ++k) {
          const u32 u0 = pk_add(perm(H[4 * r + k], lo[k], 0x05010400u), bb), u1 = pk_add(perm(H[4 * r + k], lo[k], 0x07030602u), bb);
          v[2 * k] = __builtin_amdgcn_alignbit(u0, u0, 1); v[2 * k + 1] = __builtin_amdgcn_alignbit(u1, u1, 1);
        }
        if (!NOIMG || call < 3) {
        ist128(rimg[r >> 2], lane * 16 + ((2 * r) & 7) * 1024, (u32x4){v[0], v[1], v[2], v[3]});
        ist128(rimg[r >> 2], lane * 16 + ((2 * r + 1) & 7) * 1024, (u32x4){v[4], v[5], v[6], v[7]});
        }
      }
      if (tag <= 12) STAMP_AT(14 + tag * 4);
      waitv<0>();
      if (tag <= 12) STAMP_AT(15 + tag * 4);
      if (lane == 0) __hip_atomic_fetch_add((GAS u32*)(wsl + WS_READY + ((t * 4 + (s >> 2)) * 9 + 1 + kj) * 128), 1u, __ATOMIC_RELAXED, __HIP_MEMORY_SCOPE_AGENT);
    } else {
      const u32 blob = (db >> 16) & 2047, tile = blob >> 3, w8 = blob & 7;
      const u32 so = ((tile >> 4) * 256 + (w8 & 3) * 32) * 65536u + (peer * 4096 + (tile & 15) * 256 + (w8 >> 2) * 128) * 2u;
      const u32 vo = (lane >> 2) * 65536u + (lane & 3) * 16;
      const rsrc_t rout = mkrsrc(a.out + so, OUT_BYTES - so);
      if (raw) {
#pragma unroll
        for (int ch = 0; ch < 16; ++ch)
          __builtin_amdgcn_raw_buffer_store_b128(lds_r128(B + ch * 1024 + lane * 16), rout, vo + (ch & 3) * 64 + ((ch >> 3) * 128 + ((ch >> 2) & 1) * 16) * 65536u, 0, OUT_AUX);
      } else {
        const u32 offp = ((da >> 16) & 0x7FFF) * 0x10001u;
        const u32x4 bits = lds_r128(B + lane * 16);
        cbar();
        for (u32 g = lane; g < G; g += 64) {
          const u32x4 lo = lds_r128(B + 1024 + 16 * g); const u32 code = *(LAS u32*)(B + 1024 + 16 * G + 4 * g);
          u32 v[8];
#pragma unroll
          for (int k = 0; k < 4; ++k) {
            const u32 h = (code >> (2 * k)) & 0x03030303u;
            v[2 * k] = pk_add(perm(h, lo[k], 0x05010400u), offp); v[2 * k + 1] = pk_add(perm(h, lo[k], 0x07030602u), offp);
          }
          lds_w128(A + 32 * g, (u32x4){v[0], v[1], v[2], v[3]}); lds_w128(A + 32 * g + 16, (u32x4){v[4], v[5], v[6], v[7]});
          cbar();
        }
        LAS char* ex = B + 1024 + 16 * G + ((4 * G + 15) & ~15u);
        for (u32 i = lane; i < E; i += 64) {
          const u32 en = *(LAS u16*)(ex + 2 * i); LAS u16* p = (LAS u16*)(A + 2 * (en >> 3));
          *p = (u16)(*p - ((en & 7) << 10));
        }
        cbar();
        const u32 cnt = __builtin_popcount(bits.x) + __builtin_popcount(bits.y) + __builtin_popcount(bits.z) + __builtin_popcount(bits.w);
        u32 pos = wave_scan(cnt) - cnt;
#pragma unroll
        for (int ch = 0; ch < 16; ++ch) {
          u32x4 ov;
#pragma unroll
          for (int h = 0; h < 2; ++h) {
            const u32 nib = (bits[ch >> 2] >> (28 - 4 * ((ch & 3) * 2 + h))) & 15;
            const u32x2 sel = *(LAS u32x2*)(lds + L_LUT + nib * 8);
            const u32 ad = 2 * pos; LAS u32* q = (LAS u32*)(A + (ad & ~3u));
            const u32 d0 = q[0], d1 = q[1], d2 = q[2];
            const u32 e0 = __builtin_amdgcn_alignbyte(d1, d0, ad), e1 = __builtin_amdgcn_alignbyte(d2, d1, ad);
            ov[2 * h] = perm(e1, e0, sel.x); ov[2 * h + 1] = perm(e1, e0, sel.y);
            pos += __builtin_popcount(nib);
          }
          __builtin_amdgcn_raw_buffer_store_b128(ov, rout, vo + (ch & 3) * 64 + ((ch >> 3) * 128 + ((ch >> 2) & 1) * 16) * 65536u, 0, OUT_AUX);
        }
      }
      cbar();
      if (tag <= 12) STAMP_AT(14 + tag * 4);
    }
  }
  if (fail) tag = tag == 255 ? 1 : tag + 1;
  lds_post(L_REQ + j * 4, 0x80000000u | tag);
  if (fail) {
    waitv<0>();
    __builtin_amdgcn_raw_buffer_store_b32(0x7FC07FC0u, mkrsrc(a.out, OUT_BYTES), lane * 4 + (blockIdx.x * 8 + j) * 65536u, 0, 0);
    if (lane == 0) __hip_atomic_store((GAS u32*)(wsl + WS_TMO), call, __ATOMIC_RELAXED, __HIP_MEMORY_SCOPE_AGENT);
  }
}

__global__ void __launch_bounds__(512, 2) fused_gemm_ar(const Args a) {
  extern __shared__ __attribute__((aligned(16))) char smem[];
  LAS char* lds = (LAS char*)smem;
  const u64 t0 = now();
  const int wid = __builtin_amdgcn_readfirstlane(threadIdx.x >> 6), lane = threadIdx.x & 63;
  const int xcd = vbid() & 7, li = vbid() >> 3;
  char* const wsl = a.ws[a.rank];
  GAS u32* callp = (GAS u32*)(wsl + WS_CALL) + blockIdx.x * 8 + wid;
  const u32 callv = __hip_atomic_load(callp, __ATOMIC_RELAXED, __HIP_MEMORY_SCOPE_AGENT);
  char* const wsp = a.ws[lane & 7];
  STAMP_AT(0);
  const bool is_gemm = li < 16;
  const int pos = xcd * 16 + (li & 15);
  if (!is_gemm) {
    if (wid == 0) {
      if (lane < 32) *(LAS u32*)(lds + L_CTRL + lane * 4) = 0;
      if (lane < 16) {
        u32 k = 0, sel[2] = {0, 0};
#pragma unroll
        for (int i = 0; i < 4; ++i) {
          const bool on = (lane >> (3 - i)) & 1;
          sel[i >> 1] |= (on ? (2 * k) | ((2 * k + 1) << 8) : 0x0C0Cu) << (16 * (i & 1));
          k += on;
        }
        *(LAS u32x2*)(lds + L_LUT + lane * 8) = (u32x2){sel[0], sel[1]};
      }
    } else if (wid <= NDEC) {
#pragma unroll
      for (int r = 0; r < 8; ++r) lds_w128(lds + (wid - 1) * DREG + r * 1024 + lane * 16, (u32x4){0, 0, 0, 0});
    }
    asm volatile("s_waitcnt lgkmcnt(0)" ::: "memory");
    __builtin_amdgcn_s_barrier();
  }
  const int q = li - 16;
  const bool is_help = !is_gemm && (q & 7) == 0;
  {
    int e = -1, n = 1;
    if (is_gemm) { e = wid == 0 ? 1152 + pos : 448 + (wid - 1) * 128 + pos; if (wid && e >= 1152) e = -1; }
    else if (is_help) { e = 1280 + (xcd * 2 + (q >> 3)) * 8 + wid; n = 8; }
    else if (wid >= 4) e = (wid - 4) * 112 + (xcd * 2 + (q >> 3)) * 7 + (q & 7) - 1;
    LAS char* scr = is_gemm || is_help ? lds + wid * EREG : wid == 7 ? lds + L_SCR7 : lds + (wid - 1) * DREG + 8192;
    if (GDELAY && (is_gemm || is_help)) __builtin_amdgcn_s_sleep(GDELAY);
    if (e >= 0)
      for (int i = 0; i < n; ++i, e += 128) {
        bool isx = false; int j, s, dq;
        if (e < 1152) { j = e / 144; const int r = e - j * 144; isx = r < 32; s = isx ? r >> 1 : (r >> 1) & 7; dq = (r >> 4) - 1; }
        else if (e < 1280) { j = (e >> 4) & 7; s = (e >> 1) & 7; dq = 0; }
        else { j = (e - 1280) >> 7; s = 8 + ((e >> 1) & 7); dq = e >> 4; }
        prep_piece(a, isx, s, j, e & 1, (a.rank + dq) & 7, callv, wsp, scr, lane);
        if (is_help) STAMP_AT(16 + i);
      }
  }
  const u32 call = rfl(callv) + 1;
  if (lane == 0) __hip_atomic_store(callp, call, __ATOMIC_RELAXED, __HIP_MEMORY_SCOPE_AGENT);
  STAMP_AT(1);
  if (is_gemm) {
    gemm_role(a, lds, pos, wid, call, t0);
  } else if (!is_help) {
    const int peer = (a.rank + (q & 7)) & 7, c = xcd * 2 + (q >> 3);
    if (wid == 0) loader_role(a, lds, peer, lane, t0);
    else if (wid <= NDEC) decoder_role(a, lds, peer, c, wid - 1, lane, call, t0);
  }
  STAMP_AT(8);
}

#define HIP_CHECK(expr) do { hipError_t _e = (expr); if (_e != hipSuccess) { \
    fprintf(stderr, "HIP error %s at %s:%d\n", hipGetErrorString(_e), __FILE__, __LINE__); \
    return; } } while (0)

#define DBG_FLAGS 0

extern "C" void kernel_launch(int rank, int nranks, ncclComm_t comm, hipStream_t stream,
                              void* const* d_in, const int* in_sizes, int n_in,
                              void* d_out, int out_size, const int* in_dtypes, int out_dtype,
                              void* const* d_out_all, void* const* d_ws_all, size_t ws_bytes) {
  if (nranks != NR || (unsigned)rank >= (unsigned)NR || n_in < 2 || ws_bytes < WS_USED || (size_t)out_size * 2 != OUT_BYTES ||
      in_sizes[0] != 4096 * 512 || in_sizes[1] != 512 * 32768) {
    fprintf(stderr, "unexpected problem shape\n");
    return;
  }
  static bool attr_set[NR] = {};
  if (!attr_set[rank]) {
    HIP_CHECK(hipFuncSetAttribute((const void*)fused_gemm_ar, hipFuncAttributeMaxDynamicSharedMemorySize, LDS_BYTES));
    attr_set[rank] = true;
  }
  Args a;
  a.x = (const char*)d_in[0]; a.w = (const char*)d_in[1]; a.out = (char*)d_out;
  for (int p = 0; p < NR; ++p) a.ws[p] = (char*)d_ws_all[p];
  a.rank = rank; a.flags = DBG_FLAGS;
  static const int blk[8][2] = {{0, 0}, {1, 0}, {0, 1}, {1, 1}, {2, 0}, {2, 1}, {3, 0}, {3, 1}};
  for (int k = 0; k < 256; ++k) {
    const int b = (k >> 4) & 7, i = k & 15;
    a.tiles[k] = (unsigned char)((blk[b][0] * 4 + (i >> 2)) * 16 + (blk[b][1] + (k >> 7) * 2) * 4 + (i & 3));
  }
  hipLaunchKernelGGL(fused_gemm_ar, dim3(256), dim3(512), LDS_BYTES, stream, a);
  HIP_CHECK(hipGetLastError());
}
```

```cpp
#include <hip/hip_runtime.h>
#include <rccl/rccl.h>
#include <cstdio>

typedef unsigned u32;
typedef unsigned long long u64;
typedef u32 u32x4 __attribute__((ext_vector_type(4)));
typedef u32 u32x2 __attribute__((ext_vector_type(2)));
typedef unsigned short u16;
typedef u16 u16x2 __attribute__((ext_vector_type(2)));
typedef short bf16x8 __attribute__((ext_vector_type(8)));
typedef float f32x4 __attribute__((ext_vector_type(4)));
typedef float f32x2 __attribute__((ext_vector_type(2)));
typedef __bf16 bf16x2 __attribute__((ext_vector_type(2)));
#define LAS __attribute__((address_space(3)))
#define GAS __attribute__((address_space(1)))
#define DEV __device__ __forceinline__

constexpr int NR = 8;
constexpr u32 WS_CALL = 0;
constexpr u32 WS_TMO = 8192;
constexpr u32 WS_TICKET = 8192 + 128;
constexpr u32 WS_QUEUE = 8192 + 256;
constexpr u32 WS_READY = 16384;
constexpr int NIN = 32;
constexpr u32 WS_DESC = 512u << 10;
constexpr u32 WS_DESCO = WS_DESC + 65536;
constexpr u32 WS_IMG = 1u << 20;
constexpr u32 IMG_BYTES = 32u << 20;
constexpr u32 WS_BLOB = 65u << 20;
constexpr u32 SLOT_W = 256, SLOT_O = 2304, NSLOT = 4352;
constexpr u32 WS_USED = WS_BLOB + NSLOT * 16384u;
constexpr u32 WS_STAMP = WS_USED;
constexpr u32 OUT_BYTES = 4096u * 32768u * 2u;
constexpr u64 DEADLINE = 100000000ull;
constexpr int LDS_BYTES = 163840;
constexpr int NDEC = 6;
constexpr int DREG = 26624, BOFF = 10240;
constexpr int L_CTRL = NDEC * DREG;
constexpr int L_REQ = L_CTRL, L_FULL = L_CTRL + 64, L_LUT = L_CTRL + 128, L_SCR7 = L_CTRL + 256;
constexpr int EREG = 20480;
constexpr int E_EXC = 16448;
constexpr u32 IN_EXC_CAP = 1792, OUT_EXC_CAP = 1984;
constexpr u32 QSTEP = 2048 + 16 * NDEC;
#define WIN 5
#define GDELAY 0
#define OWN_AUX 0
#define OUT_AUX 0
#define IMG_AUX 17
#define GL_AUX 17
#define PL_AUX 2
#define SEG 0
DEV int vbid() {
  const int x = blockIdx.x & 7, l = blockIdx.x >> 3;
  if (SEG == 0) return blockIdx.x;
  const int comm = SEG == 1 ? (x & 1) ^ 1 : SEG == 2 ? x & 1 : SEG == 3 ? x >> 2 : (x >> 1) & 1;
  const int g = SEG <= 2 ? x >> 1 : SEG == 3 ? x & 3 : (x & 1) | ((x >> 2) << 1);
  return comm * 128 + (l & 15) * 8 + g * 2 + (l >> 4);
}
#define LD_AUX 17
#define NOIMG 0

struct Args {
  const char* x; const char* w; char* out; char* ws[NR]; int rank; int flags; unsigned char tiles[256];
};

template <int N> DEV void waitv() { asm volatile("s_waitcnt vmcnt(%0)" ::"n"(N) : "memory"); }
DEV void cbar() { asm volatile("" ::: "memory"); }
DEV u64 now() { return __builtin_amdgcn_s_memrealtime(); }
DEV u32 perm(u32 hi, u32 lo, u32 sel) { return __builtin_amdgcn_perm(hi, lo, sel); }
DEV u32 pk_sub(u32 a, u32 b) { return __builtin_bit_cast(u32, __builtin_bit_cast(u16x2, a) - __builtin_bit_cast(u16x2, b)); }
DEV u32 pk_add(u32 a, u32 b) { return __builtin_bit_cast(u32, __builtin_bit_cast(u16x2, a) + __builtin_bit_cast(u16x2, b)); }
DEV u32 pk_max(u32 a, u32 b) { return __builtin_bit_cast(u32, __builtin_elementwise_max(__builtin_bit_cast(u16x2, a), __builtin_bit_cast(u16x2, b))); }
DEV u32 pk_min(u32 a, u32 b) { return __builtin_bit_cast(u32, __builtin_elementwise_min(__builtin_bit_cast(u16x2, a), __builtin_bit_cast(u16x2, b))); }
template <int C, int RM> DEV u32 dpp0(u32 x) { return (u32)__builtin_amdgcn_update_dpp(0, (int)x, C, RM, 0xf, false); }
DEV u32 umax(u32 a, u32 b) { return a > b ? a : b; }
DEV u32 umin(u32 a, u32 b) { return a < b ? a : b; }
DEV u32 wave_max(u32 x) {
  x = umax(x, dpp0<0x111, 0xf>(x)); x = umax(x, dpp0<0x112, 0xf>(x)); x = umax(x, dpp0<0x114, 0xf>(x)); x = umax(x, dpp0<0x118, 0xf>(x));
  x = umax(x, dpp0<0x142, 0xa>(x)); x = umax(x, dpp0<0x143, 0xc>(x));
  return (u32)__builtin_amdgcn_readlane((int)x, 63);
}
DEV u32 wave_scan(u32 x) {
  x += dpp0<0x111, 0xf>(x); x += dpp0<0x112, 0xf>(x); x += dpp0<0x114, 0xf>(x); x += dpp0<0x118, 0xf>(x);
  x += dpp0<0x142, 0xa>(x); x += dpp0<0x143, 0xc>(x);
  return x;
}
DEV u32 rfl(u32 x) { return (u32)__builtin_amdgcn_readfirstlane((int)x); }
DEV u32 mbcnt(u64 m) { return __builtin_amdgcn_mbcnt_hi((u32)(m >> 32), __builtin_amdgcn_mbcnt_lo((u32)m, 0)); }
DEV u32x4 lds_r128(LAS char* p) { return *(LAS u32x4*)p; }
DEV void lds_w128(LAS char* p, u32x4 v) { *(LAS u32x4*)p = v; }
DEV u32 lds_poll(u32 addr) { u32 v; asm volatile("ds_read_b32 %0, %1\n\ts_waitcnt lgkmcnt(0)" : "=v"(v) : "v"(addr) : "memory"); return v; }
DEV void lds_post(u32 addr, u32 v) { asm volatile("ds_write_b32 %0, %1" ::"v"(addr), "v"(v) : "memory"); }
DEV void st_sys128(void* p, u32x4 v) { asm volatile("global_store_dwordx4 %0, %1, off sc0 sc1\n\ts_nop 1" ::"v"(p), "v"(v) : "memory"); }
DEV u32x4 ld_sys128(const void* p) { u32x4 v; asm volatile("global_load_dwordx4 %0, %1, off sc0 sc1\n\ts_waitcnt vmcnt(0)" : "=&v"(v) : "v"(p) : "memory"); return v; }
typedef __amdgpu_buffer_rsrc_t rsrc_t;
DEV rsrc_t mkrsrc(const void* p, u32 bytes) { return __builtin_amdgcn_make_buffer_rsrc((void*)p, 0, (int)bytes, 0x00020000); }
DEV void bst128(rsrc_t r, u32 voff, u32x4 v) { __builtin_amdgcn_raw_buffer_store_b128(v, r, (int)voff, 0, 17); }
DEV void ist128(rsrc_t r, u32 voff, u32x4 v) { __builtin_amdgcn_raw_buffer_store_b128(v, r, (int)voff, 0, IMG_AUX); }
DEV void bst32(rsrc_t r, u32 voff, u32 v) { __builtin_amdgcn_raw_buffer_store_b32(v, r, (int)voff, 0, 17); }

#define STAMP_AT(k) do { } while (0)

DEV void emit_exc(const u32 (&U)[4], bool act, u32 gidx, u32 upz, LAS char* list, u32 cap, u32& E, bool& bad) {
  u32 F = (((U[0] + 0x7C7C7C7Cu) & 0x80808080u) >> 7) | (((U[1] + 0x7C7C7C7Cu) & 0x80808080u) >> 6) |
          (((U[2] + 0x7C7C7C7Cu) & 0x80808080u) >> 5) | (((U[3] + 0x7C7C7C7Cu) & 0x80808080u) >> 4);
  if (!act) F = 0;
  for (;;) {
    const u64 bal = __builtin_amdgcn_ballot_w64(F != 0);
    if (bal == 0) break;
    if (F != 0) {
      const u32 p = __builtin_ctz(F), k = p & 7, b = p >> 3;
      const u32 h0 = (p & 1) ? U[1] : U[0], h1 = (p & 1) ? U[3] : U[2], h = (p & 2) ? h1 : h0;
      const u32 up = (h >> (8 * b + 2)) & 31;
      u32 c = 32 - up;
      if (c > 7) { bad |= up != upz; c = 0; }
      const u32 slot = E + mbcnt(bal);
      if (slot < cap) *(LAS u16*)(list + 2 * slot) = (u16)(((gidx + 4 * k + b) << 3) | c);
      F &= F - 1;
    }
    E += __builtin_popcountll(bal);
  }
}

DEV void prep_piece(const Args& a, int mode, bool isx, int s, int j, int h, int dest, u32 callv, char* wsp, LAS char* scr, int lane) {
  const int fr = lane & 15, fq = lane >> 4;
  const int piece = j * 2 + h;
  const u32 slot = (isx ? 0 : SLOT_W + dest * 256) + s * 16 + piece;
  u32 v[64];
  {
    const rsrc_t ri = mkrsrc(isx ? a.x : a.w, isx ? 4096u * 512u * 2u : 512u * 32768u * 2u);
    const u32 lv = isx ? fr * 1024 + fq * 16 : fq * 8 * 65536 + 16 * (4 * (fr & 3) + (fr >> 2));
    const u32 ls = isx ? (s * 256 + (piece & 1) * 128) * 1024 + (piece >> 1) * 128
                       : (piece >> 1) * 64 * 65536 + (dest * 4096 + s * 256 + (piece & 1) * 128) * 2;
    const u32 s8 = isx ? 65536 : 32 * 65536, s2 = isx ? 16384 : 2 * 65536, s1 = isx ? 64 : 65536;
    u32x4 in[16];
#pragma unroll
    for (int i = 0; i < 16; ++i) in[i] = __builtin_amdgcn_raw_buffer_load_b128(ri, lv, ls + (i >> 3) * s8 + ((i >> 1) & 3) * s2 + (i & 1) * s1, PL_AUX);
    __builtin_amdgcn_sched_barrier(0);
    if (isx) {
#pragma unroll
      for (int i = 0; i < 64; ++i) v[i] = in[i >> 2][i & 3];
    } else {
#pragma unroll
      for (int ks = 0; ks < 2; ++ks)
#pragma unroll
        for (int el = 0; el < 8; ++el)
#pragma unroll
          for (int d = 0; d < 4; ++d)
            v[4 * (el * 2 + ks) + d] = perm(in[ks * 8 + 2 * d + 1][el >> 1], in[ks * 8 + 2 * d][el >> 1], (el & 1) ? 0x07060302u : 0x05040100u);
    }
  }
  GAS u32* const own = (GAS u32*)(a.ws[a.rank] + WS_READY + ((isx ? 0 : 4) + (s >> 2)) * 9 * 128);
  const int c = (s < 8 ? (s & 3) + (isx ? 0 : 4) : s - 8) * 2 + h, m = s < 8 ? j * 2 + (s >> 2) : isx ? 16 + j : 24 + j;
  const u32 doff = WS_DESC + ((a.rank * 16 + c) * NIN + m) * 16;
  if (mode & 5) {
    char* const img = a.ws[mode & 4 ? dest : a.rank] + WS_IMG;
    const u32 so = (isx ? 0 : IMG_BYTES) + (u32)(s * 64 + (mode & 4 ? 7 + j * 7 + ((a.rank - dest) & 7) : j)) * 32768u + h * (isx ? 16384 : 8192);
    const rsrc_t rimg[2] = {mkrsrc(img + so, 8192), mkrsrc(img + so + (isx ? 8192 : 16384), 8192)};
#pragma unroll
    for (int i = 0; i < 16; ++i) bst128(rimg[i >> 3], lane * 16 + (i & 7) * 1024, (u32x4){v[4 * i], v[4 * i + 1], v[4 * i + 2], v[4 * i + 3]});
    if (!(mode & 2)) {
      waitv<0>();
      asm volatile("" : "+v"(callv), "+v"(wsp));
      if (mode & 4) { if (lane == dest) st_sys128(wsp + doff, (u32x4){callv + 1, 0, 0, callv + 1}); }
      else if (lane == 0) __hip_atomic_fetch_add(own, 1u, __ATOMIC_RELAXED, __HIP_MEMORY_SCOPE_AGENT);
      return;
    }
  }
  u32 mx = 0;
#pragma unroll
  for (int i = 0; i < 64; ++i) { v[i] = __builtin_amdgcn_alignbit(v[i], v[i], 31); mx = pk_max(mx, v[i]); }
  const u32 um = wave_max(umax(mx & 0xFFFF, mx >> 16)) | 1;
  const u32 base = um > 2047 ? um - 2047 : 0, bb = base * 0x10001u, upz = ((65536 - base) >> 11) & 31;
  const rsrc_t rb = mkrsrc(a.ws[a.rank] + WS_BLOB + slot * 16384u, 16384u);
  const u32 vo = lane * 16;
  u32 H[32];
#pragma unroll
  for (int r = 0; r < 8; ++r) {
    u32x4 lo;
#pragma unroll
    for (int k = 0; k < 4; ++k) {
      const u32 t0 = pk_sub(v[8 * r + 2 * k], bb), t1 = pk_sub(v[8 * r + 2 * k + 1], bb);
      lo[k] = perm(t1, t0, 0x06040200u); H[4 * r + k] = perm(t1, t0, 0x07050301u);
    }
    bst128(rb, vo + r * 1024, lo);
  }
  u32 E = 0; bool bad = false;
#pragma unroll
  for (int r = 0; r < 8; ++r) {
    u32 U[4];
#pragma unroll
    for (int k = 0; k < 4; ++k) U[k] = (H[4 * r + k] >> 1) & 0x7C7C7C7Cu;
    emit_exc(U, true, (r * 64 + lane) * 16, upz, scr, IN_EXC_CAP, E, bad);
  }
  const bool raw = __builtin_amdgcn_ballot_w64(bad) != 0 || E > IN_EXC_CAP || (a.flags & 1);
  if (raw) {
#pragma unroll
    for (int r = 0; r < 8; ++r) bst128(rb, vo + (8 + r) * 1024, (u32x4){H[4 * r], H[4 * r + 1], H[4 * r + 2], H[4 * r + 3]});
  } else {
    u32 P[12];
#pragma unroll
    for (int g = 0; g < 4; ++g) {
      const u32* h = H + 8 * g; const u32 m7 = 0x07070707u;
      P[3 * g] = (h[0] & m7) | ((h[1] & m7) << 3) | ((h[2] & 0x03030303u) << 6);
      P[3 * g + 1] = (h[3] & m7) | ((h[4] & m7) << 3) | ((h[5] & 0x03030303u) << 6);
      P[3 * g + 2] = (h[6] & m7) | ((h[7] & m7) << 3) | ((h[2] & 0x04040404u) << 4) | ((h[5] & 0x04040404u) << 5);
    }
#pragma unroll
    for (int r = 0; r < 3; ++r) bst128(rb, vo + (8 + r) * 1024, (u32x4){P[4 * r], P[4 * r + 1], P[4 * r + 2], P[4 * r + 3]});
    cbar();
    for (u32 i = vo; i < 2 * E; i += 1024) bst128(rb, 11264 + i, lds_r128(scr + i));
    cbar();
  }
  waitv<0>();
  asm volatile("" : "+v"(callv), "+v"(wsp));
  const u32x4 d = {callv + 1, base | (raw ? 0u : E << 16), raw ? 0x80000000u : 0u, callv + 1};
  if (lane < NR && lane != a.rank && (isx || lane == dest)) st_sys128(wsp + doff, d);
  if ((mode & 1) && lane == 0) __hip_atomic_fetch_add(own, 1u, __ATOMIC_RELAXED, __HIP_MEMORY_SCOPE_AGENT);
}

DEV void pack_out(const Args& a, f32x4 (&acc)[2][2][4][2], u32 blob, u32 call, LAS char* reg, int lane) {
  u32 o[64];
#pragma unroll
  for (int bj = 0; bj < 2; ++bj)
#pragma unroll
    for (int n = 0; n < 2; ++n)
#pragma unroll
      for (int j = 0; j < 4; ++j)
#pragma unroll
        for (int d = 0; d < 4; ++d) {
          const f32x2 y = {__builtin_elementwise_maximum(acc[d >> 1][bj][(d & 1) * 2][n][j], 0.f),
                           __builtin_elementwise_maximum(acc[d >> 1][bj][(d & 1) * 2 + 1][n][j], 0.f)};
          o[((bj * 2 + n) * 4 + j) * 4 + d] = __builtin_bit_cast(u32, __builtin_convertvector(y, bf16x2)) & 0x7FFF7FFFu;
        }
  u32 mx = 0; u32x4 bits;
#pragma unroll
  for (int q = 0; q < 4; ++q) {
    u32 m16[2];
#pragma unroll
    for (int h = 0; h < 2; ++h) {
      u32 t = 0;
#pragma unroll
      for (int i = 0; i < 8; ++i) { const u32 x = o[q * 16 + h * 8 + i]; mx = pk_max(mx, x); t = (t << 2) | pk_min(x, 0x00010001u); }
      m16[h] = ((t & 0xFFFF) << 1) | (t >> 16);
    }
    bits[q] = (m16[0] << 16) | m16[1];
  }
  const u32 vmax = wave_max(umax(mx & 0xFFFF, mx >> 16));
  const u32 off = vmax > 1023 ? vmax - 1023 : 0, offp = off * 0x10001u;
  const u32 cnt = __builtin_popcount(bits.x) + __builtin_popcount(bits.y) + __builtin_popcount(bits.z) + __builtin_popcount(bits.w);
  const u32 dl = (lane & 15) * 4 + (lane >> 4);
  const u32 cd = (u32)__builtin_amdgcn_ds_bpermute((((lane & 3) * 16 + (lane >> 2)) * 4), (int)cnt);
  const u32 inc = wave_scan(cd);
  const u32 n = (u32)__builtin_amdgcn_readlane((int)inc, 63);
  const u32 start = (u32)__builtin_amdgcn_ds_bpermute((int)(dl * 4), (int)(inc - cd));
  const rsrc_t rb = mkrsrc(a.ws[a.rank] + WS_BLOB + (SLOT_O + blob) * 16384u, 16384u);
  bst128(rb, dl * 16, bits);
  {
    u32 pos = 2 * start; const u32 dump = E_EXC + lane * 4;
#pragma unroll
    for (int i = 0; i < 64; ++i) {
      const u32 x = o[i]; const bool z0 = (x & 0xFFFF) != 0, z1 = x > 0xFFFF;
      *(LAS u16*)(reg + (z0 ? pos : dump)) = (u16)x; pos += z0 ? 2 : 0;
      *(LAS u16*)(reg + (z1 ? pos : dump)) = (u16)(x >> 16); pos += z1 ? 2 : 0;
    }
    if (lane < 16) *(LAS u16*)(reg + 2 * (n + lane)) = (u16)off;
  }
  cbar();
  const u32 G = (n + 15) >> 4;
  u32 E = 0; bool bad = false;
  for (u32 g0 = 0; g0 < G; g0 += 64) {
    const u32 g = g0 + lane; const bool act = g < G;
    const u32x4 t0 = lds_r128(reg + 32 * g), t1 = lds_r128(reg + 32 * g + 16);
    u32 t[8] = {t0.x, t0.y, t0.z, t0.w, t1.x, t1.y, t1.z, t1.w}, U[4], code = 0; u32x4 lo;
#pragma unroll
    for (int k = 0; k < 4; ++k) {
      const u32 p0 = pk_sub(t[2 * k], offp), p1 = pk_sub(t[2 * k + 1], offp), h = perm(p1, p0, 0x07050301u);
      lo[k] = perm(p1, p0, 0x06040200u); code |= (h & 0x03030303u) << (2 * k); U[k] = h & 0x7C7C7C7Cu;
    }
    if (act) { bst128(rb, 1024 + 16 * g, lo); bst32(rb, 1024 + 16 * G + 4 * g, code); }
    emit_exc(U, act, g * 16, 99, reg + E_EXC, OUT_EXC_CAP, E, bad);
  }
  cbar();
  const bool raw = __builtin_amdgcn_ballot_w64(bad) != 0 || E > OUT_EXC_CAP || (a.flags & 2);
  if (raw) {
    waitv<0>();
#pragma unroll
    for (int c = 0; c < 16; ++c) bst128(rb, c * 1024 + dl * 16, (u32x4){o[4 * c], o[4 * c + 1], o[4 * c + 2], o[4 * c + 3]});
  } else {
    const u32 eo = 1024 + 16 * G + ((4 * G + 15) & ~15u);
    for (u32 i = lane * 16; i < 2 * E; i += 1024) bst128(rb, eo + i, lds_r128(reg + E_EXC + i));
  }
  cbar();
  u32 u = 0;
  if (lane == 0) u = __hip_atomic_fetch_add((GAS u32*)(a.ws[a.rank] + WS_TICKET), 1u, __ATOMIC_RELAXED, __HIP_MEMORY_SCOPE_AGENT);
  waitv<0>();
  u = rfl(u) & 2047;
  const u32x4 d = {call, n | (off << 16), (raw ? 0x80000000u : E) | (blob << 16), call};
  if (lane < NR && lane != a.rank) st_sys128(a.ws[lane] + WS_DESCO + (a.rank * 2048 + u) * 16, d);
  const u32 tile = blob >> 3, w8 = blob & 7;
  const u32 so = ((tile >> 4) * 256 + (w8 & 3) * 32) * 65536u + (a.rank * 4096 + (tile & 15) * 256 + (w8 >> 2) * 128) * 2u;
  const rsrc_t rout = mkrsrc(a.out + so, OUT_BYTES - so);
  const u32 vo = (lane & 15) * 65536u + (lane >> 4) * 16;
#pragma unroll
  for (int ch = 0; ch < 16; ++ch)
    __builtin_amdgcn_raw_buffer_store_b128((u32x4){o[4 * ch], o[4 * ch + 1], o[4 * ch + 2], o[4 * ch + 3]}, rout,
                                           vo + (ch & 3) * 64 + ((ch >> 3) * 128 + ((ch >> 2) & 1) * 16) * 65536u, 0, OWN_AUX);
}

DEV void wait_ready(const GAS u32* cp, u32 need, u64 t0, u32& bad) {
  while (!bad) {
    const u32 c = __hip_atomic_load(cp, __ATOMIC_RELAXED, __HIP_MEMORY_SCOPE_AGENT);
    if (__builtin_amdgcn_ballot_w64((int)(c - need) < 0) == 0) break;
    if (now() - t0 > DEADLINE) bad = 1;
    __builtin_amdgcn_s_sleep(4);
  }
  cbar();
}
DEV void gemm_role(const Args& a, LAS char* lds, int pos, int wid, u32 call, u64 t0) {
  char* const wsl = a.ws[a.rank];
  const int wr = wid >> 2, wc = wid & 3;
  u32 bad = 0;
  for (int round = 0; round < 2; ++round) {
    int tid = threadIdx.x; asm volatile("" : "+v"(tid));
    int lane = tid & 63;
    const int tile = a.tiles[round * 128 + pos], mt = tile >> 4, nt = tile & 15;
    const GAS u32* cp = (const GAS u32*)(wsl + WS_READY + ((lane & 1) ? 4 + (nt >> 2) : mt >> 2) * 9 * 128);
    if (wid == 0) wait_ready(cp, 64u * call, t0, bad);
    asm volatile("s_waitcnt lgkmcnt(0)" ::: "memory");
    __builtin_amdgcn_s_barrier();
    STAMP_AT(2 + round * 3);
    f32x4 acc[2][2][4][2];
#pragma unroll
    for (int i0 = 0; i0 < 2; ++i0)
#pragma unroll
      for (int i1 = 0; i1 < 2; ++i1)
#pragma unroll
        for (int m = 0; m < 4; ++m)
#pragma unroll
          for (int n = 0; n < 2; ++n) acc[i0][i1][m][n] = (f32x4){0.f, 0.f, 0.f, 0.f};
    bf16x8 At[4][2], B0[2][2], B1[2][2];
    const char* cA = wsl + WS_IMG + IMG_BYTES + (size_t)nt * (2u << 20);
    const char* cB = wsl + WS_IMG + (size_t)mt * (2u << 20);
    const u32 voff = tid * 16, ldsw = wid * 1024;
    const int aoff = wr * 8192 + lane * 16, boff = wc * 4096 + lane * 16;
    constexpr int HTB = 16384, KST = 32768, NT = 64;
#define SA(b, h) (((b) * 2 + (h)) * HTB)
#define SB(b, h) ((4 + (b) * 2 + (h)) * HTB)
#define STAGE(bufoff, gbase) do { _Pragma("unroll") for (int _i = 0; _i < 2; ++_i) \
    __builtin_amdgcn_global_load_lds((const GAS u32*)((gbase) + _i * 8192 + voff), (LAS u32*)(lds + (bufoff) + ldsw + _i * 8192), 16, 0, GL_AUX); } while (0)
#define LDA(dst, b, h) do { _Pragma("unroll") for (int m = 0; m < 4; ++m) _Pragma("unroll") for (int k = 0; k < 2; ++k) \
    dst[m][k] = *(const LAS bf16x8*)(lds + SA(b, h) + aoff + m * 2048 + k * 1024); } while (0)
#define LDB(dst, b, h) do { _Pragma("unroll") for (int n = 0; n < 2; ++n) _Pragma("unroll") for (int k = 0; k < 2; ++k) \
    dst[n][k] = *(const LAS bf16x8*)(lds + SB(b, h) + boff + n * 2048 + k * 1024); } while (0)
#define MMA(ai, bj, A_, B_) do { __builtin_amdgcn_s_setprio(1); _Pragma("unroll") for (int m = 0; m < 4; ++m) _Pragma("unroll") for (int n = 0; n < 2; ++n) \
    _Pragma("unroll") for (int k = 0; k < 2; ++k) acc[ai][bj][m][n] = __builtin_amdgcn_mfma_f32_16x16x32_bf16(A_[m][k], B_[n][k], acc[ai][bj][m][n], 0, 0, 0); \
    __builtin_amdgcn_s_setprio(0); } while (0)
#define WAIT_V(n) asm volatile("s_waitcnt vmcnt(" #n ")" ::: "memory")
#define WAIT_L(n) asm volatile("s_waitcnt lgkmcnt(" #n ")" ::: "memory")
#define BAR __builtin_amdgcn_s_barrier()
#define SCHED __builtin_amdgcn_sched_barrier(0)
    STAGE(SB(0, 0), cB); STAGE(SA(0, 0), cA); STAGE(SB(0, 1), cB + HTB); STAGE(SA(0, 1), cA + HTB);
    if (wr == 1) BAR;
    WAIT_V(4); BAR;
    STAGE(SB(1, 0), cB + KST); STAGE(SA(1, 0), cA + KST); STAGE(SB(1, 1), cB + HTB + KST);
    WAIT_V(6); BAR;
    int tn = 5;
    for (int t = 0; t < NT; t += 2) {
      if (wid == 0 && tn < 61) {
        if (t >= tn) {
          const u32 c = lds_poll(LDS_BYTES - 32 + (lane & 1) * 4);
          STAMP_AT(16 + round * 16 + (tn / 7) * 2);
          if (__builtin_amdgcn_ballot_w64((int)(c - 56u * call) < 0) != 0) wait_ready(cp, 56u * call, t0, bad);
          STAMP_AT(17 + round * 16 + (tn / 7) * 2);
          tn += 7;
        } else if (t + 2 >= tn) {
          cp += 32;
          if (lane < 2) __builtin_amdgcn_global_load_lds(cp, (LAS u32*)(lds + LDS_BYTES - 32), 4, 0, 17);
        }
      }
      const bool last = (t == NT - 2);
      const char* a1 = cA + (size_t)(t + 1) * KST;
      const char* a2 = last ? cA : cA + (size_t)(t + 2) * KST; const char* b2 = last ? cB : cB + (size_t)(t + 2) * KST;
      const char* a3 = a2 + KST; const char* b3 = b2 + KST;
      LDB(B0, 0, 0); SCHED; LDA(At, 0, 0); STAGE(SA(1, 1), a1 + HTB);
      WAIT_L(8); BAR; WAIT_L(0); MMA(0, 0, At, B0); BAR; SCHED;
      LDB(B1, 0, 1); STAGE(SB(0, 0), b2);
      BAR; WAIT_L(0); MMA(0, 1, At, B1); BAR;
      LDA(At, 0, 1); STAGE(SA(0, 0), a2);
      BAR; WAIT_L(0); MMA(1, 0, At, B0); BAR; SCHED;
      STAGE(SB(0, 1), b2 + HTB);
      WAIT_V(6); BAR; MMA(1, 1, At, B1); BAR;
      LDB(B0, 1, 0); SCHED; LDA(At, 1, 0); STAGE(SA(0, 1), a2 + HTB);
      WAIT_L(8); BAR; WAIT_L(0); MMA(0, 0, At, B0); BAR; SCHED;
      LDB(B1, 1, 1); STAGE(SB(1, 0), b3);
      BAR; WAIT_L(0); MMA(0, 1, At, B1); BAR;
      LDA(At, 1, 1); STAGE(SA(1, 0), a3);
      BAR; WAIT_L(0); MMA(1, 0, At, B0); BAR; SCHED;
      STAGE(SB(1, 1), b3 + HTB);
      WAIT_V(6); BAR; MMA(1, 1, At, B1); BAR;
    }
    if (wid == 0) lds_post(LDS_BYTES - 16, bad);
    asm volatile("s_waitcnt vmcnt(0) lgkmcnt(0)" ::: "memory");
    if (wr == 0) BAR;
    BAR;
#undef SA
#undef SB
#undef STAGE
#undef LDA
#undef LDB
#undef MMA
    int tz = threadIdx.x; asm volatile("" : "+v"(tz));
    lane = tz & 63;
    STAMP_AT(3 + round * 3);
    if (rfl(lds_poll(LDS_BYTES - 16))) {
#pragma unroll
      for (int i = 0; i < 32; ++i) acc[i >> 4][(i >> 3) & 1][(i >> 1) & 3][i & 1] = (f32x4){__builtin_nanf(""), __builtin_nanf(""), __builtin_nanf(""), __builtin_nanf("")};
    }
    pack_out(a, acc, tile * 8 + wid, call, lds + wid * EREG, lane);
    STAMP_AT(4 + round * 3);
  }
}

DEV void loader_role(const Args& a, LAS char* lds, int peer, int lane, u64 t0) {
  const char* src = a.ws[peer] + WS_BLOB + lane * 16;
  u32 served = 0; int pend = -1; u32 pend_tag = 0;
  int done = 0;
  for (int left = NDEC; left > 0;) {
    const u32 r = lds_poll(L_REQ + (lane & 7) * 4);
    u32 key = ((lane & 7) < NDEC && (r & 255) != served) ? ((255 - (r & 255)) << 3 | (7 - (lane & 7))) + 1 : 0;
    key = umax(key, dpp0<0x111, 0xf>(key)); key = umax(key, dpp0<0x112, 0xf>(key)); key = umax(key, dpp0<0x114, 0xf>(key));
    key = (u32)__builtin_amdgcn_readlane((int)key, 7);
    if (key == 0) {
      if (pend >= 0) { waitv<0>(); lds_post(L_FULL + pend * 4, pend_tag); pend = -1; }
      else { if (now() - t0 > DEADLINE + 20000000ull) break; __builtin_amdgcn_s_sleep(2); }
      continue;
    }
    const int j = 7 - ((key - 1) & 7);
    const u32 rq = (u32)__builtin_amdgcn_readlane((int)r, j);
    if ((lane & 7) == j) served = rq & 255;
    if (rq >> 31) { --left; continue; }
    ++done;
    const u32 slot = umin(rq >> 18, NSLOT - 1), bytes = (((rq >> 8) & 1023) + 1) * 16;
    const char* s = src + (size_t)slot * 16384;
    LAS char* dst = lds + j * DREG + BOFF;
    int row = 0;
    for (u32 o = 0; o < bytes; o += 1024, ++row) {
      if (o + lane * 16 < bytes) __builtin_amdgcn_global_load_lds((const GAS u32*)(s + o), (LAS u32*)(dst + o), 16, 0, LD_AUX);
      waitv<WIN - 1>();
      if (row == WIN - 2 && pend >= 0) { lds_post(L_FULL + pend * 4, pend_tag); pend = -1; }
    }
    if (pend >= 0) { waitv<0>(); lds_post(L_FULL + pend * 4, pend_tag); }
    pend = j; pend_tag = rq & 255;
  }
  waitv<0>();
  if (pend >= 0) lds_post(L_FULL + pend * 4, pend_tag);
}

DEV void decoder_role(const Args& a, LAS char* lds, int peer, int c, int j, int lane, u32 call, u64 t0) {
  char* const wsl = a.ws[a.rank];
  LAS char* A = lds + j * DREG;
  LAS char* B = A + BOFF;
  const int q = (peer - a.rank) & 7;
  bool fail = false, posted = false;
  u32 tag = 1;
  const char* dsc = wsl + WS_DESC + (size_t)((peer * 16 + c) * NIN + j) * 16;
  u32 da = 0, db = 0;
  auto post = [&](int m, u32 da, u32 db, u32 tag) {
    u32 slot, bytes;
    if (m < NIN) {
      const int kj = m < 16 ? m >> 1 : m & 7, t = m < 16 ? c >> 3 : (m >> 3) & 1, s = m < 16 ? (m & 1) * 4 + ((c >> 1) & 3) : 8 + (c >> 1);
      slot = (t ? SLOT_W + a.rank * 256 : 0) + s * 16 + kj * 2 + (c & 1);
      bytes = 11264 + ((2 * umin(da >> 16, IN_EXC_CAP) + 15) & ~15u);
    } else {
      const u32 G = (umin(da & 0xFFFF, 8192) + 15) >> 4;
      slot = SLOT_O + ((db >> 16) & 2047);
      bytes = 1024 + 16 * G + ((4 * G + 15) & ~15u) + ((2 * umin(db & 0xFFFF, OUT_EXC_CAP) + 15) & ~15u);
    }
    if (db >> 31) bytes = 16384;
    asm volatile("s_waitcnt lgkmcnt(0)" ::: "memory");
    lds_post(L_REQ + j * 4, (slot << 18) | ((bytes / 16 - 1) << 8) | tag);
  };
  for (int m = j;; m += NDEC) {
    if (!posted) {
      u32x4 d;
      for (;;) {
        d = ld_sys128(dsc);
        if (rfl(d.x) == call && rfl(d.w) == call) break;
        if (now() - t0 > DEADLINE) { fail = true; break; }
        __builtin_amdgcn_s_sleep(4);
      }
      if (fail) break;
      da = rfl(d.y); db = rfl(d.z);
      if (m) post(m, da, db, tag);
    }
    if (tag <= 12) STAMP_AT(12 + tag * 4);
    bool more = true, nok = false; u32 nda = 0, ndb = 0;
    dsc += NDEC * 16;
    if (m + NDEC >= NIN) {
      u32 u = 0;
      if (lane == 0) u = __hip_atomic_fetch_add((GAS u32*)(wsl + WS_QUEUE + peer * 128), 1u, __ATOMIC_RELAXED, __HIP_MEMORY_SCOPE_AGENT);
      u = rfl(u) - (call - 1) * QSTEP;
      more = u < 2048;
      dsc = wsl + WS_DESCO + (size_t)(peer * 2048 + (u & 2047)) * 16;
    }
    if (more) {
      const u32x4 d = ld_sys128(dsc);
      nok = rfl(d.x) == call && rfl(d.w) == call; nda = rfl(d.y); ndb = rfl(d.z);
    }
    const u32 ntag = tag == 255 ? 1 : tag + 1;
    const bool raw = db >> 31;
    const int kj = m < 16 ? m >> 1 : m & 7, t = m < 16 ? c >> 3 : (m >> 3) & 1, s = m < 16 ? (m & 1) * 4 + ((c >> 1) & 3) : 8 + (c >> 1);
    const u32 E = m < NIN ? umin(da >> 16, IN_EXC_CAP) : umin(db & 0xFFFF, OUT_EXC_CAP), G = (umin(da & 0xFFFF, 8192) + 15) >> 4;
    while (m) {
      if (rfl(lds_poll(L_FULL + j * 4)) == tag) break;
      if (now() - t0 > DEADLINE) { fail = true; break; }
      __builtin_amdgcn_s_sleep(2);
    }
    if (fail) break;
    posted = false;
    if (tag <= 12) STAMP_AT(13 + tag * 4);
    if (m == 0) {
      if (lane == 0) __hip_atomic_fetch_add((GAS u32*)(wsl + WS_READY + (t * 4 * 9 + 1) * 128), 1u, __ATOMIC_RELAXED, __HIP_MEMORY_SCOPE_AGENT);
    } else
    if (m < NIN) {
      const u32 base = da & 0xFFFF, bb = base * 0x10001u, upz = ((65536 - base) >> 11) & 31;
      u32 H[32];
      if (raw) {
#pragma unroll
        for (int r = 0; r < 8; ++r) { const u32x4 t = lds_r128(B + (8 + r) * 1024 + lane * 16); H[4 * r] = t.x; H[4 * r + 1] = t.y; H[4 * r + 2] = t.z; H[4 * r + 3] = t.w; }
      } else {
        for (u32 i = lane; i < E; i += 64) {
          const u32 en = *(LAS u16*)(B + 11264 + 2 * i), cc = en & 7;
          *(LAS unsigned char*)(A + (en >> 3)) = (unsigned char)((cc ? 32 - cc : upz) << 3);
        }
        cbar();
        u32 P[12];
#pragma unroll
        for (int r = 0; r < 3; ++r) { const u32x4 t = lds_r128(B + (8 + r) * 1024 + lane * 16); P[4 * r] = t.x; P[4 * r + 1] = t.y; P[4 * r + 2] = t.z; P[4 * r + 3] = t.w; }
#pragma unroll
        for (int r = 0; r < 8; ++r) { const u32x4 t = lds_r128(A + r * 1024 + lane * 16); H[4 * r] = t.x; H[4 * r + 1] = t.y; H[4 * r + 2] = t.z; H[4 * r + 3] = t.w; }
        cbar();
        for (u32 i = lane; i < E; i += 64) *(LAS unsigned char*)(A + (*(LAS u16*)(B + 11264 + 2 * i) >> 3)) = 0;
        cbar();
#pragma unroll
        for (int g = 0; g < 4; ++g) {
          u32* h = H + 8 * g; const u32 p0 = P[3 * g], p1 = P[3 * g + 1], p2 = P[3 * g + 2], m7 = 0x07070707u;
          h[0] |= p0 & m7; h[1] |= (p0 >> 3) & m7; h[2] |= ((p0 >> 6) & 0x03030303u) | ((p2 >> 4) & 0x04040404u);
          h[3] |= p1 & m7; h[4] |= (p1 >> 3) & m7; h[5] |= ((p1 >> 6) & 0x03030303u) | ((p2 >> 5) & 0x04040404u);
          h[6] |= p2 & m7; h[7] |= (p2 >> 3) & m7;
        }
      }
      const u32 so = t * IMG_BYTES + (u32)(s * 64 + 7 + kj * 7 + q) * 32768u + (c & 1) * (t ? 8192 : 16384);
      const rsrc_t rimg[2] = {mkrsrc(wsl + WS_IMG + so, 8192), mkrsrc(wsl + WS_IMG + so + (t ? 16384 : 8192), 8192)};
#pragma unroll
      for (int r = 0; r < 8; ++r) {
        const u32x4 lo = lds_r128(B + r * 1024 + lane * 16);
        u32 v[8];
#pragma unroll
        for (int k = 0; k < 4; ++k) {
          const u32 u0 = pk_add(perm(H[4 * r + k], lo[k], 0x05010400u), bb), u1 = pk_add(perm(H[4 * r + k], lo[k], 0x07030602u), bb);
          v[2 * k] = __builtin_amdgcn_alignbit(u0, u0, 1); v[2 * k + 1] = __builtin_amdgcn_alignbit(u1, u1, 1);
        }
        if (!NOIMG || call < 3) {
        ist128(rimg[r >> 2], lane * 16 + ((2 * r) & 7) * 1024, (u32x4){v[0], v[1], v[2], v[3]});
        ist128(rimg[r >> 2], lane * 16 + ((2 * r + 1) & 7) * 1024, (u32x4){v[4], v[5], v[6], v[7]});
        }
      }
      if (nok) { post(m + NDEC, nda, ndb, ntag); posted = true; }
      if (tag <= 12) STAMP_AT(14 + tag * 4);
      waitv<0>();
      if (tag <= 12) STAMP_AT(15 + tag * 4);
      if (lane == 0) __hip_atomic_fetch_add((GAS u32*)(wsl + WS_READY + ((t * 4 + (s >> 2)) * 9 + 1 + kj) * 128), 1u, __ATOMIC_RELAXED, __HIP_MEMORY_SCOPE_AGENT);
    } else {
      const u32 blob = (db >> 16) & 2047, tile = blob >> 3, w8 = blob & 7;
      const u32 so = ((tile >> 4) * 256 + (w8 & 3) * 32) * 65536u + (peer * 4096 + (tile & 15) * 256 + (w8 >> 2) * 128) * 2u;
      const u32 vo = (lane >> 2) * 65536u + (lane & 3) * 16;
      const rsrc_t rout = mkrsrc(a.out + so, OUT_BYTES - so);
      if (raw) {
#pragma unroll
        for (int ch = 0; ch < 16; ++ch)
          __builtin_amdgcn_raw_buffer_store_b128(lds_r128(B + ch * 1024 + lane * 16), rout, vo + (ch & 3) * 64 + ((ch >> 3) * 128 + ((ch >> 2) & 1) * 16) * 65536u, 0, OUT_AUX);
      } else {
        const u32 offp = ((da >> 16) & 0x7FFF) * 0x10001u;
        const u32x4 bits = lds_r128(B + lane * 16);
        cbar();
        for (u32 g = lane; g < G; g += 64) {
          const u32x4 lo = lds_r128(B + 1024 + 16 * g); const u32 code = *(LAS u32*)(B + 1024 + 16 * G + 4 * g);
          u32 v[8];
#pragma unroll
          for (int k = 0; k < 4; ++k) {
            const u32 h = (code >> (2 * k)) & 0x03030303u;
            v[2 * k] = pk_add(perm(h, lo[k], 0x05010400u), offp); v[2 * k + 1] = pk_add(perm(h, lo[k], 0x07030602u), offp);
          }
          lds_w128(A + 32 * g, (u32x4){v[0], v[1], v[2], v[3]}); lds_w128(A + 32 * g + 16, (u32x4){v[4], v[5], v[6], v[7]});
          cbar();
        }
        LAS char* ex = B + 1024 + 16 * G + ((4 * G + 15) & ~15u);
        for (u32 i = lane; i < E; i += 64) {
          const u32 en = *(LAS u16*)(ex + 2 * i); LAS u16* p = (LAS u16*)(A + 2 * (en >> 3));
          *p = (u16)(*p - ((en & 7) << 10));
        }
        cbar();
        if (nok && 32 * G + 16 <= BOFF) { post(m + NDEC, nda, ndb, ntag); posted = true; }
        const u32 cnt = __builtin_popcount(bits.x) + __builtin_popcount(bits.y) + __builtin_popcount(bits.z) + __builtin_popcount(bits.w);
        u32 pos = wave_scan(cnt) - cnt;
#pragma unroll
        for (int ch = 0; ch < 16; ++ch) {
          u32x4 ov;
#pragma unroll
          for (int h = 0; h < 2; ++h) {
            const u32 nib = (bits[ch >> 2] >> (28 - 4 * ((ch & 3) * 2 + h))) & 15;
            const u32x2 sel = *(LAS u32x2*)(lds + L_LUT + nib * 8);
            const u32 ad = 2 * pos; LAS u32* q = (LAS u32*)(A + (ad & ~3u));
            const u32 d0 = q[0], d1 = q[1], d2 = q[2];
            const u32 e0 = __builtin_amdgcn_alignbyte(d1, d0, ad), e1 = __builtin_amdgcn_alignbyte(d2, d1, ad);
            ov[2 * h] = perm(e1, e0, sel.x); ov[2 * h + 1] = perm(e1, e0, sel.y);
            pos += __builtin_popcount(nib);
          }
          __builtin_amdgcn_raw_buffer_store_b128(ov, rout, vo + (ch & 3) * 64 + ((ch >> 3) * 128 + ((ch >> 2) & 1) * 16) * 65536u, 0, OUT_AUX);
        }
      }
      cbar();
      if (nok && !posted) { post(m + NDEC, nda, ndb, ntag); posted = true; }
      if (tag <= 12) STAMP_AT(14 + tag * 4);
    }
    tag = ntag; da = nda; db = ndb;
    if (!more) break;
  }
  if (fail) tag = tag == 255 ? 1 : tag + 1;
  lds_post(L_REQ + j * 4, 0x80000000u | tag);
  if (fail) {
    waitv<0>();
    __builtin_amdgcn_raw_buffer_store_b32(0x7FC07FC0u, mkrsrc(a.out, OUT_BYTES), lane * 4 + (blockIdx.x * 8 + j) * 65536u, 0, 0);
    if (lane == 0) __hip_atomic_store((GAS u32*)(wsl + WS_TMO), call, __ATOMIC_RELAXED, __HIP_MEMORY_SCOPE_AGENT);
  }
}

__global__ void __launch_bounds__(512, 2) fused_gemm_ar(const Args a) {
  extern __shared__ __attribute__((aligned(16))) char smem[];
  LAS char* lds = (LAS char*)smem;
  const u64 t0 = now();
  const int wid = __builtin_amdgcn_readfirstlane(threadIdx.x >> 6), lane = threadIdx.x & 63;
  const int xcd = vbid() & 7, li = vbid() >> 3;
  char* const wsl = a.ws[a.rank];
  GAS u32* callp = (GAS u32*)(wsl + WS_CALL) + blockIdx.x * 8 + wid;
  const u32 callv = __hip_atomic_load(callp, __ATOMIC_RELAXED, __HIP_MEMORY_SCOPE_AGENT);
  char* const wsp = a.ws[lane & 7];
  STAMP_AT(0);
  const bool is_gemm = li < 16;
  const int pos = xcd * 16 + (li & 15);
  if (!is_gemm) {
    if (wid == 0) {
      if (lane < 32) *(LAS u32*)(lds + L_CTRL + lane * 4) = 0;
      if (lane < 16) {
        u32 k = 0, sel[2] = {0, 0};
#pragma unroll
        for (int i = 0; i < 4; ++i) {
          const bool on = (lane >> (3 - i)) & 1;
          sel[i >> 1] |= (on ? (2 * k) | ((2 * k + 1) << 8) : 0x0C0Cu) << (16 * (i & 1));
          k += on;
        }
        *(LAS u32x2*)(lds + L_LUT + lane * 8) = (u32x2){sel[0], sel[1]};
      }
    } else if (wid <= NDEC) {
#pragma unroll
      for (int r = 0; r < 8; ++r) lds_w128(lds + (wid - 1) * DREG + r * 1024 + lane * 16, (u32x4){0, 0, 0, 0});
    }
    asm volatile("s_waitcnt lgkmcnt(0)" ::: "memory");
    __builtin_amdgcn_s_barrier();
  }
  const int q = li - 16;
  const bool is_help = !is_gemm && (q & 7) == 0;
  {
    int e = -1, n = 1;
    const bool early = is_gemm && (xcd & 1);
    if (early) { if (wid < 4) e = 2176 + wid * 64 + (xcd >> 1) * 16 + li; }
    else if (is_gemm) { e = 512 + wid * 64 + (xcd >> 1) * 16 + li; n = 3; }
    else if (is_help) { e = 1024 + (xcd * 2 + (q >> 3)) * 8 + wid; n = 2; }
    else if (wid >= 4) e = 64 + (wid - 4) * 112 + (xcd * 2 + (q >> 3)) * 7 + (q & 7) - 1;
    else if (wid == 1) e = xcd * 2 + (q >> 3);
    LAS char* scr = is_gemm || is_help ? lds + wid * EREG : wid == 7 ? lds + L_SCR7 : lds + (wid - 1) * DREG + BOFF;
    if (GDELAY && n > 1) __builtin_amdgcn_s_sleep(GDELAY);
    if (e >= 0)
      for (int i = 0; i < n && (e < 2176 || n == 1); ++i, e += 640) {
        bool isx; int j, s, dq, mode;
        if (e < 64) { j = 0; isx = e < 8; s = (e >> 1) & 3; dq = -q; mode = 4; }
        else if (e < 1024) { j = e >> 7; dq = (e >> 3) & 7; isx = dq == 0; s = ((e >> 6) & 1) * 4 + ((e >> 1) & 3); mode = 2; }
        else if (e < 1152) { j = (e >> 4) & 7; isx = true; s = 8 + ((e >> 1) & 7); dq = 0; mode = 3; }
        else if (e < 2176) { j = (e - 1152) >> 7; isx = false; s = 8 + ((e >> 1) & 7); dq = (e >> 4) & 7; mode = dq ? 2 : 1; }
        else { j = (e >> 4) & 7; isx = e < 2304; s = (e >> 1) & 7; dq = 0; mode = 1; }
        prep_piece(a, mode, isx, s, j, e & 1, (a.rank + dq) & 7, callv, wsp, scr, lane);
        if (is_help) STAMP_AT(16 + i);
      }
  }
  const u32 call = rfl(callv) + 1;
  if (lane == 0) __hip_atomic_store(callp, call, __ATOMIC_RELAXED, __HIP_MEMORY_SCOPE_AGENT);
  STAMP_AT(1);
  if (is_gemm) {
    gemm_role(a, lds, pos, wid, call, t0);
  } else if (!is_help) {
    const int peer = (a.rank + (q & 7)) & 7, c = xcd * 2 + (q >> 3);
    if (wid == 0) loader_role(a, lds, peer, lane, t0);
    else if (wid <= NDEC) decoder_role(a, lds, peer, c, wid - 1, lane, call, t0);
  }
  STAMP_AT(8);
}

#define HIP_CHECK(expr) do { hipError_t _e = (expr); if (_e != hipSuccess) { \
    fprintf(stderr, "HIP error %s at %s:%d\n", hipGetErrorString(_e), __FILE__, __LINE__); \
    return; } } while (0)

#define DBG_FLAGS 0

extern "C" void kernel_launch(int rank, int nranks, ncclComm_t comm, hipStream_t stream,
                              void* const* d_in, const int* in_sizes, int n_in,
                              void* d_out, int out_size, const int* in_dtypes, int out_dtype,
                              void* const* d_out_all, void* const* d_ws_all, size_t ws_bytes) {
  if (nranks != NR || (unsigned)rank >= (unsigned)NR || n_in < 2 || ws_bytes < WS_USED || (size_t)out_size * 2 != OUT_BYTES ||
      in_sizes[0] != 4096 * 512 || in_sizes[1] != 512 * 32768) {
    fprintf(stderr, "unexpected problem shape\n");
    return;
  }
  static bool attr_set[NR] = {};
  if (!attr_set[rank]) {
    HIP_CHECK(hipFuncSetAttribute((const void*)fused_gemm_ar, hipFuncAttributeMaxDynamicSharedMemorySize, LDS_BYTES));
    attr_set[rank] = true;
  }
  Args a;
  a.x = (const char*)d_in[0]; a.w = (const char*)d_in[1]; a.out = (char*)d_out;
  for (int p = 0; p < NR; ++p) a.ws[p] = (char*)d_ws_all[p];
  a.rank = rank; a.flags = DBG_FLAGS;
  static const int blk[8][2] = {{2, 0}, {0, 0}, {2, 1}, {1, 0}, {3, 0}, {0, 1}, {3, 1}, {1, 1}};
  for (int k = 0; k < 256; ++k) {
    const int b = (k >> 4) & 7, i = k & 15;
    a.tiles[k] = (unsigned char)((blk[b][0] * 4 + (i >> 2)) * 16 + (blk[b][1] + (k >> 7) * 2) * 4 + (i & 3));
  }
  hipLaunchKernelGGL(fused_gemm_ar, dim3(256), dim3(512), LDS_BYTES, stream, a);
  HIP_CHECK(hipGetLastError());
}
```

```cpp
#include <hip/hip_runtime.h>
#include <rccl/rccl.h>
#include <cstdio>

typedef unsigned u32;
typedef unsigned long long u64;
typedef u32 u32x4 __attribute__((ext_vector_type(4)));
typedef u32 u32x2 __attribute__((ext_vector_type(2)));
typedef u32 u32u __attribute__((aligned(1)));
typedef u32x2 u32x2u __attribute__((aligned(2)));
typedef unsigned short u16;
typedef u16 u16x2 __attribute__((ext_vector_type(2)));
typedef short bf16x8 __attribute__((ext_vector_type(8)));
typedef float f32x4 __attribute__((ext_vector_type(4)));
typedef float f32x2 __attribute__((ext_vector_type(2)));
typedef __bf16 bf16x2 __attribute__((ext_vector_type(2)));
#define LAS __attribute__((address_space(3)))
#define GAS __attribute__((address_space(1)))
#define DEV __device__ __forceinline__

constexpr int NR = 8;
constexpr u32 WS_CALL = 0;
constexpr u32 WS_TMO = 8192;
constexpr u32 WS_TICKET = 8192 + 128;
constexpr u32 WS_QUEUE = 8192 + 256;
constexpr u32 WS_READY = 16384;
constexpr int NIN = 32;
constexpr u32 WS_DESC = 512u << 10;
constexpr u32 WS_DESCO = WS_DESC + 65536;
constexpr u32 WS_IMG = 1u << 20;
constexpr u32 IMG_BYTES = 32u << 20;
constexpr u32 WS_BLOB = 65u << 20;
constexpr u32 SLOT_W = 256, SLOT_O = 2304, NSLOT = 4352;
constexpr u32 WS_USED = WS_BLOB + NSLOT * 16384u;
constexpr u32 WS_STAMP = WS_USED;
constexpr u32 OUT_BYTES = 4096u * 32768u * 2u;
constexpr u64 DEADLINE = 100000000ull;
constexpr int LDS_BYTES = 163840;
constexpr int NDEC = 6;
constexpr int DREG = 26624, BOFF = 10240;
constexpr int L_CTRL = NDEC * DREG;
constexpr int L_REQ = L_CTRL, L_FULL = L_CTRL + 64, L_LUT = L_CTRL + 128, L_LUT2 = L_CTRL + 256, L_SCR7 = L_CTRL + 320;
constexpr int EREG = 20480;
constexpr int E_SCR = 16448;
constexpr int L_DSC = LDS_BYTES - 256;
constexpr u32 IN_UCAP = 28000, OUT_UCAP = 13600;
constexpr u32 QSTEP = 2048 + 16 * NDEC;
#define WIN 5
#define GDELAY 0
#define OWN_AUX 0
#define OUT_AUX 0
#define IMG_AUX 17
#define GL_AUX 17
#define PL_AUX 2
#define SEG 0
DEV int vbid() {
  const int x = blockIdx.x & 7, l = blockIdx.x >> 3;
  if (SEG == 0) return blockIdx.x;
  const int comm = SEG == 1 ? (x & 1) ^ 1 : SEG == 2 ? x & 1 : SEG == 3 ? x >> 2 : (x >> 1) & 1;
  const int g = SEG <= 2 ? x >> 1 : SEG == 3 ? x & 3 : (x & 1) | ((x >> 2) << 1);
  return comm * 128 + (l & 15) * 8 + g * 2 + (l >> 4);
}
#define LD_AUX 17
#define NOIMG 0

struct Args {
  const char* x; const char* w; char* out; char* ws[NR]; int rank; int flags; unsigned char tiles[256];
};

template <int N> DEV void waitv() { asm volatile("s_waitcnt vmcnt(%0)" ::"n"(N) : "memory"); }
DEV void cbar() { asm volatile("" ::: "memory"); }
DEV u64 now() { return __builtin_amdgcn_s_memrealtime(); }
DEV u32 perm(u32 hi, u32 lo, u32 sel) { return __builtin_amdgcn_perm(hi, lo, sel); }
DEV u32 pk_sub(u32 a, u32 b) { return __builtin_bit_cast(u32, __builtin_bit_cast(u16x2, a) - __builtin_bit_cast(u16x2, b)); }
DEV u32 pk_add(u32 a, u32 b) { return __builtin_bit_cast(u32, __builtin_bit_cast(u16x2, a) + __builtin_bit_cast(u16x2, b)); }
DEV u32 pk_shr1(u32 a) { return __builtin_bit_cast(u32, __builtin_bit_cast(u16x2, a) >> (u16x2){1, 1}); }
DEV u32 pk_max(u32 a, u32 b) { return __builtin_bit_cast(u32, __builtin_elementwise_max(__builtin_bit_cast(u16x2, a), __builtin_bit_cast(u16x2, b))); }
DEV u32 pk_min(u32 a, u32 b) { return __builtin_bit_cast(u32, __builtin_elementwise_min(__builtin_bit_cast(u16x2, a), __builtin_bit_cast(u16x2, b))); }
template <int C, int RM> DEV u32 dpp0(u32 x) { return (u32)__builtin_amdgcn_update_dpp(0, (int)x, C, RM, 0xf, false); }
DEV u32 umax(u32 a, u32 b) { return a > b ? a : b; }
DEV u32 umin(u32 a, u32 b) { return a < b ? a : b; }
DEV u32 wave_max(u32 x) {
  x = umax(x, dpp0<0x111, 0xf>(x)); x = umax(x, dpp0<0x112, 0xf>(x)); x = umax(x, dpp0<0x114, 0xf>(x)); x = umax(x, dpp0<0x118, 0xf>(x));
  x = umax(x, dpp0<0x142, 0xa>(x)); x = umax(x, dpp0<0x143, 0xc>(x));
  return (u32)__builtin_amdgcn_readlane((int)x, 63);
}
DEV u32 wave_scan(u32 x) {
  x += dpp0<0x111, 0xf>(x); x += dpp0<0x112, 0xf>(x); x += dpp0<0x114, 0xf>(x); x += dpp0<0x118, 0xf>(x);
  x += dpp0<0x142, 0xa>(x); x += dpp0<0x143, 0xc>(x);
  return x;
}
DEV u32 rfl(u32 x) { return (u32)__builtin_amdgcn_readfirstlane((int)x); }
DEV u32 mbcnt(u64 m) { return __builtin_amdgcn_mbcnt_hi((u32)(m >> 32), __builtin_amdgcn_mbcnt_lo((u32)m, 0)); }
DEV u32x4 lds_r128(LAS char* p) { return *(LAS u32x4*)p; }
DEV void lds_w128(LAS char* p, u32x4 v) { *(LAS u32x4*)p = v; }
DEV u32 lds_poll(u32 addr) { u32 v; asm volatile("ds_read_b32 %0, %1\n\ts_waitcnt lgkmcnt(0)" : "=v"(v) : "v"(addr) : "memory"); return v; }
DEV void lds_post(u32 addr, u32 v) { asm volatile("ds_write_b32 %0, %1" ::"v"(addr), "v"(v) : "memory"); }
DEV void st_sys128(void* p, u32x4 v) { asm volatile("global_store_dwordx4 %0, %1, off sc0 sc1\n\ts_nop 1" ::"v"(p), "v"(v) : "memory"); }
DEV u32x4 ld_sys128(const void* p) { u32x4 v; asm volatile("global_load_dwordx4 %0, %1, off sc0 sc1\n\ts_waitcnt vmcnt(0)" : "=&v"(v) : "v"(p) : "memory"); return v; }
typedef __amdgpu_buffer_rsrc_t rsrc_t;
DEV rsrc_t mkrsrc(const void* p, u32 bytes) { return __builtin_amdgcn_make_buffer_rsrc((void*)p, 0, (int)bytes, 0x00020000); }
DEV void bst128(rsrc_t r, u32 voff, u32x4 v) { __builtin_amdgcn_raw_buffer_store_b128(v, r, (int)voff, 0, 17); }
DEV void ist128(rsrc_t r, u32 voff, u32x4 v) { __builtin_amdgcn_raw_buffer_store_b128(v, r, (int)voff, 0, IMG_AUX); }
DEV void bst32(rsrc_t r, u32 voff, u32 v) { __builtin_amdgcn_raw_buffer_store_b32(v, r, (int)voff, 0, 17); }

#define STAMP_AT(k) do { } while (0)

DEV u32 pick_c(float sq, float n) {
  sq += __builtin_bit_cast(float, dpp0<0x111, 0xf>(__builtin_bit_cast(u32, sq))); sq += __builtin_bit_cast(float, dpp0<0x112, 0xf>(__builtin_bit_cast(u32, sq)));
  sq += __builtin_bit_cast(float, dpp0<0x114, 0xf>(__builtin_bit_cast(u32, sq))); sq += __builtin_bit_cast(float, dpp0<0x118, 0xf>(__builtin_bit_cast(u32, sq)));
  sq += __builtin_bit_cast(float, dpp0<0x142, 0xa>(__builtin_bit_cast(u32, sq))); sq += __builtin_bit_cast(float, dpp0<0x143, 0xc>(__builtin_bit_cast(u32, sq)));
  const float t = 0.295f * __builtin_amdgcn_sqrtf(sq * __builtin_amdgcn_rcpf(n));
  return ((((u32)__builtin_amdgcn_readlane(__builtin_bit_cast(int, t), 63) >> 16) * 2 + 1023) & 0xFFFF);
}
DEV float sq2(u32 v, float s) { return __builtin_amdgcn_fdot2_f32_bf16(__builtin_bit_cast(bf16x2, v), __builtin_bit_cast(bf16x2, v), s, false); }
DEV void split4(u32 n0, u32 n1, u32& lo, u32& G, u32& idx, u32& len) {
  lo = perm(n1, n0, 0x06040200u); G = perm(n1, n0, 0x07050301u);
  idx = perm(pk_min(n1, 0x07FF07FFu), pk_min(n0, 0x07FF07FFu), 0x07050301u);
  len = G + perm(0x01000000u, 0x00010207u, idx) - 0x03030303u;
}
DEV void lds_or(LAS char* p, u32 v) { __hip_atomic_fetch_or((LAS u32*)p, v, __ATOMIC_RELAXED, __HIP_MEMORY_SCOPE_WORKGROUP); }
DEV void emit16(const u32* L, u32& pos, LAS char* st) {
  u32 w = 0, lp = 0;
#pragma unroll
  for (int i = 0; i < 16; ++i) { lp += (L[i >> 2] >> (8 * (i & 3))) & 255; w |= 1u << (lp & 31); }
  if (lp < 32) {
    const u64 W = (u64)w << (pos & 31); LAS char* p = st + ((pos >> 3) & ~3u);
    lds_or(p, (u32)W); lds_or(p + 4, (u32)(W >> 32));
  } else {
    u32 l0 = L[0], l1 = L[1], l2 = L[2], l3 = L[3], q = pos;
#pragma unroll 1
    for (int i = 0; i < 16; ++i) {
      q += l0 & 255; lds_or(st + ((q >> 3) & ~3u), 1u << (q & 31));
      l0 = __builtin_amdgcn_alignbyte(l1, l0, 1); l1 = __builtin_amdgcn_alignbyte(l2, l1, 1); l2 = __builtin_amdgcn_alignbyte(l3, l2, 1); l3 >>= 8;
    }
  }
  pos += lp;
}
template <int CB> DEV void read_unary(LAS char* st, u32 U, LAS char* K, int lane) {
  u32 last = 0;
  for (u32 s0 = 1; s0 <= U; s0 += 64 * CB) {
    const u32 s = s0 + lane * CB;
    const LAS u32* p = (const LAS u32*)(st + ((s >> 3) & ~3u));
    const int left = (int)(U + 1 - s);
    u32 W = __builtin_amdgcn_ubfe(__builtin_amdgcn_alignbit(p[1], p[0], s), 0, left < 0 ? 0 : left > CB ? CB : left);
    const u32 cnt = __builtin_popcount(W), inc = wave_scan(cnt);
    u32 e = W ? s + 31 - __builtin_clz(W) : 0;
    e = umax(e, dpp0<0x111, 0xf>(e)); e = umax(e, dpp0<0x112, 0xf>(e)); e = umax(e, dpp0<0x114, 0xf>(e)); e = umax(e, dpp0<0x118, 0xf>(e));
    e = umax(e, dpp0<0x142, 0xa>(e)); e = umax(e, dpp0<0x143, 0xc>(e));
    u32 rel = umax(dpp0<0x138, 0xf>(e), last) - s;
    LAS char* k = K + inc - cnt;
    last = umax(last, (u32)__builtin_amdgcn_readlane((int)e, 63));
    K += (u32)__builtin_amdgcn_readlane((int)inc, 63);
    while (W) {
      const u32 b = __builtin_ctz(W), len = b - rel;
      *(LAS unsigned char*)k++ = (unsigned char)(len + perm(0x000002FCu, 0x03030300u, umin(len, 5)));
      rel = b; W &= W - 1;
    }
  }
  cbar();
}
DEV u32 in_bytes(u32 U) { return 10240 + ((U >> 7) + 1) * 16; }
DEV u32 out_bytes(u32 G, u32 U) { return 1024 + 16 * G + ((((2 * G + 3) & ~3u) + ((U >> 5) + 1) * 4 + 15) & ~15u); }

DEV void prep_piece(const Args& a, int mode, bool isx, int s, int j, int h, int dest, u32 callv, char* wsp, LAS char* scr, int lane, int it) {
#define PST(k) do { } while (0)
  const int fr = lane & 15, fq = lane >> 4;
  const int piece = j * 2 + h;
  const u32 slot = (isx ? 0 : SLOT_W + dest * 256) + s * 16 + piece;
  u32 v[64];
  {
    const rsrc_t ri = mkrsrc(isx ? a.x : a.w, isx ? 4096u * 512u * 2u : 512u * 32768u * 2u);
    const u32 lv = isx ? fr * 1024 + fq * 16 : fq * 8 * 65536 + 16 * (4 * (fr & 3) + (fr >> 2));
    const u32 ls = isx ? (s * 256 + (piece & 1) * 128) * 1024 + (piece >> 1) * 128
                       : (piece >> 1) * 64 * 65536 + (dest * 4096 + s * 256 + (piece & 1) * 128) * 2;
    const u32 s8 = isx ? 65536 : 32 * 65536, s2 = isx ? 16384 : 2 * 65536, s1 = isx ? 64 : 65536;
    u32x4 in[16];
#pragma unroll
    for (int i = 0; i < 16; ++i) in[i] = __builtin_amdgcn_raw_buffer_load_b128(ri, lv, ls + (i >> 3) * s8 + ((i >> 1) & 3) * s2 + (i & 1) * s1, PL_AUX);
    __builtin_amdgcn_sched_barrier(0);
    if (isx) {
#pragma unroll
      for (int i = 0; i < 64; ++i) v[i] = in[i >> 2][i & 3];
    } else {
#pragma unroll
      for (int ks = 0; ks < 2; ++ks)
#pragma unroll
        for (int el = 0; el < 8; ++el)
#pragma unroll
          for (int d = 0; d < 4; ++d)
            v[4 * (el * 2 + ks) + d] = perm(in[ks * 8 + 2 * d + 1][el >> 1], in[ks * 8 + 2 * d][el >> 1], (el & 1) ? 0x07060302u : 0x05040100u);
    }
  }
  GAS u32* const own = (GAS u32*)(a.ws[a.rank] + WS_READY + ((isx ? 0 : 4) + (s >> 2)) * 9 * 128);
  const int c = (s < 8 ? (s & 3) + (isx ? 0 : 4) : s - 8) * 2 + h, m = s < 8 ? j * 2 + (s >> 2) : isx ? 16 + j : 24 + j;
  const u32 doff = WS_DESC + ((a.rank * 16 + c) * NIN + m) * 16;
  if (mode & 5) {
    char* const img = a.ws[mode & 4 ? dest : a.rank] + WS_IMG;
    const u32 so = (isx ? 0 : IMG_BYTES) + (u32)(s * 64 + (mode & 4 ? 7 + j * 7 + ((a.rank - dest) & 7) : j)) * 32768u + h * (isx ? 16384 : 8192);
    const rsrc_t rimg[2] = {mkrsrc(img + so, 8192), mkrsrc(img + so + (isx ? 8192 : 16384), 8192)};
#pragma unroll
    for (int i = 0; i < 16; ++i) bst128(rimg[i >> 3], lane * 16 + (i & 7) * 1024, (u32x4){v[4 * i], v[4 * i + 1], v[4 * i + 2], v[4 * i + 3]});
    if (!(mode & 2)) {
      PST(62);
      waitv<0>();
      PST(63);
      asm volatile("" : "+v"(callv), "+v"(wsp));
      if (mode & 4) { if (lane == dest) st_sys128(wsp + doff, (u32x4){callv + 1, 0, 0, callv + 1}); }
      else if (lane == 0) __hip_atomic_fetch_add(own, 1u, __ATOMIC_RELAXED, __HIP_MEMORY_SCOPE_AGENT);
      return;
    }
  }
  float sq = 0.f;
#pragma unroll
  for (int i = 0; i < 16; ++i) sq = sq2(v[4 * i + (i & 3)], sq);
  const u32 cc = pick_c(sq, 2048.f), cp = cc * 0x10001u;
  const rsrc_t rb = mkrsrc(a.ws[a.rank] + WS_BLOB + slot * 16384u, 16384u);
  const u32 vo = lane * 16;
  u32 H[32], L[32], P[8], sum = 0;
#pragma unroll
  for (int r = 0; r < 8; ++r) {
    u32x4 lo; P[r] = 0;
#pragma unroll
    for (int k = 0; k < 4; ++k) {
      const u32 v0 = v[8 * r + 2 * k], v1 = v[8 * r + 2 * k + 1]; u32 l, idx;
      split4(pk_sub(cp, __builtin_amdgcn_alignbit(v0, v0, 31)), pk_sub(cp, __builtin_amdgcn_alignbit(v1, v1, 31)), l, H[4 * r + k], idx, L[4 * r + k]);
      lo[k] = l; P[r] |= perm(0, 0x03020100u, idx) << (2 * k); sum = __builtin_amdgcn_sad_u8(L[4 * r + k], 0, sum);
    }
    bst128(rb, vo + r * 1024, lo);
  }
  const u32 inc = wave_scan(sum), U = (u32)__builtin_amdgcn_readlane((int)inc, 63);
  const bool raw = U > IN_UCAP || (a.flags & 1);
  if (raw) {
#pragma unroll
    for (int r = 0; r < 8; ++r) bst128(rb, vo + (8 + r) * 1024, (u32x4){H[4 * r], H[4 * r + 1], H[4 * r + 2], H[4 * r + 3]});
  } else {
    const u32 ub = in_bytes(U) - 10240;
    for (u32 i = vo; i <= ub; i += 1024) lds_w128(scr + i, (u32x4){0, 0, 0, 0});
    bst128(rb, vo + 8192, (u32x4){P[0], P[1], P[2], P[3]}); bst128(rb, vo + 9216, (u32x4){P[4], P[5], P[6], P[7]});
    cbar();
    u32 pos = inc - sum;
#pragma unroll
    for (int r = 0; r < 8; ++r) emit16(L + 4 * r, pos, scr);
    cbar();
    for (u32 i = vo; i < ub; i += 1024) bst128(rb, 10240 + i, lds_r128(scr + i));
    cbar();
  }
  PST(62);
  waitv<0>();
  PST(63);
  asm volatile("" : "+v"(callv), "+v"(wsp));
  const u32x4 d = {callv + 1, cc | (raw ? 0u : U << 16), raw ? 0x80000000u : 0u, callv + 1};
  if (lane < NR && lane != a.rank && (isx || lane == dest)) st_sys128(wsp + doff, d);
  if ((mode & 1) && lane == 0) __hip_atomic_fetch_add(own, 1u, __ATOMIC_RELAXED, __HIP_MEMORY_SCOPE_AGENT);
}

DEV u32 list_starts(u32x4 bits, u32 (&st)[4]) {
  u32 run = 0;
#pragma unroll
  for (int q = 0; q < 4; ++q) {
    const u32 c = __builtin_popcount(bits[q] >> 16) | (__builtin_popcount(bits[q] & 0xFFFF) << 16), inc = wave_scan(c);
    const u32 tot = (u32)__builtin_amdgcn_readlane((int)inc, 63), r1 = run + (tot & 0xFFFF);
    st[q] = (inc - c + (run | r1 << 16)) * 2;
    run = r1 + (tot >> 16);
  }
  return run;
}
DEV void pack_out(const Args& a, f32x4 (&acc)[2][2][4][2], u32 blob, u32 call, LAS char* lds, int wid, int lane) {
  LAS char* const reg = lds + wid * EREG;
  char* wsp = a.ws[(lane >> 2) & 7];
  u32 tk = 0;
  if ((wid & 3) == 0 && lane == 0) tk = __hip_atomic_fetch_add((GAS u32*)(a.ws[a.rank] + WS_TICKET), 1u, __ATOMIC_RELAXED, __HIP_MEMORY_SCOPE_AGENT);
  u32 o[64];
#pragma unroll
  for (int bj = 0; bj < 2; ++bj)
#pragma unroll
    for (int n = 0; n < 2; ++n)
#pragma unroll
      for (int j = 0; j < 4; ++j)
#pragma unroll
        for (int d = 0; d < 4; ++d) {
          const f32x2 y = {__builtin_elementwise_maximum(acc[d >> 1][bj][(d & 1) * 2][n][j], 0.f),
                           __builtin_elementwise_maximum(acc[d >> 1][bj][(d & 1) * 2 + 1][n][j], 0.f)};
          o[((bj * 2 + n) * 4 + j) * 4 + d] = __builtin_bit_cast(u32, __builtin_convertvector(y, bf16x2)) & 0x7FFF7FFFu;
        }
  u32x4 bits; float sq = 0.f;
#pragma unroll
  for (int q = 0; q < 4; ++q) {
    u32 m16[2];
#pragma unroll
    for (int h = 0; h < 2; ++h) {
      u32 t = 0;
#pragma unroll
      for (int i = 0; i < 8; ++i) t = (t << 2) | pk_min(o[q * 16 + h * 8 + i], 0x00010001u);
      m16[h] = ((t & 0xFFFF) << 1) | (t >> 16);
    }
    bits[q] = (m16[0] << 16) | m16[1];
  }
#pragma unroll
  for (int i = 0; i < 16; ++i) sq = sq2(o[4 * i + (i & 3)], sq);
#pragma unroll
  for (int r = 0; r < 2; ++r) lds_w128(reg + E_SCR + r * 1024 + lane * 16, (u32x4){0, 0, 0, 0});
  const u32 dl = (lane & 15) * 4 + (lane >> 4);
#pragma unroll
  for (int q = 0; q < 4; ++q) bits[q] = (u32)__builtin_amdgcn_ds_bpermute(((lane & 3) * 16 + (lane >> 2)) * 4, (int)bits[q]);
  const rsrc_t rb = mkrsrc(a.ws[a.rank] + WS_BLOB + (SLOT_O + blob) * 16384u, 16384u);
  bst128(rb, lane * 16, bits);
  u32 st[4];
  const u32 n = list_starts(bits, st);
  const u32 cc = pick_c(sq, 0.25f * (float)n);
#pragma unroll
  for (int q = 0; q < 4; ++q) st[q] = (u32)__builtin_amdgcn_ds_bpermute((int)(dl * 4), (int)st[q]);
#pragma unroll
  for (int h = 0; h < 8; ++h) {
    u32 pos = (u32)(size_t)reg + ((h & 1) ? st[h >> 1] >> 16 : st[h >> 1] & 0xFFFF);
#pragma unroll
    for (int i = 0; i < 8; ++i)
      asm volatile("v_cmpx_ne_u16_e32 vcc, 0, %1\n\tds_write_b16 %0, %1\n\tv_add_u32_e32 %0, 2, %0\n\ts_mov_b64 exec, -1\n\t"
                   "v_cmpx_lt_u32_e32 vcc, %2, %1\n\tds_write_b16_d16_hi %0, %1\n\tv_add_u32_e32 %0, 2, %0\n\ts_mov_b64 exec, -1"
                   : "+v"(pos) : "v"(o[h * 8 + i]), "s"(0xFFFFu) : "vcc", "memory");
  }
  if (lane < 16) *(LAS u16*)(reg + 2 * (n + lane)) = (u16)(((cc - 257) & 0xFFFF) >> 1);
  cbar();
  const u32 G = (n + 15) >> 4, ce = (cc - 1) * 0x10001u, ucap = (15296 - 18 * G) * 8;
  LAS char* const us = reg + E_SCR + ((2 * G + 3) & ~3u);
  u32 U = 0; bool raw = a.flags & 2;
  for (u32 g0 = 0; g0 < G && !raw; g0 += 64) {
    const u32 g = g0 + lane; const bool act = g < G;
    const u32x4 t0 = lds_r128(reg + 32 * g), t1 = lds_r128(reg + 32 * g + 16);
    u32 t[8] = {t0.x, t0.y, t0.z, t0.w, t1.x, t1.y, t1.z, t1.w}, L[4], x = 0, sum = 0; u32x4 lo;
#pragma unroll
    for (int k = 0; k < 4; ++k) {
      u32 l, h, idx;
      split4(pk_sub(ce, t[2 * k] << 1), pk_sub(ce, t[2 * k + 1] << 1), l, h, idx, L[k]);
      lo[k] = l | perm(0, 0x01000100u, idx); x |= perm(0, 0x01010000u, idx) << k; sum = __builtin_amdgcn_sad_u8(L[k], 0, sum);
    }
    if (!act) sum = 0;
    const u32 inc = wave_scan(sum);
    u32 pos = U + inc - sum;
    U += (u32)__builtin_amdgcn_readlane((int)inc, 63);
    raw = 9 * U > ucap;
    if (act && !raw) {
      bst128(rb, 1024 + 16 * g, lo);
      *(LAS u16*)(reg + E_SCR + 2 * g) = (u16)(((x << 20) | x) >> 16);
      emit16(L, pos, us);
    }
  }
  cbar();
  if (raw) {
    waitv<0>();
#pragma unroll
    for (int c = 0; c < 16; ++c) bst128(rb, c * 1024 + dl * 16, (u32x4){o[4 * c], o[4 * c + 1], o[4 * c + 2], o[4 * c + 3]});
  } else {
    const u32 eo = 1024 + 16 * G, eb = out_bytes(G, U) - eo;
    for (u32 i = lane * 16; i < eb; i += 1024) bst128(rb, eo + i, lds_r128(reg + E_SCR + i));
  }
  waitv<0>();
  asm volatile("" : "+v"(tk), "+v"(wsp));
  u32 old = 0;
  if (lane == 0) {
    *(LAS u32x2*)(lds + L_DSC + wid * 8) = (u32x2){n | (cc << 16), (raw ? 0x80000000u : U) | (blob << 16)};
    if ((wid & 3) == 0) *(LAS u32*)(lds + L_DSC + 72 + wid) = tk;
    cbar();
    old = __hip_atomic_fetch_add((LAS u32*)(lds + L_DSC + 64 + (wid & 4)), 1u, __ATOMIC_RELAXED, __HIP_MEMORY_SCOPE_WORKGROUP);
  }
  if ((rfl(old) & 3) == 3) {
    cbar();
    const u32x2 e = *(LAS u32x2*)(lds + L_DSC + ((wid & 4) + (lane & 3)) * 8);
    const u32 u = (*(LAS u32*)(lds + L_DSC + 72 + (wid & 4)) & 511) * 4 + (lane & 3);
    if (lane < 4 * NR && (lane >> 2) != a.rank) st_sys128(wsp + WS_DESCO + (a.rank * 2048 + u) * 16, (u32x4){call, e.x, e.y, call});
  }
  const u32 tile = blob >> 3, w8 = blob & 7;
  const u32 so = ((tile >> 4) * 256 + (w8 & 3) * 32) * 65536u + (a.rank * 4096 + (tile & 15) * 256 + (w8 >> 2) * 128) * 2u;
  const rsrc_t rout = mkrsrc(a.out + so, OUT_BYTES - so);
  const u32 vo = (lane & 15) * 65536u + (lane >> 4) * 16;
#pragma unroll
  for (int ch = 0; ch < 16; ++ch)
    __builtin_amdgcn_raw_buffer_store_b128((u32x4){o[4 * ch], o[4 * ch + 1], o[4 * ch + 2], o[4 * ch + 3]}, rout,
                                           vo + (ch & 3) * 64 + ((ch >> 3) * 128 + ((ch >> 2) & 1) * 16) * 65536u, 0, OWN_AUX);
}

DEV void wait_ready(const GAS u32* cp, u32 need, u64 t0, u32& bad) {
  while (!bad) {
    const u32 c = __hip_atomic_load(cp, __ATOMIC_RELAXED, __HIP_MEMORY_SCOPE_AGENT);
    if (__builtin_amdgcn_ballot_w64((int)(c - need) < 0) == 0) break;
    if (now() - t0 > DEADLINE) bad = 1;
    __builtin_amdgcn_s_sleep(4);
  }
  cbar();
}
DEV void gemm_role(const Args& a, LAS char* lds, int pos, int wid, u32 call, u64 t0) {
  char* const wsl = a.ws[a.rank];
  const int wr = wid >> 2, wc = wid & 3;
  u32 bad = 0;
  if (wid == 0) *(LAS u32x2*)(lds + L_DSC + 64) = (u32x2){0, 0};
  for (int round = 0; round < 2; ++round) {
    int tid = threadIdx.x; asm volatile("" : "+v"(tid));
    int lane = tid & 63;
    const int tile = a.tiles[round * 128 + pos], mt = tile >> 4, nt = tile & 15;
    const GAS u32* cp = (const GAS u32*)(wsl + WS_READY + ((lane & 1) ? 4 + (nt >> 2) : mt >> 2) * 9 * 128);
    if (wid == 0) wait_ready(cp, 64u * call, t0, bad);
    asm volatile("s_waitcnt lgkmcnt(0)" ::: "memory");
    __builtin_amdgcn_s_barrier();
    STAMP_AT(2 + round * 3);
    f32x4 acc[2][2][4][2];
#pragma unroll
    for (int i0 = 0; i0 < 2; ++i0)
#pragma unroll
      for (int i1 = 0; i1 < 2; ++i1)
#pragma unroll
        for (int m = 0; m < 4; ++m)
#pragma unroll
          for (int n = 0; n < 2; ++n) acc[i0][i1][m][n] = (f32x4){0.f, 0.f, 0.f, 0.f};
    bf16x8 At[4][2], B0[2][2], B1[2][2];
    const char* cA = wsl + WS_IMG + IMG_BYTES + (size_t)nt * (2u << 20);
    const char* cB = wsl + WS_IMG + (size_t)mt * (2u << 20);
    const u32 voff = tid * 16, ldsw = wid * 1024;
    const int aoff = wr * 8192 + lane * 16, boff = wc * 4096 + lane * 16;
    constexpr int HTB = 16384, KST = 32768, NT = 64;
#define SA(b, h) (((b) * 2 + (h)) * HTB)
#define SB(b, h) ((4 + (b) * 2 + (h)) * HTB)
#define STAGE(bufoff, gbase) do { _Pragma("unroll") for (int _i = 0; _i < 2; ++_i) \
    __builtin_amdgcn_global_load_lds((const GAS u32*)((gbase) + _i * 8192 + voff), (LAS u32*)(lds + (bufoff) + ldsw + _i * 8192), 16, 0, GL_AUX); } while (0)
#define LDA(dst, b, h) do { _Pragma("unroll") for (int m = 0; m < 4; ++m) _Pragma("unroll") for (int k = 0; k < 2; ++k) \
    dst[m][k] = *(const LAS bf16x8*)(lds + SA(b, h) + aoff + m * 2048 + k * 1024); } while (0)
#define LDB(dst, b, h) do { _Pragma("unroll") for (int n = 0; n < 2; ++n) _Pragma("unroll") for (int k = 0; k < 2; ++k) \
    dst[n][k] = *(const LAS bf16x8*)(lds + SB(b, h) + boff + n * 2048 + k * 1024); } while (0)
#define MMA(ai, bj, A_, B_) do { __builtin_amdgcn_s_setprio(1); _Pragma("unroll") for (int m = 0; m < 4; ++m) _Pragma("unroll") for (int n = 0; n < 2; ++n) \
    _Pragma("unroll") for (int k = 0; k < 2; ++k) acc[ai][bj][m][n] = __builtin_amdgcn_mfma_f32_16x16x32_bf16(A_[m][k], B_[n][k], acc[ai][bj][m][n], 0, 0, 0); \
    __builtin_amdgcn_s_setprio(0); } while (0)
#define WAIT_V(n) asm volatile("s_waitcnt vmcnt(" #n ")" ::: "memory")
#define WAIT_L(n) asm volatile("s_waitcnt lgkmcnt(" #n ")" ::: "memory")
#define BAR __builtin_amdgcn_s_barrier()
#define SCHED __builtin_amdgcn_sched_barrier(0)
    STAGE(SB(0, 0), cB); STAGE(SA(0, 0), cA); STAGE(SB(0, 1), cB + HTB); STAGE(SA(0, 1), cA + HTB);
    if (wr == 1) BAR;
    WAIT_V(4); BAR;
    STAGE(SB(1, 0), cB + KST); STAGE(SA(1, 0), cA + KST); STAGE(SB(1, 1), cB + HTB + KST);
    WAIT_V(6); BAR;
    int tn = 5;
    for (int t = 0; t < NT; t += 2) {
      if (wid == 0 && tn < 61) {
        if (t >= tn) {
          const u32 c = lds_poll(LDS_BYTES - 32 + (lane & 1) * 4);
          STAMP_AT(16 + round * 16 + (tn / 7) * 2);
          if (__builtin_amdgcn_ballot_w64((int)(c - 56u * call) < 0) != 0) wait_ready(cp, 56u * call, t0, bad);
          STAMP_AT(17 + round * 16 + (tn / 7) * 2);
          tn += 7;
        } else if (t + 2 >= tn) {
          cp += 32;
          if (lane < 2) __builtin_amdgcn_global_load_lds(cp, (LAS u32*)(lds + LDS_BYTES - 32), 4, 0, 17);
        }
      }
      const bool last = (t == NT - 2);
      const char* a1 = cA + (size_t)(t + 1) * KST;
      const char* a2 = last ? cA : cA + (size_t)(t + 2) * KST; const char* b2 = last ? cB : cB + (size_t)(t + 2) * KST;
      const char* a3 = a2 + KST; const char* b3 = b2 + KST;
      LDB(B0, 0, 0); SCHED; LDA(At, 0, 0); STAGE(SA(1, 1), a1 + HTB);
      WAIT_L(8); BAR; WAIT_L(0); MMA(0, 0, At, B0); BAR; SCHED;
      LDB(B1, 0, 1); STAGE(SB(0, 0), b2);
      BAR; WAIT_L(0); MMA(0, 1, At, B1); BAR;
      LDA(At, 0, 1); STAGE(SA(0, 0), a2);
      BAR; WAIT_L(0); MMA(1, 0, At, B0); BAR; SCHED;
      STAGE(SB(0, 1), b2 + HTB);
      WAIT_V(6); BAR; MMA(1, 1, At, B1); BAR;
      LDB(B0, 1, 0); SCHED; LDA(At, 1, 0); STAGE(SA(0, 1), a2 + HTB);
      WAIT_L(8); BAR; WAIT_L(0); MMA(0, 0, At, B0); BAR; SCHED;
      LDB(B1, 1, 1); STAGE(SB(1, 0), b3);
      BAR; WAIT_L(0); MMA(0, 1, At, B1); BAR;
      LDA(At, 1, 1); STAGE(SA(1, 0), a3);
      BAR; WAIT_L(0); MMA(1, 0, At, B0); BAR; SCHED;
      STAGE(SB(1, 1), b3 + HTB);
      WAIT_V(6); BAR; MMA(1, 1, At, B1); BAR;
    }
    if (wid == 0) lds_post(LDS_BYTES - 16, bad);
    asm volatile("s_waitcnt vmcnt(0) lgkmcnt(0)" ::: "memory");
    if (wr == 0) BAR;
    BAR;
#undef SA
#undef SB
#undef STAGE
#undef LDA
#undef LDB
#undef MMA
    int tz = threadIdx.x; asm volatile("" : "+v"(tz));
    lane = tz & 63;
    STAMP_AT(3 + round * 3);
    if (rfl(lds_poll(LDS_BYTES - 16))) {
#pragma unroll
      for (int i = 0; i < 32; ++i) acc[i >> 4][(i >> 3) & 1][(i >> 1) & 3][i & 1] = (f32x4){__builtin_nanf(""), __builtin_nanf(""), __builtin_nanf(""), __builtin_nanf("")};
    }
    pack_out(a, acc, tile * 8 + wid, call, lds, wid, lane);
    STAMP_AT(4 + round * 3);
  }
}

DEV void loader_role(const Args& a, LAS char* lds, int peer, int lane, u64 t0) {
  const char* src = a.ws[peer] + WS_BLOB + lane * 16;
  u32 served = 0; int pend = -1; u32 pend_tag = 0;
  int done = 0;
  for (int left = NDEC; left > 0;) {
    const u32 r = lds_poll(L_REQ + (lane & 7) * 4);
    u32 key = ((lane & 7) < NDEC && (r & 255) != served) ? ((255 - (r & 255)) << 3 | (7 - (lane & 7))) + 1 : 0;
    key = umax(key, dpp0<0x111, 0xf>(key)); key = umax(key, dpp0<0x112, 0xf>(key)); key = umax(key, dpp0<0x114, 0xf>(key));
    key = (u32)__builtin_amdgcn_readlane((int)key, 7);
    if (key == 0) {
      if (pend >= 0) { waitv<0>(); lds_post(L_FULL + pend * 4, pend_tag); pend = -1; }
      else { if (now() - t0 > DEADLINE + 20000000ull) break; __builtin_amdgcn_s_sleep(2); }
      continue;
    }
    const int j = 7 - ((key - 1) & 7);
    const u32 rq = (u32)__builtin_amdgcn_readlane((int)r, j);
    if ((lane & 7) == j) served = rq & 255;
    if (rq >> 31) { --left; continue; }
    ++done;
    const u32 slot = umin(rq >> 18, NSLOT - 1), bytes = (((rq >> 8) & 1023) + 1) * 16;
    const char* s = src + (size_t)slot * 16384;
    LAS char* dst = lds + j * DREG + BOFF;
    int row = 0;
    for (u32 o = 0; o < bytes; o += 1024, ++row) {
      if (o + lane * 16 < bytes) __builtin_amdgcn_global_load_lds((const GAS u32*)(s + o), (LAS u32*)(dst + o), 16, 0, LD_AUX);
      waitv<WIN - 1>();
      if (row == WIN - 2 && pend >= 0) { lds_post(L_FULL + pend * 4, pend_tag); pend = -1; }
    }
    if (pend >= 0) { waitv<0>(); lds_post(L_FULL + pend * 4, pend_tag); }
    pend = j; pend_tag = rq & 255;
  }
  waitv<0>();
  if (pend >= 0) lds_post(L_FULL + pend * 4, pend_tag);
}

DEV void decoder_role(const Args& a, LAS char* lds, int peer, int c, int j, int lane, u32 call, u64 t0) {
  char* const wsl = a.ws[a.rank];
  LAS char* A = lds + j * DREG;
  LAS char* B = A + BOFF;
  const int q = (peer - a.rank) & 7;
  bool fail = false, posted = false;
  u32 tag = 1;
  const char* dsc = wsl + WS_DESC + (size_t)((peer * 16 + c) * NIN + j) * 16;
  u32 da = 0, db = 0;
  auto post = [&](int m, u32 da, u32 db, u32 tag) {
    u32 slot, bytes;
    if (m < NIN) {
      const int kj = m < 16 ? m >> 1 : m & 7, t = m < 16 ? c >> 3 : (m >> 3) & 1, s = m < 16 ? (m & 1) * 4 + ((c >> 1) & 3) : 8 + (c >> 1);
      slot = (t ? SLOT_W + a.rank * 256 : 0) + s * 16 + kj * 2 + (c & 1);
      bytes = in_bytes(umin(da >> 16, IN_UCAP));
    } else {
      const u32 G = (umin(da & 0xFFFF, 8192) + 15) >> 4;
      slot = SLOT_O + ((db >> 16) & 2047);
      bytes = out_bytes(G, umin(db & 0xFFFF, OUT_UCAP));
    }
    if (db >> 31) bytes = 16384;
    asm volatile("s_waitcnt lgkmcnt(0)" ::: "memory");
    lds_post(L_REQ + j * 4, (slot << 18) | ((bytes / 16 - 1) << 8) | tag);
  };
  for (int m = j;; m += NDEC) {
    if (!posted) {
      u32x4 d;
      for (;;) {
        d = ld_sys128(dsc);
        if (rfl(d.x) == call && rfl(d.w) == call) break;
        if (now() - t0 > DEADLINE) { fail = true; break; }
        __builtin_amdgcn_s_sleep(4);
      }
      if (fail) break;
      da = rfl(d.y); db = rfl(d.z);
      if (m) post(m, da, db, tag);
    }
    if (tag <= 11) STAMP_AT(12 + tag * 4);
    bool more = true, nok = false; u32 nda = 0, ndb = 0;
    dsc += NDEC * 16;
    if (m + NDEC >= NIN) {
      u32 u = 0;
      if (lane == 0) u = __hip_atomic_fetch_add((GAS u32*)(wsl + WS_QUEUE + peer * 128), 1u, __ATOMIC_RELAXED, __HIP_MEMORY_SCOPE_AGENT);
      u = rfl(u) - (call - 1) * QSTEP;
      more = u < 2048;
      dsc = wsl + WS_DESCO + (size_t)(peer * 2048 + (u & 2047)) * 16;
    }
    if (more) {
      const u32x4 d = ld_sys128(dsc);
      nok = rfl(d.x) == call && rfl(d.w) == call; nda = rfl(d.y); ndb = rfl(d.z);
    }
    const u32 ntag = tag == 255 ? 1 : tag + 1;
    const bool raw = db >> 31;
    const int kj = m < 16 ? m >> 1 : m & 7, t = m < 16 ? c >> 3 : (m >> 3) & 1, s = m < 16 ? (m & 1) * 4 + ((c >> 1) & 3) : 8 + (c >> 1);
    const u32 U = m < NIN ? umin(da >> 16, IN_UCAP) : umin(db & 0xFFFF, OUT_UCAP), G = (umin(da & 0xFFFF, 8192) + 15) >> 4;
    while (m) {
      if (rfl(lds_poll(L_FULL + j * 4)) == tag) break;
      if (now() - t0 > DEADLINE) { fail = true; break; }
      __builtin_amdgcn_s_sleep(2);
    }
    if (fail) break;
    posted = false;
    if (tag <= 11) STAMP_AT(13 + tag * 4);
    if (m == 0) {
      if (lane == 0) __hip_atomic_fetch_add((GAS u32*)(wsl + WS_READY + (t * 4 * 9 + 1) * 128), 1u, __ATOMIC_RELAXED, __HIP_MEMORY_SCOPE_AGENT);
    } else
    if (m < NIN) {
      const u32 cp = (da & 0xFFFF) * 0x10001u;
      u32 H[32];
      if (raw) {
#pragma unroll
        for (int r = 0; r < 8; ++r) { const u32x4 t = lds_r128(B + (8 + r) * 1024 + lane * 16); H[4 * r] = t.x; H[4 * r + 1] = t.y; H[4 * r + 2] = t.z; H[4 * r + 3] = t.w; }
      } else {
        const u32x4 p0 = lds_r128(B + 8192 + lane * 16), p1 = lds_r128(B + 9216 + lane * 16);
        read_unary<26>(B + 10240, U, A, lane);
        const u32 P[8] = {p0.x, p0.y, p0.z, p0.w, p1.x, p1.y, p1.z, p1.w};
        u32 E[8], cnt = 0;
#pragma unroll
        for (int r = 0; r < 8; ++r) { E[r] = ~(P[r] | (P[r] >> 1)) & 0x55555555u; cnt += __builtin_popcount(E[r]); }
        LAS char* k1 = A + (wave_scan(cnt) - cnt);
#pragma unroll
        for (int i = 0; i < 32; ++i) {
          const u32 e = (E[i >> 2] >> (i & 2) * 2) & (0x01010101u << (i & 1) * 2);
          const u32 sel = *(LAS u32*)(lds + __builtin_amdgcn_udot4(e, (i & 1) ? 0x08040201u : 0x20100804u, L_LUT2, false));
          H[i] = perm((P[i >> 2] >> (i & 3) * 2) & 0x03030303u, *(LAS u32u*)k1, sel);
          k1 += __builtin_popcount(e);
        }
      }
      const u32 so = t * IMG_BYTES + (u32)(s * 64 + 7 + kj * 7 + q) * 32768u + (c & 1) * (t ? 8192 : 16384);
      const rsrc_t rimg[2] = {mkrsrc(wsl + WS_IMG + so, 8192), mkrsrc(wsl + WS_IMG + so + (t ? 16384 : 8192), 8192)};
#pragma unroll
      for (int r = 0; r < 8; ++r) {
        const u32x4 lo = lds_r128(B + r * 1024 + lane * 16);
        u32 v[8];
#pragma unroll
        for (int k = 0; k < 4; ++k) {
          const u32 u0 = pk_sub(cp, perm(H[4 * r + k], lo[k], 0x05010400u)), u1 = pk_sub(cp, perm(H[4 * r + k], lo[k], 0x07030602u));
          v[2 * k] = __builtin_amdgcn_alignbit(u0, u0, 1); v[2 * k + 1] = __builtin_amdgcn_alignbit(u1, u1, 1);
        }
        if (!NOIMG || call < 3) {
        ist128(rimg[r >> 2], lane * 16 + ((2 * r) & 7) * 1024, (u32x4){v[0], v[1], v[2], v[3]});
        ist128(rimg[r >> 2], lane * 16 + ((2 * r + 1) & 7) * 1024, (u32x4){v[4], v[5], v[6], v[7]});
        }
      }
      if (nok) { post(m + NDEC, nda, ndb, ntag); posted = true; }
      if (tag <= 11) STAMP_AT(14 + tag * 4);
      waitv<0>();
      if (tag <= 11) STAMP_AT(15 + tag * 4);
      if (lane == 0) __hip_atomic_fetch_add((GAS u32*)(wsl + WS_READY + ((t * 4 + (s >> 2)) * 9 + 1 + kj) * 128), 1u, __ATOMIC_RELAXED, __HIP_MEMORY_SCOPE_AGENT);
    } else {
      const u32 blob = (db >> 16) & 2047, tile = blob >> 3, w8 = blob & 7;
      const u32 so = ((tile >> 4) * 256 + (w8 & 3) * 32) * 65536u + (peer * 4096 + (tile & 15) * 256 + (w8 >> 2) * 128) * 2u;
      const u32 vo = (lane >> 2) * 65536u + (lane & 3) * 16;
      const rsrc_t rout = mkrsrc(a.out + so, OUT_BYTES - so);
      if (raw) {
#pragma unroll
        for (int ch = 0; ch < 16; ++ch)
          __builtin_amdgcn_raw_buffer_store_b128(lds_r128(B + ch * 1024 + lane * 16), rout, vo + (ch & 3) * 64 + ((ch >> 3) * 128 + ((ch >> 2) & 1) * 16) * 65536u, 0, OUT_AUX);
      } else {
        const u32 cp = (da >> 16) * 0x10001u;
        const u32x4 bits = lds_r128(B + lane * 16);
        LAS char* const pl = B + 1024 + 16 * G;
        LAS char* k0 = B + out_bytes(G, U);
        read_unary<20>(pl + ((2 * G + 3) & ~3u), U, k0, lane);
        for (u32 g0 = 0; g0 < G; g0 += 64) {
          const u32 g = g0 + lane;
          const u32x4 lo = lds_r128(B + 1024 + 16 * g); const u32 w = *(LAS u16*)(pl + 2 * g), x = (w << 16) | (w >> 4);
          u32 v[8], e[4], c[5];
          c[0] = 0;
#pragma unroll
          for (int k = 0; k < 4; ++k) { e[k] = ~((x >> k) | lo[k]) & 0x01010101u; c[k + 1] = c[k] + __builtin_popcount(e[k]); }
          const u32 inc = wave_scan(c[4]);
          LAS char* const k1 = k0 + (inc - c[4]);
          k0 += (u32)__builtin_amdgcn_readlane((int)inc, 63);
#pragma unroll
          for (int k = 0; k < 4; ++k) {
            const u32 sel = *(LAS u32*)(lds + __builtin_amdgcn_udot4(e[k], 0x20100804u, L_LUT2, false));
            const u32 h = perm((lo[k] & 0x01010101u) | ((k ? x >> (k - 1) : x << 1) & 0x02020202u), *(LAS u32u*)(k1 + c[k]), sel);
            v[2 * k] = pk_shr1(pk_sub(cp, perm(h, lo[k], 0x05010400u))); v[2 * k + 1] = pk_shr1(pk_sub(cp, perm(h, lo[k], 0x07030602u)));
          }
          if (g < G) { lds_w128(A + 32 * g, (u32x4){v[0], v[1], v[2], v[3]}); lds_w128(A + 32 * g + 16, (u32x4){v[4], v[5], v[6], v[7]}); }
          cbar();
        }
        if (nok && 32 * G + 16 <= BOFF) { post(m + NDEC, nda, ndb, ntag); posted = true; }
        u32 st[4], pos = 0;
        list_starts(bits, st);
#pragma unroll
        for (int ch = 0; ch < 16; ++ch) {
          u32x4 ov;
          if (!(ch & 1)) pos = (ch & 2) ? st[ch >> 2] >> 16 : st[ch >> 2] & 0xFFFF;
#pragma unroll
          for (int h = 0; h < 2; ++h) {
            const u32 nib = (bits[ch >> 2] >> (28 - 4 * ((ch & 3) * 2 + h))) & 15;
            const u32x2 sel = *(LAS u32x2*)(lds + L_LUT + nib * 8);
            LAS u32* q = (LAS u32*)(A + (pos & ~3u));
            const u32 d0 = q[0], d1 = q[1], d2 = q[2];
            const u32 e0 = __builtin_amdgcn_alignbyte(d1, d0, pos), e1 = __builtin_amdgcn_alignbyte(d2, d1, pos);
            ov[2 * h] = perm(e1, e0, sel.x); ov[2 * h + 1] = perm(e1, e0, sel.y);
            pos += 2 * __builtin_popcount(nib);
          }
          __builtin_amdgcn_raw_buffer_store_b128(ov, rout, vo + (ch & 3) * 64 + ((ch >> 3) * 128 + ((ch >> 2) & 1) * 16) * 65536u, 0, OUT_AUX);
        }
      }
      cbar();
      if (nok && !posted) { post(m + NDEC, nda, ndb, ntag); posted = true; }
      if (tag <= 11) STAMP_AT(14 + tag * 4);
    }
    tag = ntag; da = nda; db = ndb;
    if (!more) break;
  }
  if (fail) tag = tag == 255 ? 1 : tag + 1;
  lds_post(L_REQ + j * 4, 0x80000000u | tag);
  if (fail) {
    waitv<0>();
    __builtin_amdgcn_raw_buffer_store_b32(0x7FC07FC0u, mkrsrc(a.out, OUT_BYTES), lane * 4 + (blockIdx.x * 8 + j) * 65536u, 0, 0);
    if (lane == 0) __hip_atomic_store((GAS u32*)(wsl + WS_TMO), call, __ATOMIC_RELAXED, __HIP_MEMORY_SCOPE_AGENT);
  }
}

__global__ void __launch_bounds__(512, 2) fused_gemm_ar(const Args a) {
  extern __shared__ __attribute__((aligned(16))) char smem[];
  LAS char* lds = (LAS char*)smem;
  const u64 t0 = now();
  const int wid = __builtin_amdgcn_readfirstlane(threadIdx.x >> 6), lane = threadIdx.x & 63;
  const int xcd = vbid() & 7, li = vbid() >> 3;
  char* const wsl = a.ws[a.rank];
  GAS u32* callp = (GAS u32*)(wsl + WS_CALL) + blockIdx.x * 8 + wid;
  const u32 callv = __hip_atomic_load(callp, __ATOMIC_RELAXED, __HIP_MEMORY_SCOPE_AGENT);
  char* const wsp = a.ws[lane & 7];
  STAMP_AT(0);
  const bool is_gemm = li < 16;
  const int pos = xcd * 16 + (li & 15);
  if (!is_gemm) {
    if (wid == 0) {
      if (lane < 32) *(LAS u32*)(lds + L_CTRL + lane * 4) = 0;
      if (lane < 16) {
        u32 k = 0, sel[2] = {0, 0};
#pragma unroll
        for (int i = 0; i < 4; ++i) {
          const bool on = (lane >> (3 - i)) & 1;
          sel[i >> 1] |= (on ? (2 * k) | ((2 * k + 1) << 8) : 0x0C0Cu) << (16 * (i & 1));
          k += on;
        }
        *(LAS u32x2*)(lds + L_LUT + lane * 8) = (u32x2){sel[0], sel[1]};
        u32 s2 = 0; k = 0;
#pragma unroll
        for (int i = 0; i < 4; ++i) { const bool on = (lane >> i) & 1; s2 |= (on ? k : 4 + i) << (8 * i); k += on; }
        *(LAS u32*)(lds + L_LUT2 + lane * 4) = s2;
      }
    }
    asm volatile("s_waitcnt lgkmcnt(0)" ::: "memory");
    __builtin_amdgcn_s_barrier();
  }
  const int q = li - 16;
  const bool is_help = !is_gemm && (q & 7) == 0;
  {
    int e = -1, n = 1;
    const bool early = is_gemm && (xcd & 1);
    if (early) { if (wid < 4) e = 2176 + wid * 64 + (xcd >> 1) * 16 + li; }
    else if (is_gemm) { e = 512 + wid * 64 + (xcd >> 1) * 16 + li; n = 3; }
    else if (is_help) { e = 1024 + (xcd * 2 + (q >> 3)) * 8 + wid; n = 2; }
    else if (wid >= 4) e = 64 + (wid - 4) * 112 + (xcd * 2 + (q >> 3)) * 7 + (q & 7) - 1;
    else if (wid == 1) e = xcd * 2 + (q >> 3);
    LAS char* scr = is_gemm || is_help ? lds + wid * EREG : wid == 7 ? lds + L_SCR7 : lds + (wid - 1) * DREG + BOFF;
    if (GDELAY && n > 1) for (int i = 0; i < GDELAY; ++i) __builtin_amdgcn_s_sleep(37);
    if (e >= 0)
      for (int i = 0; i < n && (e < 2176 || n == 1); ++i, e += 640) {
        bool isx; int j, s, dq, mode;
        if (e < 64) { j = 0; isx = e < 8; s = (e >> 1) & 3; dq = -q; mode = 4; }
        else if (e < 1024) { j = e >> 7; dq = (e >> 3) & 7; isx = dq == 0; s = ((e >> 6) & 1) * 4 + ((e >> 1) & 3); mode = 2; }
        else if (e < 1152) { j = (e >> 4) & 7; isx = true; s = 8 + ((e >> 1) & 7); dq = 0; mode = 3; }
        else if (e < 2176) { j = (e - 1152) >> 7; isx = false; s = 8 + ((e >> 1) & 7); dq = (e >> 4) & 7; mode = dq ? 2 : 1; }
        else { j = (e >> 4) & 7; isx = e < 2304; s = (e >> 1) & 7; dq = 0; mode = 1; }
        prep_piece(a, mode, isx, s, j, e & 1, (a.rank + dq) & 7, callv, wsp, scr, lane, i);
        if (is_help) STAMP_AT(16 + i);
      }
  }
  const u32 call = rfl(callv) + 1;
  if (lane == 0) __hip_atomic_store(callp, call, __ATOMIC_RELAXED, __HIP_MEMORY_SCOPE_AGENT);
  STAMP_AT(1);
  if (is_gemm) {
    gemm_role(a, lds, pos, wid, call, t0);
  } else if (!is_help) {
    const int peer = (a.rank + (q & 7)) & 7, c = xcd * 2 + (q >> 3);
    if (wid == 0) loader_role(a, lds, peer, lane, t0);
    else if (wid <= NDEC) decoder_role(a, lds, peer, c, wid - 1, lane, call, t0);
  }
  STAMP_AT(8);
}

#define HIP_CHECK(expr) do { hipError_t _e = (expr); if (_e != hipSuccess) { \
    fprintf(stderr, "HIP error %s at %s:%d\n", hipGetErrorString(_e), __FILE__, __LINE__); \
    return; } } while (0)

#define DBG_FLAGS 0

extern "C" void kernel_launch(int rank, int nranks, ncclComm_t comm, hipStream_t stream,
                              void* const* d_in, const int* in_sizes, int n_in,
                              void* d_out, int out_size, const int* in_dtypes, int out_dtype,
                              void* const* d_out_all, void* const* d_ws_all, size_t ws_bytes) {
  if (nranks != NR || (unsigned)rank >= (unsigned)NR || n_in < 2 || ws_bytes < WS_USED || (size_t)out_size * 2 != OUT_BYTES ||
      in_sizes[0] != 4096 * 512 || in_sizes[1] != 512 * 32768) {
    fprintf(stderr, "unexpected problem shape\n");
    return;
  }
  static bool attr_set[NR] = {};
  if (!attr_set[rank]) {
    HIP_CHECK(hipFuncSetAttribute((const void*)fused_gemm_ar, hipFuncAttributeMaxDynamicSharedMemorySize, LDS_BYTES));
    attr_set[rank] = true;
  }
  Args a;
  a.x = (const char*)d_in[0]; a.w = (const char*)d_in[1]; a.out = (char*)d_out;
  for (int p = 0; p < NR; ++p) a.ws[p] = (char*)d_ws_all[p];
  a.rank = rank; a.flags = DBG_FLAGS;
  static const int blk[8][2] = {{2, 0}, {0, 0}, {2, 1}, {1, 0}, {3, 0}, {0, 1}, {3, 1}, {1, 1}};
  for (int k = 0; k < 256; ++k) {
    const int b = (k >> 4) & 7, i = k & 15;
    a.tiles[k] = (unsigned char)((blk[b][0] * 4 + (i >> 2)) * 16 + (blk[b][1] + (k >> 7) * 2) * 4 + (i & 3));
  }
  hipLaunchKernelGGL(fused_gemm_ar, dim3(256), dim3(512), LDS_BYTES, stream, a);
  HIP_CHECK(hipGetLastError());
}
```

```cpp
#include <hip/hip_runtime.h>
#include <rccl/rccl.h>
#include <cstdio>

typedef unsigned u32;
typedef unsigned long long u64;
typedef u32 u32x4 __attribute__((ext_vector_type(4)));
typedef u32 u32x2 __attribute__((ext_vector_type(2)));
typedef u32 u32u __attribute__((aligned(1)));
typedef u32x2 u32x2u __attribute__((aligned(2)));
typedef unsigned short u16;
typedef u16 u16x2 __attribute__((ext_vector_type(2)));
typedef short bf16x8 __attribute__((ext_vector_type(8)));
typedef float f32x4 __attribute__((ext_vector_type(4)));
typedef float f32x2 __attribute__((ext_vector_type(2)));
typedef __bf16 bf16x2 __attribute__((ext_vector_type(2)));
#define LAS __attribute__((address_space(3)))
#define GAS __attribute__((address_space(1)))
#define DEV __device__ __forceinline__

constexpr int NR = 8;
constexpr u32 WS_CALL = 0;
constexpr u32 WS_TMO = 8192;
constexpr u32 WS_TICKET = 8192 + 128;
constexpr u32 WS_QUEUE = 8192 + 256;
constexpr u32 WS_READY = 16384;
constexpr int NIN = 32;
constexpr u32 WS_DESC = 512u << 10;
constexpr u32 WS_DESCO = WS_DESC + 65536;
constexpr u32 WS_IMG = 1u << 20;
constexpr u32 IMG_BYTES = 32u << 20;
constexpr u32 WS_BLOB = 65u << 20;
constexpr u32 SLOT_W = 256, SLOT_O = 2304, NSLOT = 4352;
constexpr u32 WS_USED = WS_BLOB + NSLOT * 16384u;
constexpr u32 WS_STAMP = WS_USED;
constexpr u32 OUT_BYTES = 4096u * 32768u * 2u;
constexpr u64 DEADLINE = 100000000ull;
constexpr int LDS_BYTES = 163840;
constexpr int NDEC = 6;
constexpr int DREG = 26624, BOFF = 10240;
constexpr int L_REQ = 0, L_FULL = 64, L_LUT = 128, L_LUT2 = 256;
constexpr int L_DEC = 512, L_SCR7 = L_DEC + NDEC * DREG;
constexpr int EREG = 20480;
constexpr int E_SCR = 16448;
constexpr int L_DSC = LDS_BYTES - 256;
constexpr u32 IN_UCAP = 28000, OUT_UCAP = 13600;
#define NHELP 2
constexpr u32 QSTEP = 2048 + (16 + NHELP) * NDEC;
#define WIN 4
#define GDELAY 0
#define OWN_AUX 0
#define OUT_AUX 0
#define IMG_AUX 17
#define GL_AUX 17
#define PL_AUX 2
#define SEG 0
DEV int vbid() {
  const int x = blockIdx.x & 7, l = blockIdx.x >> 3;
  if (SEG == 0) return blockIdx.x;
  const int comm = SEG == 1 ? (x & 1) ^ 1 : SEG == 2 ? x & 1 : SEG == 3 ? x >> 2 : (x >> 1) & 1;
  const int g = SEG <= 2 ? x >> 1 : SEG == 3 ? x & 3 : (x & 1) | ((x >> 2) << 1);
  return comm * 128 + (l & 15) * 8 + g * 2 + (l >> 4);
}
#define LD_AUX 17
#define NOIMG 0

struct Args {
  const char* x; const char* w; char* out; char* ws[NR]; int rank; int flags; unsigned char tiles[256];
};

template <int N> DEV void waitv() { asm volatile("s_waitcnt vmcnt(%0)" ::"n"(N) : "memory"); }
DEV void cbar() { asm volatile("" ::: "memory"); }
DEV u64 now() { return __builtin_amdgcn_s_memrealtime(); }
DEV u32 perm(u32 hi, u32 lo, u32 sel) { return __builtin_amdgcn_perm(hi, lo, sel); }
DEV u32 pk_sub(u32 a, u32 b) { return __builtin_bit_cast(u32, __builtin_bit_cast(u16x2, a) - __builtin_bit_cast(u16x2, b)); }
DEV u32 pk_add(u32 a, u32 b) { return __builtin_bit_cast(u32, __builtin_bit_cast(u16x2, a) + __builtin_bit_cast(u16x2, b)); }
DEV u32 pk_shr1(u32 a) { return __builtin_bit_cast(u32, __builtin_bit_cast(u16x2, a) >> (u16x2){1, 1}); }
DEV u32 pk_max(u32 a, u32 b) { return __builtin_bit_cast(u32, __builtin_elementwise_max(__builtin_bit_cast(u16x2, a), __builtin_bit_cast(u16x2, b))); }
DEV u32 pk_min(u32 a, u32 b) { return __builtin_bit_cast(u32, __builtin_elementwise_min(__builtin_bit_cast(u16x2, a), __builtin_bit_cast(u16x2, b))); }
template <int C, int RM> DEV u32 dpp0(u32 x) { return (u32)__builtin_amdgcn_update_dpp(0, (int)x, C, RM, 0xf, false); }
DEV u32 umax(u32 a, u32 b) { return a > b ? a : b; }
DEV u32 umin(u32 a, u32 b) { return a < b ? a : b; }
DEV u32 wave_max(u32 x) {
  x = umax(x, dpp0<0x111, 0xf>(x)); x = umax(x, dpp0<0x112, 0xf>(x)); x = umax(x, dpp0<0x114, 0xf>(x)); x = umax(x, dpp0<0x118, 0xf>(x));
  x = umax(x, dpp0<0x142, 0xa>(x)); x = umax(x, dpp0<0x143, 0xc>(x));
  return (u32)__builtin_amdgcn_readlane((int)x, 63);
}
DEV u32 wave_scan(u32 x) {
  x += dpp0<0x111, 0xf>(x); x += dpp0<0x112, 0xf>(x); x += dpp0<0x114, 0xf>(x); x += dpp0<0x118, 0xf>(x);
  x += dpp0<0x142, 0xa>(x); x += dpp0<0x143, 0xc>(x);
  return x;
}
DEV u32 rfl(u32 x) { return (u32)__builtin_amdgcn_readfirstlane((int)x); }
DEV u32 mbcnt(u64 m) { return __builtin_amdgcn_mbcnt_hi((u32)(m >> 32), __builtin_amdgcn_mbcnt_lo((u32)m, 0)); }
DEV u32x4 lds_r128(LAS char* p) { return *(LAS u32x4*)p; }
DEV void lds_w128(LAS char* p, u32x4 v) { *(LAS u32x4*)p = v; }
DEV u32 lds_poll(u32 addr) { u32 v; asm volatile("ds_read_b32 %0, %1\n\ts_waitcnt lgkmcnt(0)" : "=v"(v) : "v"(addr) : "memory"); return v; }
DEV void lds_post(u32 addr, u32 v) { asm volatile("ds_write_b32 %0, %1" ::"v"(addr), "v"(v) : "memory"); }
DEV void st_sys128(void* p, u32x4 v) { asm volatile("global_store_dwordx4 %0, %1, off sc0 sc1\n\ts_nop 1" ::"v"(p), "v"(v) : "memory"); }
DEV u32x4 ld_sys128(const void* p) { u32x4 v; asm volatile("global_load_dwordx4 %0, %1, off sc0 sc1\n\ts_waitcnt vmcnt(0)" : "=&v"(v) : "v"(p) : "memory"); return v; }
DEV LAS char* lp(u32 addr) { return (LAS char*)(size_t)addr; }
DEV u32 draw(const void* p, bool on) {
  u32 u; asm volatile("s_mov_b32 exec_lo, %4\n\ts_mov_b32 exec_hi, 0\n\tglobal_atomic_add %0, %1, %2, %3 sc0\n\ts_mov_b64 exec, -1" : "=&v"(u) : "v"(0), "v"(1), "s"(p), "s"(rfl(on)) : "memory");
  return u;
}
template <int N> DEV u32 collect(u32 u) { u32 r; asm volatile("s_waitcnt vmcnt(%2)\n\tv_readfirstlane_b32 %0, %1" : "=s"(r) : "v"(u), "n"(N) : "memory"); return r; }
#define SCHED __builtin_amdgcn_sched_barrier(0)
typedef __amdgpu_buffer_rsrc_t rsrc_t;
DEV rsrc_t mkrsrc(const void* p, u32 bytes) { return __builtin_amdgcn_make_buffer_rsrc((void*)p, 0, (int)bytes, 0x00020000); }
DEV void bst128(rsrc_t r, u32 voff, u32x4 v) { __builtin_amdgcn_raw_buffer_store_b128(v, r, (int)voff, 0, 17); }
DEV void ist128(rsrc_t r, u32 voff, u32x4 v) { __builtin_amdgcn_raw_buffer_store_b128(v, r, (int)voff, 0, IMG_AUX); }
DEV void bst32(rsrc_t r, u32 voff, u32 v) { __builtin_amdgcn_raw_buffer_store_b32(v, r, (int)voff, 0, 17); }

#define STAMP_AT(k) do { } while (0)

DEV u32 pick_c(float sq, float n) {
  sq += __builtin_bit_cast(float, dpp0<0x111, 0xf>(__builtin_bit_cast(u32, sq))); sq += __builtin_bit_cast(float, dpp0<0x112, 0xf>(__builtin_bit_cast(u32, sq)));
  sq += __builtin_bit_cast(float, dpp0<0x114, 0xf>(__builtin_bit_cast(u32, sq))); sq += __builtin_bit_cast(float, dpp0<0x118, 0xf>(__builtin_bit_cast(u32, sq)));
  sq += __builtin_bit_cast(float, dpp0<0x142, 0xa>(__builtin_bit_cast(u32, sq))); sq += __builtin_bit_cast(float, dpp0<0x143, 0xc>(__builtin_bit_cast(u32, sq)));
  const float t = 0.295f * __builtin_amdgcn_sqrtf(sq * __builtin_amdgcn_rcpf(n));
  return ((((u32)__builtin_amdgcn_readlane(__builtin_bit_cast(int, t), 63) >> 16) * 2 + 1023) & 0xFFFF);
}
DEV float sq2(u32 v, float s) { return __builtin_amdgcn_fdot2_f32_bf16(__builtin_bit_cast(bf16x2, v), __builtin_bit_cast(bf16x2, v), s, false); }
DEV void split4(u32 n0, u32 n1, u32& lo, u32& G, u32& idx, u32& len) {
  lo = perm(n1, n0, 0x06040200u); G = perm(n1, n0, 0x07050301u);
  idx = perm(pk_min(n1, 0x07FF07FFu), pk_min(n0, 0x07FF07FFu), 0x07050301u);
  len = G + perm(0x01000000u, 0x00010207u, idx) - 0x03030303u;
}
DEV void lds_or(LAS char* p, u32 v) { __hip_atomic_fetch_or((LAS u32*)p, v, __ATOMIC_RELAXED, __HIP_MEMORY_SCOPE_WORKGROUP); }
DEV void emit16(const u32* L, u32& pos, LAS char* st) {
  u32 w = 0, lp = 0;
#pragma unroll
  for (int i = 0; i < 16; ++i) { lp += (L[i >> 2] >> (8 * (i & 3))) & 255; w |= 1u << (lp & 31); }
  if (lp < 32) {
    const u64 W = (u64)w << (pos & 31); LAS char* p = st + ((pos >> 3) & ~3u);
    lds_or(p, (u32)W); lds_or(p + 4, (u32)(W >> 32));
  } else {
    u32 l0 = L[0], l1 = L[1], l2 = L[2], l3 = L[3], q = pos;
#pragma unroll 1
    for (int i = 0; i < 16; ++i) {
      q += l0 & 255; lds_or(st + ((q >> 3) & ~3u), 1u << (q & 31));
      l0 = __builtin_amdgcn_alignbyte(l1, l0, 1); l1 = __builtin_amdgcn_alignbyte(l2, l1, 1); l2 = __builtin_amdgcn_alignbyte(l3, l2, 1); l3 >>= 8;
    }
  }
  pos += lp;
}
#define USTEP(i) "v_ffbl_b32_e32 %1, %0\n\tv_add_u32_e32 %1, 1, %1\n\tv_lshrrev_b32_e32 %0, %1, %0\n\tv_min_u32_e32 %2, 5, %1\n\tv_perm_b32 %2, %5, %6, %2\n\t" \
                 "v_add_u32_e32 %2, %2, %1\n\tds_write_b8 %3, %2 offset:" #i "\n\tv_cmpx_ne_u32_e32 vcc, 0, %0\n\t"
#define USTEP2(i, k) USTEP(i) USTEP(k) "s_cbranch_execz 9f\n\t"
#define USTEP1(i) USTEP(i) "s_cbranch_execz 9f\n\t"
DEV void read_unary(u32 st, u32 U, u32 K, int lane) {
  const u32 np = U / 1920 + 1, cb = rfl((u32)((float)U * __builtin_amdgcn_rcpf((float)(64 * np)))) + 1;
  u32 last = 0, s = 1 + lane * cb;
  const LAS u32* p = (const LAS u32*)lp(st + ((s >> 3) & ~3u));
  u32 w0 = p[0], w1 = p[1];
  for (u32 s0 = 1; s0 <= U; s0 += 64 * cb) {
    const int left = (int)(U + 1 - s);
    u32 W = __builtin_amdgcn_ubfe(__builtin_amdgcn_alignbit(w1, w0, s), 0, left < 0 ? 0 : left > (int)cb ? cb : left);
    const u32 sn = s + 64 * cb;
    p = (const LAS u32*)lp(st + ((sn >> 3) & ~3u));
    w0 = p[0]; w1 = p[1];
    const u32 cnt = __builtin_popcount(W), inc = wave_scan(cnt);
    u32 e = W ? s + 31 - __builtin_clz(W) : 0;
    e = umax(e, dpp0<0x111, 0xf>(e)); e = umax(e, dpp0<0x112, 0xf>(e)); e = umax(e, dpp0<0x114, 0xf>(e)); e = umax(e, dpp0<0x118, 0xf>(e));
    e = umax(e, dpp0<0x142, 0xa>(e)); e = umax(e, dpp0<0x143, 0xc>(e));
    const u32 gap = s - 1 - umax(dpp0<0x138, 0xf>(e), last), k = K + inc - cnt;
    last = umax(last, (u32)__builtin_amdgcn_readlane((int)e, 63));
    K += (u32)__builtin_amdgcn_readlane((int)inc, 63);
    u32 t0, t1;
    asm volatile("v_cmpx_ne_u32_e32 vcc, 0, %0\n\tv_ffbl_b32_e32 %1, %0\n\tv_add_u32_e32 %1, 1, %1\n\tv_lshrrev_b32_e32 %0, %1, %0\n\tv_add_u32_e32 %1, %4, %1\n\t"
                 "v_min_u32_e32 %2, 5, %1\n\tv_perm_b32 %2, %5, %6, %2\n\tv_add_u32_e32 %2, %2, %1\n\tds_write_b8 %3, %2\n\tv_cmpx_ne_u32_e32 vcc, 0, %0\n\t"
                 USTEP2(1, 2) USTEP2(3, 4) USTEP2(5, 6) USTEP2(7, 8) USTEP2(9, 10) USTEP1(11) USTEP1(12) USTEP1(13) USTEP1(14) USTEP1(15) USTEP1(16)
                 USTEP1(17) USTEP1(18) USTEP1(19) USTEP1(20) USTEP2(21, 22) USTEP2(23, 24) USTEP2(25, 26) USTEP2(27, 28) USTEP(29) USTEP(30)
                 "9:\n\ts_mov_b64 exec, -1\n\ts_nop 1"
                 : "+v"(W), "=&v"(t0), "=&v"(t1) : "v"(k), "v"(gap), "s"(0x000002FCu), "v"(0x03030300u) : "vcc", "memory");
    s = sn;
  }
}
DEV u32 in_bytes(u32 U) { return 10240 + ((U >> 7) + 1) * 16; }
DEV u32 out_bytes(u32 G, u32 U) { return 1024 + 16 * G + ((((2 * G + 3) & ~3u) + ((U >> 5) + 1) * 4 + 15) & ~15u); }

DEV void prep_piece(const Args& a, int mode, bool isx, int s, int j, int h, int dest, u32 callv, char* wsp, LAS char* scr, int lane, int it) {
#define PST(k) do { } while (0)
  const int fr = lane & 15, fq = lane >> 4;
  const int piece = j * 2 + h;
  const u32 slot = (isx ? 0 : SLOT_W + dest * 256) + s * 16 + piece;
  u32 v[64];
  {
    const rsrc_t ri = mkrsrc(isx ? a.x : a.w, isx ? 4096u * 512u * 2u : 512u * 32768u * 2u);
    const u32 lv = isx ? fr * 1024 + fq * 16 : fq * 8 * 65536 + 16 * (4 * (fr & 3) + (fr >> 2));
    const u32 ls = isx ? (s * 256 + (piece & 1) * 128) * 1024 + (piece >> 1) * 128
                       : (piece >> 1) * 64 * 65536 + (dest * 4096 + s * 256 + (piece & 1) * 128) * 2;
    const u32 s8 = isx ? 65536 : 32 * 65536, s2 = isx ? 16384 : 2 * 65536, s1 = isx ? 64 : 65536;
    u32x4 in[16];
#pragma unroll
    for (int i = 0; i < 16; ++i) in[i] = __builtin_amdgcn_raw_buffer_load_b128(ri, lv, ls + (i >> 3) * s8 + ((i >> 1) & 3) * s2 + (i & 1) * s1, PL_AUX);
    __builtin_amdgcn_sched_barrier(0);
    if (isx) {
#pragma unroll
      for (int i = 0; i < 64; ++i) v[i] = in[i >> 2][i & 3];
    } else {
#pragma unroll
      for (int ks = 0; ks < 2; ++ks)
#pragma unroll
        for (int el = 0; el < 8; ++el)
#pragma unroll
          for (int d = 0; d < 4; ++d)
            v[4 * (el * 2 + ks) + d] = perm(in[ks * 8 + 2 * d + 1][el >> 1], in[ks * 8 + 2 * d][el >> 1], (el & 1) ? 0x07060302u : 0x05040100u);
    }
  }
  GAS u32* const own = (GAS u32*)(a.ws[a.rank] + WS_READY + ((isx ? 0 : 4) + (s >> 2)) * 9 * 128);
  const int c = (s < 8 ? (s & 3) + (isx ? 0 : 4) : s - 8) * 2 + h, m = s < 8 ? j * 2 + (s >> 2) : isx ? 16 + j : 24 + j;
  const u32 doff = WS_DESC + ((a.rank * 16 + c) * NIN + m) * 16;
  if (mode & 5) {
    char* const img = a.ws[mode & 4 ? dest : a.rank] + WS_IMG;
    const u32 so = (isx ? 0 : IMG_BYTES) + (u32)(s * 64 + (mode & 4 ? 7 + j * 7 + ((a.rank - dest) & 7) : j)) * 32768u + h * (isx ? 16384 : 8192);
    const rsrc_t rimg[2] = {mkrsrc(img + so, 8192), mkrsrc(img + so + (isx ? 8192 : 16384), 8192)};
#pragma unroll
    for (int i = 0; i < 16; ++i) bst128(rimg[i >> 3], lane * 16 + (i & 7) * 1024, (u32x4){v[4 * i], v[4 * i + 1], v[4 * i + 2], v[4 * i + 3]});
    if (!(mode & 2)) {
      PST(62);
      waitv<0>();
      PST(63);
      asm volatile("" : "+v"(callv), "+v"(wsp));
      if (mode & 4) { if (lane == dest) st_sys128(wsp + doff, (u32x4){callv + 1, 0, 0, callv + 1}); }
      else if (lane == 0) __hip_atomic_fetch_add(own, 1u, __ATOMIC_RELAXED, __HIP_MEMORY_SCOPE_AGENT);
      return;
    }
  }
  float sq = 0.f;
#pragma unroll
  for (int i = 0; i < 16; ++i) sq = sq2(v[4 * i + (i & 3)], sq);
  const u32 cc = pick_c(sq, 2048.f), cp = cc * 0x10001u;
  const rsrc_t rb = mkrsrc(a.ws[a.rank] + WS_BLOB + slot * 16384u, 16384u);
  const u32 vo = lane * 16;
  u32 H[32], L[32], P[8], sum = 0;
#pragma unroll
  for (int r = 0; r < 8; ++r) {
    u32x4 lo; P[r] = 0;
#pragma unroll
    for (int k = 0; k < 4; ++k) {
      const u32 v0 = v[8 * r + 2 * k], v1 = v[8 * r + 2 * k + 1]; u32 l, idx;
      split4(pk_sub(cp, __builtin_amdgcn_alignbit(v0, v0, 31)), pk_sub(cp, __builtin_amdgcn_alignbit(v1, v1, 31)), l, H[4 * r + k], idx, L[4 * r + k]);
      lo[k] = l; P[r] |= perm(0, 0x03020100u, idx) << (2 * k); sum = __builtin_amdgcn_sad_u8(L[4 * r + k], 0, sum);
    }
    bst128(rb, vo + r * 1024, lo);
  }
  const u32 inc = wave_scan(sum), U = (u32)__builtin_amdgcn_readlane((int)inc, 63);
  const bool raw = U > IN_UCAP || (a.flags & 1);
  if (raw) {
#pragma unroll
    for (int r = 0; r < 8; ++r) bst128(rb, vo + (8 + r) * 1024, (u32x4){H[4 * r], H[4 * r + 1], H[4 * r + 2], H[4 * r + 3]});
  } else {
    const u32 ub = in_bytes(U) - 10240;
    for (u32 i = vo; i <= ub; i += 1024) lds_w128(scr + i, (u32x4){0, 0, 0, 0});
    bst128(rb, vo + 8192, (u32x4){P[0], P[1], P[2], P[3]}); bst128(rb, vo + 9216, (u32x4){P[4], P[5], P[6], P[7]});
    cbar();
    u32 pos = inc - sum;
#pragma unroll
    for (int r = 0; r < 8; ++r) emit16(L + 4 * r, pos, scr);
    cbar();
    for (u32 i = vo; i < ub; i += 1024) bst128(rb, 10240 + i, lds_r128(scr + i));
    cbar();
  }
  PST(62);
  waitv<0>();
  PST(63);
  asm volatile("" : "+v"(callv), "+v"(wsp));
  const u32x4 d = {callv + 1, cc | (raw ? 0u : U << 16), raw ? 0x80000000u : 0u, callv + 1};
  if (lane < NR && lane != a.rank && (isx || lane == dest)) st_sys128(wsp + doff, d);
  if ((mode & 1) && lane == 0) __hip_atomic_fetch_add(own, 1u, __ATOMIC_RELAXED, __HIP_MEMORY_SCOPE_AGENT);
}

DEV u32 list_starts(u32x4 bits, u32 (&st)[4]) {
  u32 run = 0;
#pragma unroll
  for (int q = 0; q < 4; ++q) {
    const u32 c = __builtin_popcount(bits[q] >> 16) | (__builtin_popcount(bits[q] & 0xFFFF) << 16), inc = wave_scan(c);
    const u32 tot = (u32)__builtin_amdgcn_readlane((int)inc, 63), r1 = run + (tot & 0xFFFF);
    st[q] = (inc - c + (run | r1 << 16)) * 2;
    run = r1 + (tot >> 16);
  }
  return run;
}
DEV void pack_out(const Args& a, f32x4 (&acc)[2][2][4][2], u32 blob, u32 call, LAS char* lds, int wid, int lane) {
  LAS char* const reg = lds + wid * EREG;
  char* wsp = a.ws[(lane >> 2) & 7];
  u32 tk = draw(a.ws[a.rank] + WS_TICKET, (wid & 3) == 0);
  u32 o[64];
#pragma unroll
  for (int bj = 0; bj < 2; ++bj)
#pragma unroll
    for (int n = 0; n < 2; ++n)
#pragma unroll
      for (int j = 0; j < 4; ++j)
#pragma unroll
        for (int d = 0; d < 4; ++d) {
          const f32x2 y = {__builtin_elementwise_maximum(acc[d >> 1][bj][(d & 1) * 2][n][j], 0.f),
                           __builtin_elementwise_maximum(acc[d >> 1][bj][(d & 1) * 2 + 1][n][j], 0.f)};
          o[((bj * 2 + n) * 4 + j) * 4 + d] = __builtin_bit_cast(u32, __builtin_convertvector(y, bf16x2)) & 0x7FFF7FFFu;
        }
  u32x4 bits; float sq = 0.f;
#pragma unroll
  for (int q = 0; q < 4; ++q) {
    u32 m16[2];
#pragma unroll
    for (int h = 0; h < 2; ++h) {
      u32 t = 0;
#pragma unroll
      for (int i = 0; i < 8; ++i) t = (t << 2) | pk_min(o[q * 16 + h * 8 + i], 0x00010001u);
      m16[h] = ((t & 0xFFFF) << 1) | (t >> 16);
    }
    bits[q] = (m16[0] << 16) | m16[1];
  }
#pragma unroll
  for (int i = 0; i < 16; ++i) sq = sq2(o[4 * i + (i & 3)], sq);
#pragma unroll
  for (int r = 0; r < 2; ++r) lds_w128(reg + E_SCR + r * 1024 + lane * 16, (u32x4){0, 0, 0, 0});
  const u32 dl = (lane & 15) * 4 + (lane >> 4);
#pragma unroll
  for (int q = 0; q < 4; ++q) bits[q] = (u32)__builtin_amdgcn_ds_bpermute(((lane & 3) * 16 + (lane >> 2)) * 4, (int)bits[q]);
  const rsrc_t rb = mkrsrc(a.ws[a.rank] + WS_BLOB + (SLOT_O + blob) * 16384u, 16384u);
  bst128(rb, lane * 16, bits);
  u32 st[4];
  const u32 n = list_starts(bits, st);
  const u32 cc = pick_c(sq, 0.25f * (float)n);
#pragma unroll
  for (int q = 0; q < 4; ++q) st[q] = (u32)__builtin_amdgcn_ds_bpermute((int)(dl * 4), (int)st[q]);
#pragma unroll
  for (int h = 0; h < 8; ++h) {
    u32 pos = (u32)(size_t)reg + ((h & 1) ? st[h >> 1] >> 16 : st[h >> 1] & 0xFFFF);
#pragma unroll
    for (int i = 0; i < 8; ++i)
      asm volatile("v_cmpx_ne_u16_e32 vcc, 0, %1\n\tds_write_b16 %0, %1\n\tv_add_u32_e32 %0, 2, %0\n\ts_mov_b64 exec, -1\n\t"
                   "v_cmpx_lt_u32_e32 vcc, %2, %1\n\tds_write_b16_d16_hi %0, %1\n\tv_add_u32_e32 %0, 2, %0\n\ts_mov_b64 exec, -1"
                   : "+v"(pos) : "v"(o[h * 8 + i]), "s"(0xFFFFu) : "vcc", "memory");
  }
  if (lane < 16) *(LAS u16*)(reg + 2 * (n + lane)) = (u16)(((cc - 257) & 0xFFFF) >> 1);
  cbar();
  const u32 G = (n + 15) >> 4, ce = (cc - 1) * 0x10001u, ucap = (15296 - 18 * G) * 8;
  LAS char* const us = reg + E_SCR + ((2 * G + 3) & ~3u);
  u32 U = 0; bool raw = a.flags & 2;
  for (u32 g0 = 0; g0 < G && !raw; g0 += 64) {
    const u32 g = g0 + lane; const bool act = g < G;
    const u32x4 t0 = lds_r128(reg + 32 * g), t1 = lds_r128(reg + 32 * g + 16);
    u32 t[8] = {t0.x, t0.y, t0.z, t0.w, t1.x, t1.y, t1.z, t1.w}, L[4], x = 0, sum = 0; u32x4 lo;
#pragma unroll
    for (int k = 0; k < 4; ++k) {
      u32 l, h, idx;
      split4(pk_sub(ce, t[2 * k] << 1), pk_sub(ce, t[2 * k + 1] << 1), l, h, idx, L[k]);
      lo[k] = l | perm(0, 0x01000100u, idx); x |= perm(0, 0x01010000u, idx) << k; sum = __builtin_amdgcn_sad_u8(L[k], 0, sum);
    }
    if (!act) sum = 0;
    const u32 inc = wave_scan(sum);
    u32 pos = U + inc - sum;
    U += (u32)__builtin_amdgcn_readlane((int)inc, 63);
    raw = 9 * U > ucap;
    if (act && !raw) {
      bst128(rb, 1024 + 16 * g, lo);
      *(LAS u16*)(reg + E_SCR + 2 * g) = (u16)(((x << 20) | x) >> 16);
      emit16(L, pos, us);
    }
  }
  cbar();
  if (raw) {
    waitv<0>();
#pragma unroll
    for (int c = 0; c < 16; ++c) bst128(rb, c * 1024 + dl * 16, (u32x4){o[4 * c], o[4 * c + 1], o[4 * c + 2], o[4 * c + 3]});
  } else {
    const u32 eo = 1024 + 16 * G, eb = out_bytes(G, U) - eo;
    for (u32 i = lane * 16; i < eb; i += 1024) bst128(rb, eo + i, lds_r128(reg + E_SCR + i));
  }
  waitv<0>();
  tk = collect<0>(tk);
  asm volatile("" : "+v"(wsp));
  u32 old = 0;
  if (lane == 0) {
    *(LAS u32x2*)(lds + L_DSC + wid * 8) = (u32x2){n | (cc << 16), (raw ? 0x80000000u : U) | (blob << 16)};
    if ((wid & 3) == 0) *(LAS u32*)(lds + L_DSC + 72 + wid) = tk;
    cbar();
    old = __hip_atomic_fetch_add((LAS u32*)(lds + L_DSC + 64 + (wid & 4)), 1u, __ATOMIC_RELAXED, __HIP_MEMORY_SCOPE_WORKGROUP);
  }
  if ((rfl(old) & 3) == 3) {
    cbar();
    const u32x2 e = *(LAS u32x2*)(lds + L_DSC + ((wid & 4) + (lane & 3)) * 8);
    const u32 u = (*(LAS u32*)(lds + L_DSC + 72 + (wid & 4)) & 511) * 4 + (lane & 3);
    if (lane < 4 * NR && (lane >> 2) != a.rank) st_sys128(wsp + WS_DESCO + (a.rank * 2048 + u) * 16, (u32x4){call, e.x, e.y, call});
  }
  const u32 tile = blob >> 3, w8 = blob & 7;
  const u32 so = ((tile >> 4) * 256 + (w8 & 3) * 32) * 65536u + (a.rank * 4096 + (tile & 15) * 256 + (w8 >> 2) * 128) * 2u;
  const rsrc_t rout = mkrsrc(a.out + so, OUT_BYTES - so);
  const u32 vo = (lane & 15) * 65536u + (lane >> 4) * 16;
#pragma unroll
  for (int ch = 0; ch < 16; ++ch)
    __builtin_amdgcn_raw_buffer_store_b128((u32x4){o[4 * ch], o[4 * ch + 1], o[4 * ch + 2], o[4 * ch + 3]}, rout,
                                           vo + (ch & 3) * 64 + ((ch >> 3) * 128 + ((ch >> 2) & 1) * 16) * 65536u, 0, OWN_AUX);
}

DEV void wait_ready(const GAS u32* cp, u32 need, u64 t0, u32& bad) {
  while (!bad) {
    const u32 c = __hip_atomic_load(cp, __ATOMIC_RELAXED, __HIP_MEMORY_SCOPE_AGENT);
    if (__builtin_amdgcn_ballot_w64((int)(c - need) < 0) == 0) break;
    if (now() - t0 > DEADLINE) bad = 1;
    __builtin_amdgcn_s_sleep(4);
  }
  cbar();
}
DEV void gemm_role(const Args& a, LAS char* lds, int pos, int wid, u32 call, u64 t0) {
  char* const wsl = a.ws[a.rank];
  const int wr = wid >> 2, wc = wid & 3;
  u32 bad = 0;
  if (wid == 0) *(LAS u32x2*)(lds + L_DSC + 64) = (u32x2){0, 0};
  for (int round = 0; round < 2; ++round) {
    int tid = threadIdx.x; asm volatile("" : "+v"(tid));
    int lane = tid & 63;
    const int tile = a.tiles[round * 128 + pos], mt = tile >> 4, nt = tile & 15;
    const GAS u32* cp = (const GAS u32*)(wsl + WS_READY + ((lane & 1) ? 4 + (nt >> 2) : mt >> 2) * 9 * 128);
    if (wid == 0) wait_ready(cp, 64u * call, t0, bad);
    asm volatile("s_waitcnt lgkmcnt(0)" ::: "memory");
    __builtin_amdgcn_s_barrier();
    STAMP_AT(2 + round * 3);
    f32x4 acc[2][2][4][2];
#pragma unroll
    for (int i0 = 0; i0 < 2; ++i0)
#pragma unroll
      for (int i1 = 0; i1 < 2; ++i1)
#pragma unroll
        for (int m = 0; m < 4; ++m)
#pragma unroll
          for (int n = 0; n < 2; ++n) acc[i0][i1][m][n] = (f32x4){0.f, 0.f, 0.f, 0.f};
    bf16x8 At[4][2], B0[2][2], B1[2][2];
    const char* cA = wsl + WS_IMG + IMG_BYTES + (size_t)nt * (2u << 20);
    const char* cB = wsl + WS_IMG + (size_t)mt * (2u << 20);
    const u32 voff = tid * 16, ldsw = wid * 1024;
    const int aoff = wr * 8192 + lane * 16, boff = wc * 4096 + lane * 16;
    constexpr int HTB = 16384, KST = 32768, NT = 64;
#define SA(b, h) (((b) * 2 + (h)) * HTB)
#define SB(b, h) ((4 + (b) * 2 + (h)) * HTB)
#define STAGE(bufoff, gbase) do { _Pragma("unroll") for (int _i = 0; _i < 2; ++_i) \
    __builtin_amdgcn_global_load_lds((const GAS u32*)((gbase) + _i * 8192 + voff), (LAS u32*)(lds + (bufoff) + ldsw + _i * 8192), 16, 0, GL_AUX); } while (0)
#define LDA(dst, b, h) do { _Pragma("unroll") for (int m = 0; m < 4; ++m) _Pragma("unroll") for (int k = 0; k < 2; ++k) \
    dst[m][k] = *(const LAS bf16x8*)(lds + SA(b, h) + aoff + m * 2048 + k * 1024); } while (0)
#define LDB(dst, b, h) do { _Pragma("unroll") for (int n = 0; n < 2; ++n) _Pragma("unroll") for (int k = 0; k < 2; ++k) \
    dst[n][k] = *(const LAS bf16x8*)(lds + SB(b, h) + boff + n * 2048 + k * 1024); } while (0)
#define MMA(ai, bj, A_, B_) do { __builtin_amdgcn_s_setprio(1); _Pragma("unroll") for (int m = 0; m < 4; ++m) _Pragma("unroll") for (int n = 0; n < 2; ++n) \
    _Pragma("unroll") for (int k = 0; k < 2; ++k) acc[ai][bj][m][n] = __builtin_amdgcn_mfma_f32_16x16x32_bf16(A_[m][k], B_[n][k], acc[ai][bj][m][n], 0, 0, 0); \
    __builtin_amdgcn_s_setprio(0); } while (0)
#define WAIT_V(n) asm volatile("s_waitcnt vmcnt(" #n ")" ::: "memory")
#define WAIT_L(n) asm volatile("s_waitcnt lgkmcnt(" #n ")" ::: "memory")
#define BAR __builtin_amdgcn_s_barrier()
    STAGE(SB(0, 0), cB); STAGE(SA(0, 0), cA); STAGE(SB(0, 1), cB + HTB); STAGE(SA(0, 1), cA + HTB);
    if (wr == 1) BAR;
    WAIT_V(4); BAR;
    STAGE(SB(1, 0), cB + KST); STAGE(SA(1, 0), cA + KST); STAGE(SB(1, 1), cB + HTB + KST);
    WAIT_V(6); BAR;
    int tn = 5;
    for (int t = 0; t < NT; t += 2) {
      if (wid == 0 && tn < 61) {
        if (t >= tn) {
          const u32 c = lds_poll(LDS_BYTES - 32 + (lane & 1) * 4);
          STAMP_AT(16 + round * 16 + (tn / 7) * 2);
          if (__builtin_amdgcn_ballot_w64((int)(c - 56u * call) < 0) != 0) wait_ready(cp, 56u * call, t0, bad);
          STAMP_AT(17 + round * 16 + (tn / 7) * 2);
          tn += 7;
        } else if (t + 2 >= tn) {
          cp += 32;
          if (lane < 2) __builtin_amdgcn_global_load_lds(cp, (LAS u32*)(lds + LDS_BYTES - 32), 4, 0, 17);
        }
      }
      const bool last = (t == NT - 2);
      const char* a1 = cA + (size_t)(t + 1) * KST;
      const char* a2 = last ? cA : cA + (size_t)(t + 2) * KST; const char* b2 = last ? cB : cB + (size_t)(t + 2) * KST;
      const char* a3 = a2 + KST; const char* b3 = b2 + KST;
      LDB(B0, 0, 0); SCHED; LDA(At, 0, 0); STAGE(SA(1, 1), a1 + HTB);
      WAIT_L(8); BAR; WAIT_L(0); MMA(0, 0, At, B0); BAR; SCHED;
      LDB(B1, 0, 1); STAGE(SB(0, 0), b2);
      BAR; WAIT_L(0); MMA(0, 1, At, B1); BAR;
      LDA(At, 0, 1); STAGE(SA(0, 0), a2);
      BAR; WAIT_L(0); MMA(1, 0, At, B0); BAR; SCHED;
      STAGE(SB(0, 1), b2 + HTB);
      WAIT_V(6); BAR; MMA(1, 1, At, B1); BAR;
      LDB(B0, 1, 0); SCHED; LDA(At, 1, 0); STAGE(SA(0, 1), a2 + HTB);
      WAIT_L(8); BAR; WAIT_L(0); MMA(0, 0, At, B0); BAR; SCHED;
      LDB(B1, 1, 1); STAGE(SB(1, 0), b3);
      BAR; WAIT_L(0); MMA(0, 1, At, B1); BAR;
      LDA(At, 1, 1); STAGE(SA(1, 0), a3);
      BAR; WAIT_L(0); MMA(1, 0, At, B0); BAR; SCHED;
      STAGE(SB(1, 1), b3 + HTB);
      WAIT_V(6); BAR; MMA(1, 1, At, B1); BAR;
    }
    if (wid == 0) lds_post(LDS_BYTES - 16, bad);
    asm volatile("s_waitcnt vmcnt(0) lgkmcnt(0)" ::: "memory");
    if (wr == 0) BAR;
    BAR;
#undef SA
#undef SB
#undef STAGE
#undef LDA
#undef LDB
#undef MMA
    int tz = threadIdx.x; asm volatile("" : "+v"(tz));
    lane = tz & 63;
    STAMP_AT(3 + round * 3);
    if (rfl(lds_poll(LDS_BYTES - 16))) {
#pragma unroll
      for (int i = 0; i < 32; ++i) acc[i >> 4][(i >> 3) & 1][(i >> 1) & 3][i & 1] = (f32x4){__builtin_nanf(""), __builtin_nanf(""), __builtin_nanf(""), __builtin_nanf("")};
    }
    pack_out(a, acc, tile * 8 + wid, call, lds, wid, lane);
    STAMP_AT(4 + round * 3);
  }
}

DEV void loader_role(const Args& a, LAS char* lds, int peer, int lane, u64 t0) {
  const char* src = a.ws[peer] + WS_BLOB + lane * 16;
  u32 served = 0; int pend = -1; u32 pend_tag = 0;
  int done = 0;
  for (int left = NDEC; left > 0;) {
    const u32 r = lds_poll(L_REQ + (lane & 7) * 4);
    u32 key = ((lane & 7) < NDEC && (r & 255) != served) ? ((255 - (r & 255)) << 3 | (7 - (lane & 7))) + 1 : 0;
    key = umax(key, dpp0<0x111, 0xf>(key)); key = umax(key, dpp0<0x112, 0xf>(key)); key = umax(key, dpp0<0x114, 0xf>(key));
    key = (u32)__builtin_amdgcn_readlane((int)key, 7);
    if (key == 0) {
      if (pend >= 0) { waitv<0>(); lds_post(L_FULL + pend * 4, pend_tag); pend = -1; }
      else { if (now() - t0 > DEADLINE + 20000000ull) break; __builtin_amdgcn_s_sleep(2); }
      continue;
    }
    const int j = 7 - ((key - 1) & 7);
    const u32 rq = (u32)__builtin_amdgcn_readlane((int)r, j);
    if ((lane & 7) == j) served = rq & 255;
    if (rq >> 31) { --left; continue; }
    ++done;
    const u32 slot = umin(rq >> 18, NSLOT - 1), bytes = (((rq >> 8) & 1023) + 1) * 16;
    const char* s = src + (size_t)slot * 16384;
    LAS char* dst = lds + L_DEC + j * DREG + BOFF;
    int row = 0;
    for (u32 o = 0; o < bytes; o += 1024, ++row) {
      if (o + lane * 16 < bytes) __builtin_amdgcn_global_load_lds((const GAS u32*)(s + o), (LAS u32*)(dst + o), 16, 0, LD_AUX);
      waitv<WIN - 1>();
      if (row == WIN - 2 && pend >= 0) { lds_post(L_FULL + pend * 4, pend_tag); pend = -1; }
    }
    if (pend >= 0) { waitv<0>(); lds_post(L_FULL + pend * 4, pend_tag); }
    pend = j; pend_tag = rq & 255;
  }
  waitv<0>();
  if (pend >= 0) lds_post(L_FULL + pend * 4, pend_tag);
}

template <int NB> DEV void out_groups(u32 A, u32 B, u32 pl, u32 g0, u32 G, u32& k0, u32 cp, int lane) {
  u32x4 lo[NB]; u32 x[NB], e[NB][4], k[NB][5], sel[NB][4], d0[NB][4], d1[NB][4];
#pragma unroll
  for (int b = 0; b < NB; ++b) { x[b] = *(LAS u16*)lp(pl + 2 * (g0 + lane) + 128 * b); lo[b] = lds_r128(lp(B + 1024 + 16 * (g0 + lane) + 1024 * b)); }
  SCHED;
#pragma unroll
  for (int b = 0; b < NB; ++b) {
    x[b] = (x[b] << 16) | (x[b] >> 4); k[b][0] = 0;
#pragma unroll
    for (int i = 0; i < 4; ++i) { e[b][i] = ~((x[b] >> i) | lo[b][i]) & 0x01010101u; k[b][i + 1] = k[b][i] + __builtin_popcount(e[b][i]); }
  }
  u32 k1[NB];
#pragma unroll
  for (int b = 0; b < NB; b += 2) {
    const u32 c = NB > 1 ? k[b][4] | (k[(b + 1) % NB][4] << 16) : k[b][4], inc = wave_scan(c), tot = (u32)__builtin_amdgcn_readlane((int)inc, 63);
    k1[b] = k0 + ((inc - c) & 0xFFFF); k0 += tot & 0xFFFF;
    if (NB > 1) { k1[(b + 1) % NB] = k0 + ((inc - c) >> 16); k0 += tot >> 16; }
  }
  auto issue = [&](int b) {
#pragma unroll
    for (int i = 0; i < 4; ++i) {
      k[b][i] += k1[b];
      sel[b][i] = *(LAS u32*)lp(__builtin_amdgcn_udot4(e[b][i], 0x20100804u, L_LUT2, false));
      const LAS u32* q = (const LAS u32*)lp(k[b][i] & ~3u); d0[b][i] = q[0]; d1[b][i] = q[1];
    }
  };
  issue(0);
#pragma unroll
  for (int b = 0; b < NB; ++b) {
    if (b + 1 < NB) issue(b + 1);
    SCHED;
    u32 v[8];
#pragma unroll
    for (int i = 0; i < 4; ++i) {
      const u32 h = perm((lo[b][i] & 0x01010101u) | ((i ? x[b] >> (i - 1) : x[b] << 1) & 0x02020202u), __builtin_amdgcn_alignbyte(d1[b][i], d0[b][i], k[b][i]), sel[b][i]);
      v[2 * i] = pk_shr1(pk_sub(cp, perm(h, lo[b][i], 0x05010400u))); v[2 * i + 1] = pk_shr1(pk_sub(cp, perm(h, lo[b][i], 0x07030602u)));
    }
    const u32 g = umin(g0 + lane + 64 * b, G);
    lds_w128(lp(A + 32 * g), (u32x4){v[0], v[1], v[2], v[3]}); lds_w128(lp(A + 32 * g + 16), (u32x4){v[4], v[5], v[6], v[7]});
    SCHED;
  }
}
DEV void decoder_role(const Args& a, int peer, int c, int j, int lane, u32 call, u64 t0) {
  char* const wsl = a.ws[a.rank];
  const u32 A = L_DEC + j * DREG, B = A + BOFF;
  const int q = (peer - a.rank) & 7;
  bool fail = false, posted = false, more = true;
  u32 tag = 1;
  const char* const queue = wsl + WS_QUEUE + peer * 128;
  const char* const dsco = wsl + WS_DESCO + (size_t)peer * 32768;
  const u32 qbase = (call - 1) * QSTEP;
  const char* dsc = wsl + WS_DESC + (size_t)((peer * 16 + (c & 15)) * NIN + j) * 16;
  const char* ndsc = dsc + NDEC * 16;
  u32 da = 0, db = 0, nda = 0, ndb = 0;
  int m = j;
  if (c >= 16) {
    m = NIN;
    const u32 u0 = collect<0>(draw(queue, true)) - qbase, u1 = collect<0>(draw(queue, u0 < 2048)) - qbase;
    dsc = dsco + (u0 & 2047) * 16; ndsc = dsco + (u1 & 2047) * 16; more = u1 < 2048;
    if (u0 >= 2048) m = -1;
  }
  auto post = [&](int m, u32 da, u32 db, u32 tag) {
    u32 slot, bytes;
    if (m < NIN) {
      const int kj = m < 16 ? m >> 1 : m & 7, t = m < 16 ? c >> 3 : (m >> 3) & 1, s = m < 16 ? (m & 1) * 4 + ((c >> 1) & 3) : 8 + (c >> 1);
      slot = (t ? SLOT_W + a.rank * 256 : 0) + s * 16 + kj * 2 + (c & 1);
      bytes = in_bytes(umin(da >> 16, IN_UCAP));
    } else {
      const u32 G = (umin(da & 0xFFFF, 8192) + 15) >> 4;
      slot = SLOT_O + ((db >> 16) & 2047);
      bytes = out_bytes(G, umin(db & 0xFFFF, OUT_UCAP));
    }
    if (db >> 31) bytes = 16384;
    asm volatile("s_waitcnt lgkmcnt(0)" ::: "memory");
    lds_post(L_REQ + j * 4, (slot << 18) | ((bytes / 16 - 1) << 8) | tag);
  };
  auto peek = [&](u32x4 d) {
    u32 c0, c1;
    asm volatile("s_waitcnt vmcnt(0)\n\tv_readfirstlane_b32 %0, %4\n\tv_readfirstlane_b32 %1, %5\n\tv_readfirstlane_b32 %2, %6\n\tv_readfirstlane_b32 %3, %7"
                 : "=s"(c0), "=s"(nda), "=s"(ndb), "=s"(c1) : "v"(d.x), "v"(d.y), "v"(d.z), "v"(d.w) : "memory");
    return more && c0 == call && c1 == call;
  };
  for (; m >= 0; m += NDEC) {
    if (!posted) {
      u32x4 d;
      for (;;) {
        d = ld_sys128(dsc);
        if (rfl(d.x) == call && rfl(d.w) == call) break;
        if (now() - t0 > DEADLINE) { fail = true; break; }
        __builtin_amdgcn_s_sleep(4);
      }
      if (fail) break;
      da = rfl(d.y); db = rfl(d.z);
      if (m) post(m, da, db, tag);
    }
    if (tag <= 11) STAMP_AT(12 + tag * 4);
    u32x4 nd; asm volatile("global_load_dwordx4 %0, %1, %2 sc0 sc1" : "=&v"(nd) : "v"(0), "s"(ndsc) : "memory");
    const bool drawn = more && m + 2 * NDEC >= NIN;
    u32 u = 0;
    const u32 ntag = tag == 255 ? 1 : tag + 1;
    const bool raw = db >> 31;
    const int kj = m < 16 ? m >> 1 : m & 7, t = m < 16 ? c >> 3 : (m >> 3) & 1, s = m < 16 ? (m & 1) * 4 + ((c >> 1) & 3) : 8 + (c >> 1);
    const u32 U = m < NIN ? umin(da >> 16, IN_UCAP) : umin(db & 0xFFFF, OUT_UCAP), G = (umin(da & 0xFFFF, 8192) + 15) >> 4;
    while (m) {
      if (rfl(lds_poll(L_FULL + j * 4)) == tag) break;
      if (now() - t0 > DEADLINE) { fail = true; break; }
      __builtin_amdgcn_s_sleep(2);
    }
    if (fail) { peek(nd); break; }
    posted = false;
    if (tag <= 11) STAMP_AT(13 + tag * 4);
    if (m == 0) {
      if (lane == 0) __hip_atomic_fetch_add((GAS u32*)(wsl + WS_READY + (t * 4 * 9 + 1) * 128), 1u, __ATOMIC_RELAXED, __HIP_MEMORY_SCOPE_AGENT);
      peek(nd);
    } else if (m < NIN) {
      const u32 cp = (da & 0xFFFF) * 0x10001u;
      u32 H[32];
      if (raw) {
#pragma unroll
        for (int r = 0; r < 8; ++r) { const u32x4 t = lds_r128(lp(B + (8 + r) * 1024 + lane * 16)); H[4 * r] = t.x; H[4 * r + 1] = t.y; H[4 * r + 2] = t.z; H[4 * r + 3] = t.w; }
      } else {
        const u32x4 p0 = lds_r128(lp(B + 8192 + lane * 16)), p1 = lds_r128(lp(B + 9216 + lane * 16));
        read_unary(B + 10240, U, A, lane);
        const u32 P[8] = {p0.x, p0.y, p0.z, p0.w, p1.x, p1.y, p1.z, p1.w};
        u32 E[8], cnt = 0, k[33], sel[32], d0[32], d1[32];
#pragma unroll
        for (int r = 0; r < 8; ++r) { E[r] = ~(P[r] | (P[r] >> 1)) & 0x55555555u; cnt += __builtin_popcount(E[r]); }
        k[0] = A + (wave_scan(cnt) - cnt);
        auto issue = [&](int r) {
#pragma unroll
          for (int i = 4 * r; i < 4 * r + 4; ++i) {
            const u32 e = (E[r] >> (i & 2) * 2) & (0x01010101u << (i & 1) * 2);
            sel[i] = *(LAS u32*)lp(__builtin_amdgcn_udot4(e, (i & 1) ? 0x08040201u : 0x20100804u, L_LUT2, false));
            const LAS u32* w = (const LAS u32*)lp(k[i] & ~3u); d0[i] = w[0]; d1[i] = w[1];
            k[i + 1] = k[i] + __builtin_popcount(e);
          }
        };
        issue(0);
#pragma unroll
        for (int r = 0; r < 8; ++r) {
          if (r < 7) issue(r + 1);
          SCHED;
#pragma unroll
          for (int i = 4 * r; i < 4 * r + 4; ++i) H[i] = perm((P[r] >> (i & 3) * 2) & 0x03030303u, __builtin_amdgcn_alignbyte(d1[i], d0[i], k[i]), sel[i]);
          SCHED;
        }
      }
      u32x4 lo[8];
#pragma unroll
      for (int r = 0; r < 8; ++r) lo[r] = lds_r128(lp(B + r * 1024 + lane * 16));
      if (peek(nd)) { post(m + NDEC, nda, ndb, ntag); posted = true; }
      const u32 tk = draw(queue, drawn);
      const u32 so = t * IMG_BYTES + (u32)(s * 64 + 7 + kj * 7 + q) * 32768u + (c & 1) * (t ? 8192 : 16384);
      const rsrc_t rimg[2] = {mkrsrc(wsl + WS_IMG + so, 8192), mkrsrc(wsl + WS_IMG + so + (t ? 16384 : 8192), 8192)};
#pragma unroll
      for (int r = 0; r < 8; ++r) {
        u32 v[8];
#pragma unroll
        for (int k = 0; k < 4; ++k) {
          const u32 u0 = pk_sub(cp, perm(H[4 * r + k], lo[r][k], 0x05010400u)), u1 = pk_sub(cp, perm(H[4 * r + k], lo[r][k], 0x07030602u));
          v[2 * k] = __builtin_amdgcn_alignbit(u0, u0, 1); v[2 * k + 1] = __builtin_amdgcn_alignbit(u1, u1, 1);
        }
        ist128(rimg[r >> 2], lane * 16 + ((2 * r) & 7) * 1024, (u32x4){v[0], v[1], v[2], v[3]});
        ist128(rimg[r >> 2], lane * 16 + ((2 * r + 1) & 7) * 1024, (u32x4){v[4], v[5], v[6], v[7]});
      }
      if (tag <= 11) STAMP_AT(14 + tag * 4);
      u = collect<0>(tk);
      if (tag <= 11) STAMP_AT(15 + tag * 4);
      if (lane == 0) __hip_atomic_fetch_add((GAS u32*)(wsl + WS_READY + ((t * 4 + (s >> 2)) * 9 + 1 + kj) * 128), 1u, __ATOMIC_RELAXED, __HIP_MEMORY_SCOPE_AGENT);
    } else {
      const u32 blob = (db >> 16) & 2047, tile = blob >> 3, w8 = blob & 7;
      const u32 so = ((tile >> 4) * 256 + (w8 & 3) * 32) * 65536u + (peer * 4096 + (tile & 15) * 256 + (w8 >> 2) * 128) * 2u;
      const u32 vo = (lane >> 2) * 65536u + (lane & 3) * 16;
      const rsrc_t rout = mkrsrc(a.out + so, OUT_BYTES - so);
      bool nok;
      if (raw) {
        nok = peek(nd);
        const u32 tk = draw(queue, drawn);
#pragma unroll
        for (int ch = 0; ch < 16; ++ch)
          __builtin_amdgcn_raw_buffer_store_b128(lds_r128(lp(B + ch * 1024 + lane * 16)), rout, vo + (ch & 3) * 64 + ((ch >> 3) * 128 + ((ch >> 2) & 1) * 16) * 65536u, 0, OUT_AUX);
        u = collect<16>(tk);
      } else {
        const u32 cp = (da >> 16) * 0x10001u;
        const u32x4 bits = lds_r128(lp(B + lane * 16));
        const u32 pl = B + 1024 + 16 * G;
        u32 k0 = B + out_bytes(G, U), g0 = 0;
        read_unary(pl + ((2 * G + 3) & ~3u), U, k0, lane);
        for (; g0 + 128 < G; g0 += 256) out_groups<4>(A, B, pl, g0, G, k0, cp, lane);
        for (; g0 < G; g0 += 64) out_groups<1>(A, B, pl, g0, G, k0, cp, lane);
        nok = peek(nd);
        if (nok && 32 * G + 32 <= BOFF) { post(m + NDEC, nda, ndb, ntag); posted = true; }
        const u32 tk = draw(queue, drawn);
        u32 st[4], pos[8][4], d[8][4][3]; u32x2 sel[8][4];
        list_starts(bits, st);
        auto issue = [&](int h) {
          const u32 n0 = (bits[h >> 1] >> 1) & 0x78787878u, n1 = (bits[h >> 1] << 3) & 0x78787878u;
          const u32 p0 = A + ((h & 1) ? st[h >> 1] >> 16 : st[h >> 1] & 0xFFFF);
          u32 cnt = 0;
#pragma unroll
          for (int i = 0; i < 4; ++i) {
            const u32 n8 = (((i & 1) ? n1 : n0) >> (24 - 16 * (h & 1) - 8 * (i >> 1))) & 255;
            pos[h][i] = p0 + 2 * cnt; cnt += __builtin_popcount(n8);
            sel[h][i] = *(LAS u32x2*)lp(L_LUT + n8);
            const LAS u32* w = (const LAS u32*)lp(pos[h][i] & ~3u);
            d[h][i][0] = w[0]; d[h][i][1] = w[1]; d[h][i][2] = w[2];
          }
        };
        issue(0);
#pragma unroll
        for (int h = 0; h < 8; ++h) {
          if (h < 7) issue(h + 1);
          SCHED;
#pragma unroll
          for (int i = 0; i < 2; ++i) {
            u32x4 ov; const int ch = 2 * h + i;
#pragma unroll
            for (int n = 2 * i; n < 2 * i + 2; ++n) {
              const u32 e0 = __builtin_amdgcn_alignbyte(d[h][n][1], d[h][n][0], pos[h][n]), e1 = __builtin_amdgcn_alignbyte(d[h][n][2], d[h][n][1], pos[h][n]);
              ov[2 * (n & 1)] = perm(e1, e0, sel[h][n].x); ov[2 * (n & 1) + 1] = perm(e1, e0, sel[h][n].y);
            }
            __builtin_amdgcn_raw_buffer_store_b128(ov, rout, vo + (ch & 3) * 64 + ((ch >> 3) * 128 + ((ch >> 2) & 1) * 16) * 65536u, 0, OUT_AUX);
          }
          SCHED;
        }
        u = collect<16>(tk);
      }
      if (nok && !posted) { post(m + NDEC, nda, ndb, ntag); posted = true; }
      if (tag <= 11) STAMP_AT(14 + tag * 4);
    }
    tag = ntag; da = nda; db = ndb;
    if (!more) break;
    dsc = ndsc; ndsc += NDEC * 16;
    if (drawn) { u -= qbase; more = u < 2048; ndsc = dsco + (u & 2047) * 16; }
  }
  if (fail) tag = tag == 255 ? 1 : tag + 1;
  lds_post(L_REQ + j * 4, 0x80000000u | tag);
  waitv<0>();
  if (fail) {
    __builtin_amdgcn_raw_buffer_store_b32(0x7FC07FC0u, mkrsrc(a.out, OUT_BYTES), lane * 4 + (blockIdx.x * 8 + j) * 65536u, 0, 0);
    if (lane == 0) __hip_atomic_store((GAS u32*)(wsl + WS_TMO), call, __ATOMIC_RELAXED, __HIP_MEMORY_SCOPE_AGENT);
  }
}

__global__ void __launch_bounds__(512, 2) fused_gemm_ar(const Args a) {
  extern __shared__ __attribute__((aligned(16))) char smem[];
  LAS char* lds = (LAS char*)smem;
  const u64 t0 = now();
  const int wid = __builtin_amdgcn_readfirstlane(threadIdx.x >> 6), lane = threadIdx.x & 63;
  const int xcd = vbid() & 7, li = vbid() >> 3;
  char* const wsl = a.ws[a.rank];
  GAS u32* callp = (GAS u32*)(wsl + WS_CALL) + blockIdx.x * 8 + wid;
  const u32 callv = __hip_atomic_load(callp, __ATOMIC_RELAXED, __HIP_MEMORY_SCOPE_AGENT);
  char* const wsp = a.ws[lane & 7];
  STAMP_AT(0);
  const bool is_gemm = li < 16;
  const int pos = xcd * 16 + (li & 15);
  if (!is_gemm) {
    if (wid == 0) {
      if (lane < 32) *(LAS u32*)(lds + L_REQ + lane * 4) = 0;
      if (lane < 16) {
        u32 k = 0, sel[2] = {0, 0};
#pragma unroll
        for (int i = 0; i < 4; ++i) {
          const bool on = (lane >> (3 - i)) & 1;
          sel[i >> 1] |= (on ? (2 * k) | ((2 * k + 1) << 8) : 0x0C0Cu) << (16 * (i & 1));
          k += on;
        }
        *(LAS u32x2*)(lds + L_LUT + lane * 8) = (u32x2){sel[0], sel[1]};
        u32 s2 = 0; k = 0;
#pragma unroll
        for (int i = 0; i < 4; ++i) { const bool on = (lane >> i) & 1; s2 |= (on ? k : 4 + i) << (8 * i); k += on; }
        *(LAS u32*)(lds + L_LUT2 + lane * 4) = s2;
      }
    }
    asm volatile("s_waitcnt lgkmcnt(0)" ::: "memory");
    __builtin_amdgcn_s_barrier();
  }
  const int q = li - 16;
  const bool is_help = !is_gemm && (q & 7) == 0;
  {
    int e = -1, n = 1;
    const bool early = is_gemm && (xcd & 1);
    if (early) { if (wid < 4) e = 2176 + wid * 64 + (xcd >> 1) * 16 + li; }
    else if (is_gemm) { e = 512 + wid * 64 + (xcd >> 1) * 16 + li; n = 3; }
    else if (is_help) { e = 1024 + (xcd * 2 + (q >> 3)) * 8 + wid; n = 2; }
    else if (wid >= 4) e = 64 + (wid - 4) * 112 + (xcd * 2 + (q >> 3)) * 7 + (q & 7) - 1;
    else if (wid == 1) e = xcd * 2 + (q >> 3);
    LAS char* scr = is_gemm ? lds + wid * EREG : wid == 7 ? lds + L_SCR7 : wid == 0 ? lds + L_DEC : lds + L_DEC + (wid - 1) * DREG + BOFF;
    if (GDELAY && n > 1) for (int i = 0; i < GDELAY; ++i) __builtin_amdgcn_s_sleep(37);
    if (e >= 0)
      for (int i = 0; i < n && (e < 2176 || n == 1); ++i, e += 640) {
        bool isx; int j, s, dq, mode;
        if (e < 64) { j = 0; isx = e < 8; s = (e >> 1) & 3; dq = -q; mode = 4; }
        else if (e < 1024) { j = e >> 7; dq = (e >> 3) & 7; isx = dq == 0; s = ((e >> 6) & 1) * 4 + ((e >> 1) & 3); mode = 2; }
        else if (e < 1152) { j = (e >> 4) & 7; isx = true; s = 8 + ((e >> 1) & 7); dq = 0; mode = 3; }
        else if (e < 2176) { j = (e - 1152) >> 7; isx = false; s = 8 + ((e >> 1) & 7); dq = (e >> 4) & 7; mode = dq ? 2 : 1; }
        else { j = (e >> 4) & 7; isx = e < 2304; s = (e >> 1) & 7; dq = 0; mode = 1; }
        prep_piece(a, mode, isx, s, j, e & 1, (a.rank + dq) & 7, callv, wsp, scr, lane, i);
        if (is_help) STAMP_AT(16 + i);
      }
  }
  const u32 call = rfl(callv) + 1;
  if (lane == 0) __hip_atomic_store(callp, call, __ATOMIC_RELAXED, __HIP_MEMORY_SCOPE_AGENT);
  STAMP_AT(1);
  if (is_gemm) {
    gemm_role(a, lds, pos, wid, call, t0);
  } else if (!is_help || (xcd && (q >> 3) < NHELP)) {
    const int peer = (a.rank + (is_help ? xcd : q & 7)) & 7, c = is_help ? 16 : xcd * 2 + (q >> 3);
    if (wid == 0) loader_role(a, lds, peer, lane, t0);
    else if (wid <= NDEC) decoder_role(a, peer, c, wid - 1, lane, call, t0);
  }
  STAMP_AT(8);
}

#define HIP_CHECK(expr) do { hipError_t _e = (expr); if (_e != hipSuccess) { \
    fprintf(stderr, "HIP error %s at %s:%d\n", hipGetErrorString(_e), __FILE__, __LINE__); \
    return; } } while (0)

#define DBG_FLAGS 0

extern "C" void kernel_launch(int rank, int nranks, ncclComm_t comm, hipStream_t stream,
                              void* const* d_in, const int* in_sizes, int n_in,
                              void* d_out, int out_size, const int* in_dtypes, int out_dtype,
                              void* const* d_out_all, void* const* d_ws_all, size_t ws_bytes) {
  if (nranks != NR || (unsigned)rank >= (unsigned)NR || n_in < 2 || ws_bytes < WS_USED || (size_t)out_size * 2 != OUT_BYTES ||
      in_sizes[0] != 4096 * 512 || in_sizes[1] != 512 * 32768) {
    fprintf(stderr, "unexpected problem shape\n");
    return;
  }
  static bool attr_set[NR] = {};
  if (!attr_set[rank]) {
    HIP_CHECK(hipFuncSetAttribute((const void*)fused_gemm_ar, hipFuncAttributeMaxDynamicSharedMemorySize, LDS_BYTES));
    attr_set[rank] = true;
  }
  Args a;
  a.x = (const char*)d_in[0]; a.w = (const char*)d_in[1]; a.out = (char*)d_out;
  for (int p = 0; p < NR; ++p) a.ws[p] = (char*)d_ws_all[p];
  a.rank = rank; a.flags = DBG_FLAGS;
  static const int blk[8][2] = {{2, 0}, {0, 0}, {2, 1}, {1, 0}, {3, 0}, {0, 1}, {3, 1}, {1, 1}};
  for (int k = 0; k < 256; ++k) {
    const int b = (k >> 4) & 7, i = k & 15;
    a.tiles[k] = (unsigned char)((blk[b][0] * 4 + (i >> 2)) * 16 + (blk[b][1] + (k >> 7) * 2) * 4 + (i & 3));
  }
  hipLaunchKernelGGL(fused_gemm_ar, dim3(256), dim3(512), LDS_BYTES, stream, a);
  HIP_CHECK(hipGetLastError());
}
```

```cpp
#include <hip/hip_runtime.h>
#include <rccl/rccl.h>
#include <cstdio>

typedef unsigned u32;
typedef unsigned long long u64;
typedef u32 u32x4 __attribute__((ext_vector_type(4)));
typedef u32 u32x2 __attribute__((ext_vector_type(2)));
typedef unsigned short u16;
typedef u16 u16x2 __attribute__((ext_vector_type(2)));
typedef short bf16x8 __attribute__((ext_vector_type(8)));
typedef float f32x4 __attribute__((ext_vector_type(4)));
typedef float f32x2 __attribute__((ext_vector_type(2)));
typedef __bf16 bf16x2 __attribute__((ext_vector_type(2)));
#define LAS __attribute__((address_space(3)))
#define GAS __attribute__((address_space(1)))
#define DEV __device__ __forceinline__

constexpr int NR = 8;
constexpr u32 WS_CALL = 0;
constexpr u32 WS_TMO = 8192;
constexpr u32 WS_TICKET = 8192 + 128;
constexpr u32 WS_QUEUE = 8192 + 256;
constexpr u32 WS_READY = 16384;
constexpr int NIN = 32;
constexpr u32 WS_DESC = 512u << 10;
constexpr u32 WS_DESCO = WS_DESC + 65536;
constexpr u32 WS_IMG = 1u << 20;
constexpr u32 IMG_BYTES = 32u << 20;
constexpr u32 WS_BLOB = 65u << 20;
constexpr u32 SLOT_W = 256, SLOT_O = 2304, NSLOT = 4352;
constexpr u32 WS_PART = WS_BLOB + NSLOT * 16384u;
constexpr u32 WS_PFLAG = 32768;
constexpr u32 WS_USED = WS_PART + (4u << 20);
constexpr u32 WS_STAMP = WS_USED;
constexpr u32 OUT_BYTES = 4096u * 32768u * 2u;
constexpr u64 DEADLINE = 100000000ull;
constexpr int LDS_BYTES = 163840;
constexpr int NDEC = 6, NPUSH = 2;
constexpr int DREG = 26624, BOFF = 10240;
constexpr int L_REQ = 0, L_FULL = 64, L_LUT = 128, L_LUT2 = 256;
constexpr int L_DEC = 512, L_SCR7 = L_DEC + NDEC * DREG;
constexpr int EREG = 20480;
constexpr int E_SCR = 16448;
constexpr int L_DSC = LDS_BYTES - 256;
constexpr u32 IN_UCAP = 28000, OUT_UCAP = 13600;
#define NHELP 2
constexpr u32 QSTEP = 2048 + (16 + NHELP) * NDEC;
#define WIN 4
#define PWIN 4
#define PW1 2
#define D2 20
#define NEARLY 0
#define GDELAY 8
#define OWN_AUX 17
#define OUT_AUX 17
#define IMG_AUX 17
#define GL_AUX 17
#define PL_AUX 2
#define SEG 0
DEV int vbid() {
  const int x = blockIdx.x & 7, l = blockIdx.x >> 3;
  if (SEG == 0) return blockIdx.x;
  if (SEG == 9) return blockIdx.x ^ 128;
  const int comm = SEG == 1 ? (x & 1) ^ 1 : SEG == 2 ? x & 1 : SEG == 3 ? x >> 2 : (x >> 1) & 1;
  const int g = SEG <= 2 ? x >> 1 : SEG == 3 ? x & 3 : (x & 1) | ((x >> 2) << 1);
  return comm * 128 + (l & 15) * 8 + g * 2 + (l >> 4);
}
#define LD_AUX 17

struct Args {
  const char* x; const char* w; char* wsl; int rank; int flags; char* out; char* ws[NR]; unsigned char tiles[256];
};

template <int N> DEV void waitv() { asm volatile("s_waitcnt vmcnt(%0)" ::"n"(N) : "memory"); }
DEV void cbar() { asm volatile("" ::: "memory"); }
DEV u64 now() { return __builtin_amdgcn_s_memrealtime(); }
DEV u32 perm(u32 hi, u32 lo, u32 sel) { return __builtin_amdgcn_perm(hi, lo, sel); }
DEV u32 pk_sub(u32 a, u32 b) { return __builtin_bit_cast(u32, __builtin_bit_cast(u16x2, a) - __builtin_bit_cast(u16x2, b)); }
DEV u32 pk_shr1(u32 a) { return __builtin_bit_cast(u32, __builtin_bit_cast(u16x2, a) >> (u16x2){1, 1}); }
DEV u32 pk_min(u32 a, u32 b) { return __builtin_bit_cast(u32, __builtin_elementwise_min(__builtin_bit_cast(u16x2, a), __builtin_bit_cast(u16x2, b))); }
template <int C, int RM> DEV u32 dpp0(u32 x) { return (u32)__builtin_amdgcn_update_dpp(0, (int)x, C, RM, 0xf, false); }
DEV u32 umax(u32 a, u32 b) { return a > b ? a : b; }
DEV u32 umin(u32 a, u32 b) { return a < b ? a : b; }
DEV u32 wave_scan(u32 x) {
  x += dpp0<0x111, 0xf>(x); x += dpp0<0x112, 0xf>(x); x += dpp0<0x114, 0xf>(x); x += dpp0<0x118, 0xf>(x);
  x += dpp0<0x142, 0xa>(x); x += dpp0<0x143, 0xc>(x);
  return x;
}
DEV u32 rfl(u32 x) { return (u32)__builtin_amdgcn_readfirstlane((int)x); }
DEV u32x4 lds_r128(LAS char* p) { return *(LAS u32x4*)p; }
DEV void lds_w128(LAS char* p, u32x4 v) { *(LAS u32x4*)p = v; }
DEV u32 lds_poll(u32 addr) { u32 v; asm volatile("ds_read_b32 %0, %1\n\ts_waitcnt lgkmcnt(0)" : "=v"(v) : "v"(addr) : "memory"); return v; }
DEV void lds_post(u32 addr, u32 v) { asm volatile("ds_write_b32 %0, %1" ::"v"(addr), "v"(v) : "memory"); }
DEV void st_sys128(void* p, u32x4 v) { asm volatile("global_store_dwordx4 %0, %1, off sc0 sc1\n\ts_nop 1" ::"v"(p), "v"(v) : "memory"); }
DEV u32x4 ld_sys128(const void* p) { u32x4 v; asm volatile("global_load_dwordx4 %0, %1, off sc0 sc1\n\ts_waitcnt vmcnt(0)" : "=&v"(v) : "v"(p) : "memory"); return v; }
DEV LAS char* lp(u32 addr) { return (LAS char*)(size_t)addr; }
DEV u32 draw(const void* p, bool on) {
  u32 u; asm volatile("s_mov_b32 exec_lo, %4\n\ts_mov_b32 exec_hi, 0\n\tglobal_atomic_add %0, %1, %2, %3 sc0\n\ts_mov_b64 exec, -1" : "=&v"(u) : "v"(0), "v"(1), "s"(p), "s"(rfl(on)) : "memory");
  return u;
}
template <int N> DEV u32 collect(u32 u) { u32 r; asm volatile("s_waitcnt vmcnt(%2)\n\tv_readfirstlane_b32 %0, %1" : "=s"(r) : "v"(u), "n"(N) : "memory"); return r; }
#define SCHED __builtin_amdgcn_sched_barrier(0)
typedef __amdgpu_buffer_rsrc_t rsrc_t;
DEV rsrc_t mkrsrc(const void* p, u32 bytes) { return __builtin_amdgcn_make_buffer_rsrc((void*)p, 0, (int)bytes, 0x00020000); }
DEV void bst128(rsrc_t r, u32 voff, u32x4 v) { __builtin_amdgcn_raw_buffer_store_b128(v, r, (int)voff, 0, 17); }
DEV void ist128(rsrc_t r, u32 voff, u32x4 v) { __builtin_amdgcn_raw_buffer_store_b128(v, r, (int)voff, 0, IMG_AUX); }

#define STAMP_AT(k) do { } while (0)

DEV u32 pick_c(float sq, float n) {
  sq += __builtin_bit_cast(float, dpp0<0x111, 0xf>(__builtin_bit_cast(u32, sq))); sq += __builtin_bit_cast(float, dpp0<0x112, 0xf>(__builtin_bit_cast(u32, sq)));
  sq += __builtin_bit_cast(float, dpp0<0x114, 0xf>(__builtin_bit_cast(u32, sq))); sq += __builtin_bit_cast(float, dpp0<0x118, 0xf>(__builtin_bit_cast(u32, sq)));
  sq += __builtin_bit_cast(float, dpp0<0x142, 0xa>(__builtin_bit_cast(u32, sq))); sq += __builtin_bit_cast(float, dpp0<0x143, 0xc>(__builtin_bit_cast(u32, sq)));
  const float t = 0.295f * __builtin_amdgcn_sqrtf(sq * __builtin_amdgcn_rcpf(n));
  return ((((u32)__builtin_amdgcn_readlane(__builtin_bit_cast(int, t), 63) >> 16) * 2 + 1023) & 0xFFFF);
}
DEV float sq2(u32 v, float s) { return __builtin_amdgcn_fdot2_f32_bf16(__builtin_bit_cast(bf16x2, v), __builtin_bit_cast(bf16x2, v), s, false); }
DEV void split4(u32 n0, u32 n1, u32& lo, u32& G, u32& idx, u32& len) {
  lo = perm(n1, n0, 0x06040200u); G = perm(n1, n0, 0x07050301u);
  idx = perm(pk_min(n1, 0x07FF07FFu), pk_min(n0, 0x07FF07FFu), 0x07050301u);
  len = G + perm(0x01000000u, 0x00010207u, idx) - 0x03030303u;
}
DEV void lds_or(LAS char* p, u32 v) { __hip_atomic_fetch_or((LAS u32*)p, v, __ATOMIC_RELAXED, __HIP_MEMORY_SCOPE_WORKGROUP); }
DEV void emit16(const u32* L, u32& pos, LAS char* st) {
  u32 w = 0, lp = 0;
#pragma unroll
  for (int i = 0; i < 16; ++i) { lp += (L[i >> 2] >> (8 * (i & 3))) & 255; w |= 1u << (lp & 31); }
  if (lp < 32) {
    const u64 W = (u64)w << (pos & 31); LAS char* p = st + ((pos >> 3) & ~3u);
    lds_or(p, (u32)W); lds_or(p + 4, (u32)(W >> 32));
  } else {
    u32 l0 = L[0], l1 = L[1], l2 = L[2], l3 = L[3], q = pos;
#pragma unroll 1
    for (int i = 0; i < 16; ++i) {
      q += l0 & 255; lds_or(st + ((q >> 3) & ~3u), 1u << (q & 31));
      l0 = __builtin_amdgcn_alignbyte(l1, l0, 1); l1 = __builtin_amdgcn_alignbyte(l2, l1, 1); l2 = __builtin_amdgcn_alignbyte(l3, l2, 1); l3 >>= 8;
    }
  }
  pos += lp;
}
#define USTEP(i) "v_ffbl_b32_e32 %1, %0\n\tv_add_u32_e32 %1, 1, %1\n\tv_lshrrev_b32_e32 %0, %1, %0\n\tv_min_u32_e32 %2, 5, %1\n\tv_perm_b32 %2, %5, %6, %2\n\t" \
                 "v_add_u32_e32 %2, %2, %1\n\tds_write_b8 %3, %2 offset:" #i "\n\tv_cmpx_ne_u32_e32 vcc, 0, %0\n\t"
#define USTEP2(i, k) USTEP(i) USTEP(k) "s_cbranch_execz 9f\n\t"
#define USTEP1(i) USTEP(i) "s_cbranch_execz 9f\n\t"
DEV void read_unary(u32 st, u32 U, u32 K, int lane) {
  const u32 np = U / 1920 + 1, cb = rfl((u32)((float)U * __builtin_amdgcn_rcpf((float)(64 * np)))) + 1;
  u32 last = 0, s = 1 + lane * cb;
  const LAS u32* p = (const LAS u32*)lp(st + ((s >> 3) & ~3u));
  u32 w0 = p[0], w1 = p[1];
  for (u32 s0 = 1; s0 <= U; s0 += 64 * cb) {
    const int left = (int)(U + 1 - s);
    u32 W = __builtin_amdgcn_ubfe(__builtin_amdgcn_alignbit(w1, w0, s), 0, left < 0 ? 0 : left > (int)cb ? cb : left);
    const u32 sn = s + 64 * cb;
    p = (const LAS u32*)lp(st + ((sn >> 3) & ~3u));
    w0 = p[0]; w1 = p[1];
    const u32 cnt = __builtin_popcount(W), inc = wave_scan(cnt);
    u32 e = W ? s + 31 - __builtin_clz(W) : 0;
    e = umax(e, dpp0<0x111, 0xf>(e)); e = umax(e, dpp0<0x112, 0xf>(e)); e = umax(e, dpp0<0x114, 0xf>(e)); e = umax(e, dpp0<0x118, 0xf>(e));
    e = umax(e, dpp0<0x142, 0xa>(e)); e = umax(e, dpp0<0x143, 0xc>(e));
    const u32 gap = s - 1 - umax(dpp0<0x138, 0xf>(e), last), k = K + inc - cnt;
    last = umax(last, (u32)__builtin_amdgcn_readlane((int)e, 63));
    K += (u32)__builtin_amdgcn_readlane((int)inc, 63);
    u32 t0, t1;
    asm volatile("v_cmpx_ne_u32_e32 vcc, 0, %0\n\tv_ffbl_b32_e32 %1, %0\n\tv_add_u32_e32 %1, 1, %1\n\tv_lshrrev_b32_e32 %0, %1, %0\n\tv_add_u32_e32 %1, %4, %1\n\t"
                 "v_min_u32_e32 %2, 5, %1\n\tv_perm_b32 %2, %5, %6, %2\n\tv_add_u32_e32 %2, %2, %1\n\tds_write_b8 %3, %2\n\tv_cmpx_ne_u32_e32 vcc, 0, %0\n\t"
                 USTEP2(1, 2) USTEP2(3, 4) USTEP2(5, 6) USTEP2(7, 8) USTEP2(9, 10) USTEP1(11) USTEP1(12) USTEP1(13) USTEP1(14) USTEP1(15) USTEP1(16)
                 USTEP1(17) USTEP1(18) USTEP1(19) USTEP1(20) USTEP2(21, 22) USTEP2(23, 24) USTEP2(25, 26) USTEP2(27, 28) USTEP(29) USTEP(30)
                 "9:\n\ts_mov_b64 exec, -1\n\ts_nop 1"
                 : "+v"(W), "=&v"(t0), "=&v"(t1) : "v"(k), "v"(gap), "s"(0x000002FCu), "v"(0x03030300u) : "vcc", "memory");
    s = sn;
  }
}
DEV u32 in_bytes(u32 U) { return 10240 + ((U >> 7) + 1) * 16; }
DEV u32 out_bytes(u32 G, u32 U) { return 1024 + 16 * G + ((((2 * G + 3) & ~3u) + ((U >> 5) + 1) * 4 + 15) & ~15u); }

struct Head { const char* x; const char* w; char* wsl; int rank; int flags; };
DEV void prep_piece(const Head& a, bool bar, int mode, bool isx, int s, int j, int h, int dest, u32 callv, char* wsp, LAS char* scr, int lane, int it) {
#define PST(k) do { } while (0)
  const int fr = lane & 15, fq = lane >> 4;
  const int piece = j * 2 + h;
  u32 v[64];
  {
    const rsrc_t ri = mkrsrc(isx ? a.x : a.w, isx ? 4096u * 512u * 2u : 512u * 32768u * 2u);
    const u32 lv = isx ? fr * 1024 + fq * 16 : fq * 8 * 65536 + 16 * (4 * (fr & 3) + (fr >> 2));
    const u32 ls = isx ? (s * 256 + (piece & 1) * 128) * 1024 + (piece >> 1) * 128
                       : (piece >> 1) * 64 * 65536 + (dest * 4096 + s * 256 + (piece & 1) * 128) * 2;
    const u32 s8 = isx ? 65536 : 32 * 65536, s2 = isx ? 16384 : 2 * 65536, s1 = isx ? 64 : 65536;
    u32x4 in[16];
#pragma unroll
    for (int i = 0; i < 16; ++i) in[i] = __builtin_amdgcn_raw_buffer_load_b128(ri, lv, ls + (i >> 3) * s8 + ((i >> 1) & 3) * s2 + (i & 1) * s1, PL_AUX);
    if (bar) __builtin_amdgcn_s_barrier();
    __builtin_amdgcn_sched_barrier(0);
    if (isx) {
#pragma unroll
      for (int i = 0; i < 64; ++i) v[i] = in[i >> 2][i & 3];
    } else {
#pragma unroll
      for (int ks = 0; ks < 2; ++ks)
#pragma unroll
        for (int el = 0; el < 8; ++el)
#pragma unroll
          for (int d = 0; d < 4; ++d)
            v[4 * (el * 2 + ks) + d] = perm(in[ks * 8 + 2 * d + 1][el >> 1], in[ks * 8 + 2 * d][el >> 1], (el & 1) ? 0x07060302u : 0x05040100u);
    }
  }
  GAS u32* const own = (GAS u32*)(a.wsl + WS_READY + ((isx ? 0 : 4) + (s >> 2)) * 9 * 128);
  const int c = (s < 8 ? (s & 3) + (isx ? 0 : 4) : s - 8) * 2 + h, m = s < 8 ? j * 2 + (s >> 2) : isx ? 16 + j : 24 + j;
  const u32 doff = WS_DESC + ((a.rank * 16 + c) * NIN + m) * 16;
  if (mode & 1) {
    char* const img = a.wsl + WS_IMG;
    const u32 so = (isx ? 0 : IMG_BYTES) + (u32)(s * 64 + j) * 32768u + h * (isx ? 16384 : 8192);
    const rsrc_t rimg[2] = {mkrsrc(img + so, 8192), mkrsrc(img + so + (isx ? 8192 : 16384), 8192)};
#pragma unroll
    for (int i = 0; i < 16; ++i) bst128(rimg[i >> 3], lane * 16 + (i & 7) * 1024, (u32x4){v[4 * i], v[4 * i + 1], v[4 * i + 2], v[4 * i + 3]});
    if (!(mode & 2)) {
      PST(62);
      waitv<0>();
      PST(63);
      if (lane == 0) __hip_atomic_fetch_add(own, 1u, __ATOMIC_RELAXED, __HIP_MEMORY_SCOPE_AGENT);
      return;
    }
  }
  float sq = 0.f;
#pragma unroll
  for (int i = 0; i < 16; ++i) sq = sq2(v[4 * i + (i & 3)], sq);
  const u32 cc = pick_c(sq, 2048.f), cp = cc * 0x10001u;
  const bool push = mode & 4;
  const u32 slot = push ? SLOT_W + dest * 256 + a.rank * 32 + c * 2 + m : (isx ? 0 : SLOT_W + dest * 256) + s * 16 + piece;
  const int tr = push ? dest : a.rank;
  const rsrc_t rb = mkrsrc((char*)((u64)(u32)__builtin_amdgcn_readlane((int)((u64)wsp >> 32), tr) << 32 | (u32)__builtin_amdgcn_readlane((int)(u64)wsp, tr)) + WS_BLOB + slot * 16384u, 16384u);
  const u32 vo = lane * 16;
  auto put = [&](u32 off, u32x4 d, int w = 1) { if (push && w) { if (w == 2 && m) waitv<PW1 - 1>(); else waitv<PWIN - 1>(); } bst128(rb, off, d); };
  u32 H[32], L[32], P[8], sum = 0; u32x4 lo[8];
#pragma unroll
  for (int r = 0; r < 8; ++r) {
    P[r] = 0;
#pragma unroll
    for (int k = 0; k < 4; ++k) {
      const u32 v0 = v[8 * r + 2 * k], v1 = v[8 * r + 2 * k + 1]; u32 l, idx;
      split4(pk_sub(cp, __builtin_amdgcn_alignbit(v0, v0, 31)), pk_sub(cp, __builtin_amdgcn_alignbit(v1, v1, 31)), l, H[4 * r + k], idx, L[4 * r + k]);
      lo[r][k] = l; P[r] |= perm(0, 0x03020100u, idx) << (2 * k); sum = __builtin_amdgcn_sad_u8(L[4 * r + k], 0, sum);
    }
    if (r < 2) put(vo + r * 1024, lo[r], 2);
    if (r == 4 || r == 7) put(vo + (r / 3 + 1) * 1024, lo[r / 3 + 1], 2);
  }
  const u32 inc = wave_scan(sum), U = (u32)__builtin_amdgcn_readlane((int)inc, 63);
  const bool raw = U > IN_UCAP || (a.flags & 1);
  if (raw) {
#pragma unroll
    for (int r = 0; r < 8; ++r) put(vo + (8 + r) * 1024, (u32x4){H[4 * r], H[4 * r + 1], H[4 * r + 2], H[4 * r + 3]});
#pragma unroll
    for (int r = 4; r < 8; ++r) put(vo + r * 1024, lo[r]);
  } else {
    const u32 ub = in_bytes(U) - 10240;
    for (u32 i = vo; i <= ub; i += 1024) lds_w128(scr + i, (u32x4){0, 0, 0, 0});
    cbar();
    u32 pos = inc - sum;
#pragma unroll
    for (int r = 0; r < 8; ++r) { emit16(L + 4 * r, pos, scr); if (r & 1) put(vo + (4 + r / 2) * 1024, lo[4 + r / 2], r < NEARLY ? 2 : 1); }
    put(vo + 8192, (u32x4){P[0], P[1], P[2], P[3]}); put(vo + 9216, (u32x4){P[4], P[5], P[6], P[7]});
    cbar();
    for (u32 i = vo; i < ub; i += 1024) put(10240 + i, lds_r128(scr + i), 0);
    cbar();
  }
  PST(62);
  waitv<0>();
  PST(63);
  asm volatile("" : "+v"(callv), "+v"(wsp));
  const u32x4 d = {callv + 1, cc | (raw ? 0u : U << 16), raw ? 0x80000000u : 0u, callv + 1};
  if (lane < NR && lane != a.rank && ((isx && !push) || lane == dest)) st_sys128(wsp + doff, d);
  if ((mode & 1) && lane == 0) __hip_atomic_fetch_add(own, 1u, __ATOMIC_RELAXED, __HIP_MEMORY_SCOPE_AGENT);
}

DEV u32 list_starts(u32x4 bits, u32 (&st)[4]) {
  u32 run = 0;
#pragma unroll
  for (int q = 0; q < 4; ++q) {
    const u32 c = __builtin_popcount(bits[q] >> 16) | (__builtin_popcount(bits[q] & 0xFFFF) << 16), inc = wave_scan(c);
    const u32 tot = (u32)__builtin_amdgcn_readlane((int)inc, 63), r1 = run + (tot & 0xFFFF);
    st[q] = (inc - c + (run | r1 << 16)) * 2;
    run = r1 + (tot >> 16);
  }
  return run;
}
DEV void pack_out(const Args& a, f32x4 (&acc)[2][2][4][2], u32 blob, u32 call, LAS char* lds, int wid, int lane) {
  LAS char* const reg = lds + wid * EREG;
  char* wsp = a.ws[(lane >> 2) & 7];
  u32 tk = draw(a.wsl + WS_TICKET, (wid & 3) == 0);
  u32 o[64];
#pragma unroll
  for (int bj = 0; bj < 2; ++bj)
#pragma unroll
    for (int n = 0; n < 2; ++n)
#pragma unroll
      for (int j = 0; j < 4; ++j)
#pragma unroll
        for (int d = 0; d < 4; ++d) {
          const f32x2 y = {__builtin_elementwise_maximum(acc[d >> 1][bj][(d & 1) * 2][n][j], 0.f),
                           __builtin_elementwise_maximum(acc[d >> 1][bj][(d & 1) * 2 + 1][n][j], 0.f)};
          o[((bj * 2 + n) * 4 + j) * 4 + d] = __builtin_bit_cast(u32, __builtin_convertvector(y, bf16x2)) & 0x7FFF7FFFu;
        }
  u32x4 bits; float sq = 0.f;
#pragma unroll
  for (int q = 0; q < 4; ++q) {
    u32 m16[2];
#pragma unroll
    for (int h = 0; h < 2; ++h) {
      u32 t = 0;
#pragma unroll
      for (int i = 0; i < 8; ++i) t = (t << 2) | pk_min(o[q * 16 + h * 8 + i], 0x00010001u);
      m16[h] = ((t & 0xFFFF) << 1) | (t >> 16);
    }
    bits[q] = (m16[0] << 16) | m16[1];
  }
#pragma unroll
  for (int i = 0; i < 16; ++i) sq = sq2(o[4 * i + (i & 3)], sq);
#pragma unroll
  for (int r = 0; r < 2; ++r) lds_w128(reg + E_SCR + r * 1024 + lane * 16, (u32x4){0, 0, 0, 0});
  const u32 dl = (lane & 15) * 4 + (lane >> 4);
#pragma unroll
  for (int q = 0; q < 4; ++q) bits[q] = (u32)__builtin_amdgcn_ds_bpermute(((lane & 3) * 16 + (lane >> 2)) * 4, (int)bits[q]);
  const rsrc_t rb = mkrsrc(a.wsl + WS_BLOB + (SLOT_O + blob) * 16384u, 16384u);
  bst128(rb, lane * 16, bits);
  u32 st[4];
  const u32 n = list_starts(bits, st);
  const u32 cc = pick_c(sq, 0.25f * (float)n);
#pragma unroll
  for (int q = 0; q < 4; ++q) st[q] = (u32)__builtin_amdgcn_ds_bpermute((int)(dl * 4), (int)st[q]);
#pragma unroll
  for (int h = 0; h < 8; ++h) {
    u32 pos = (u32)(size_t)reg + ((h & 1) ? st[h >> 1] >> 16 : st[h >> 1] & 0xFFFF);
#pragma unroll
    for (int i = 0; i < 8; ++i)
      asm volatile("v_cmpx_ne_u16_e32 vcc, 0, %1\n\tds_write_b16 %0, %1\n\tv_add_u32_e32 %0, 2, %0\n\ts_mov_b64 exec, -1\n\t"
                   "v_cmpx_lt_u32_e32 vcc, %2, %1\n\tds_write_b16_d16_hi %0, %1\n\tv_add_u32_e32 %0, 2, %0\n\ts_mov_b64 exec, -1"
                   : "+v"(pos) : "v"(o[h * 8 + i]), "s"(0xFFFFu) : "vcc", "memory");
  }
  if (lane < 16) *(LAS u16*)(reg + 2 * (n + lane)) = (u16)(((cc - 257) & 0xFFFF) >> 1);
  cbar();
  const u32 G = (n + 15) >> 4, ce = (cc - 1) * 0x10001u, ucap = (15296 - 18 * G) * 8;
  LAS char* const us = reg + E_SCR + ((2 * G + 3) & ~3u);
  u32 U = 0; bool raw = a.flags & 2;
  for (u32 g0 = 0; g0 < G && !raw; g0 += 64) {
    const u32 g = g0 + lane; const bool act = g < G;
    const u32x4 t0 = lds_r128(reg + 32 * g), t1 = lds_r128(reg + 32 * g + 16);
    u32 t[8] = {t0.x, t0.y, t0.z, t0.w, t1.x, t1.y, t1.z, t1.w}, L[4], x = 0, sum = 0; u32x4 lo;
#pragma unroll
    for (int k = 0; k < 4; ++k) {
      u32 l, h, idx;
      split4(pk_sub(ce, t[2 * k] << 1), pk_sub(ce, t[2 * k + 1] << 1), l, h, idx, L[k]);
      lo[k] = l | perm(0, 0x01000100u, idx); x |= perm(0, 0x01010000u, idx) << k; sum = __builtin_amdgcn_sad_u8(L[k], 0, sum);
    }
    if (!act) sum = 0;
    const u32 inc = wave_scan(sum);
    u32 pos = U + inc - sum;
    U += (u32)__builtin_amdgcn_readlane((int)inc, 63);
    raw = 9 * U > ucap;
    if (act && !raw) {
      bst128(rb, 1024 + 16 * g, lo);
      *(LAS u16*)(reg + E_SCR + 2 * g) = (u16)(((x << 20) | x) >> 16);
      emit16(L, pos, us);
    }
  }
  cbar();
  if (raw) {
    waitv<0>();
#pragma unroll
    for (int c = 0; c < 16; ++c) bst128(rb, c * 1024 + dl * 16, (u32x4){o[4 * c], o[4 * c + 1], o[4 * c + 2], o[4 * c + 3]});
  } else {
    const u32 eo = 1024 + 16 * G, eb = out_bytes(G, U) - eo;
    for (u32 i = lane * 16; i < eb; i += 1024) bst128(rb, eo + i, lds_r128(reg + E_SCR + i));
  }
  waitv<0>();
  tk = collect<0>(tk);
  asm volatile("" : "+v"(wsp));
  u32 old = 0;
  if (lane == 0) {
    *(LAS u32x2*)(lds + L_DSC + wid * 8) = (u32x2){n | (cc << 16), (raw ? 0x80000000u : U) | (blob << 16)};
    if ((wid & 3) == 0) *(LAS u32*)(lds + L_DSC + 72 + wid) = tk;
    cbar();
    old = __hip_atomic_fetch_add((LAS u32*)(lds + L_DSC + 64 + (wid & 4)), 1u, __ATOMIC_RELAXED, __HIP_MEMORY_SCOPE_WORKGROUP);
  }
  if ((rfl(old) & 3) == 3) {
    cbar();
    const u32x2 e = *(LAS u32x2*)(lds + L_DSC + ((wid & 4) + (lane & 3)) * 8);
    const u32 u = (*(LAS u32*)(lds + L_DSC + 72 + (wid & 4)) & 511) * 4 + (lane & 3);
    if (lane < 4 * NR && (lane >> 2) != a.rank) st_sys128(wsp + WS_DESCO + (a.rank * 2048 + u) * 16, (u32x4){call, e.x, e.y, call});
  }
  const u32 tile = blob >> 3, w8 = blob & 7;
  const u32 so = ((tile >> 4) * 256 + (w8 & 3) * 32) * 65536u + (a.rank * 4096 + (tile & 15) * 256 + (w8 >> 2) * 128) * 2u;
  const rsrc_t rout = mkrsrc(a.out + so, OUT_BYTES - so);
  const u32 vo = (lane & 15) * 65536u + (lane >> 4) * 16;
#pragma unroll
  for (int ch = 0; ch < 16; ++ch)
    __builtin_amdgcn_raw_buffer_store_b128((u32x4){o[4 * ch], o[4 * ch + 1], o[4 * ch + 2], o[4 * ch + 3]}, rout,
                                           vo + (ch & 3) * 64 + ((ch >> 3) * 128 + ((ch >> 2) & 1) * 16) * 65536u, 0, OWN_AUX);
}

DEV void wait_ready(const GAS u32* cp, u32 need, u64 t0, u32& bad) {
  while (!bad) {
    const u32 c = __hip_atomic_load(cp, __ATOMIC_RELAXED, __HIP_MEMORY_SCOPE_AGENT);
    if (__builtin_amdgcn_ballot_w64((int)(c - need) < 0) == 0) break;
    if (now() - t0 > DEADLINE) bad = 1;
    __builtin_amdgcn_s_sleep(4);
  }
  cbar();
}
#define KSTEP 6
constexpr int KSPLIT = 8 + 7 * KSTEP;
DEV void gemm_role(const Args& a, LAS char* lds, int pos, bool help, int wid, u32 call, u64 t0) {
  char* const wsl = a.wsl;
  const int wr = wid >> 2, wc = wid & 3;
  u32 bad = 0;
  if (wid == 0) *(LAS u32x2*)(lds + L_DSC + 64) = (u32x2){0, 0};
  for (int round = 0; round < 2; ++round) {
    int tid = threadIdx.x; asm volatile("" : "+v"(tid));
    int lane = tid & 63;
    const int tile = a.tiles[round * 128 + pos], mt = tile >> 4, nt = tile & 15;
    const bool split = round == 0 && (pos >> 4) == 1;
    const int tb = help ? KSPLIT : 0, te = split && !help ? KSPLIT : 64;
    const GAS u32* cp = (const GAS u32*)(wsl + WS_READY + ((lane & 1) ? 4 + (nt >> 2) : mt >> 2) * 9 * 128) + (help ? (KSTEP + 1) * 32 : 0);
    if (wid == 0) wait_ready(cp, (help ? 56u : 64u) * call, t0, bad);
    asm volatile("s_waitcnt lgkmcnt(0)" ::: "memory");
    __builtin_amdgcn_s_barrier();
    STAMP_AT(2 + round * 3);
    f32x4 acc[2][2][4][2];
#pragma unroll
    for (int i0 = 0; i0 < 2; ++i0)
#pragma unroll
      for (int i1 = 0; i1 < 2; ++i1)
#pragma unroll
        for (int m = 0; m < 4; ++m)
#pragma unroll
          for (int n = 0; n < 2; ++n) acc[i0][i1][m][n] = (f32x4){0.f, 0.f, 0.f, 0.f};
    bf16x8 At[4][2], B0[2][2], B1[2][2];
    const char* cA = wsl + WS_IMG + IMG_BYTES + (size_t)nt * (2u << 20);
    const char* cB = wsl + WS_IMG + (size_t)mt * (2u << 20);
    const u32 voff = tid * 16, ldsw = wid * 1024;
    const int aoff = wr * 8192 + lane * 16, boff = wc * 4096 + lane * 16;
    constexpr int HTB = 16384, KST = 32768;
#define SA(b, h) (((b) * 2 + (h)) * HTB)
#define SB(b, h) ((4 + (b) * 2 + (h)) * HTB)
#define STAGE(bufoff, gbase) do { _Pragma("unroll") for (int _i = 0; _i < 2; ++_i) \
    __builtin_amdgcn_global_load_lds((const GAS u32*)((gbase) + _i * 8192 + voff), (LAS u32*)(lds + (bufoff) + ldsw + _i * 8192), 16, 0, GL_AUX); } while (0)
#define LDA(dst, b, h) do { _Pragma("unroll") for (int m = 0; m < 4; ++m) _Pragma("unroll") for (int k = 0; k < 2; ++k) \
    dst[m][k] = *(const LAS bf16x8*)(lds + SA(b, h) + aoff + m * 2048 + k * 1024); } while (0)
#define LDB(dst, b, h) do { _Pragma("unroll") for (int n = 0; n < 2; ++n) _Pragma("unroll") for (int k = 0; k < 2; ++k) \
    dst[n][k] = *(const LAS bf16x8*)(lds + SB(b, h) + boff + n * 2048 + k * 1024); } while (0)
#define MMA(ai, bj, A_, B_) do { __builtin_amdgcn_s_setprio(1); _Pragma("unroll") for (int m = 0; m < 4; ++m) _Pragma("unroll") for (int n = 0; n < 2; ++n) \
    _Pragma("unroll") for (int k = 0; k < 2; ++k) acc[ai][bj][m][n] = __builtin_amdgcn_mfma_f32_16x16x32_bf16(A_[m][k], B_[n][k], acc[ai][bj][m][n], 0, 0, 0); \
    __builtin_amdgcn_s_setprio(0); } while (0)
#define WAIT_V(n) asm volatile("s_waitcnt vmcnt(" #n ")" ::: "memory")
#define WAIT_L(n) asm volatile("s_waitcnt lgkmcnt(" #n ")" ::: "memory")
#define BAR __builtin_amdgcn_s_barrier()
    const char* a0 = cA + (size_t)tb * KST; const char* b0 = cB + (size_t)tb * KST;
    STAGE(SB(0, 0), b0); STAGE(SA(0, 0), a0); STAGE(SB(0, 1), b0 + HTB); STAGE(SA(0, 1), a0 + HTB);
    if (wr == 1) BAR;
    WAIT_V(4); BAR;
    STAGE(SB(1, 0), b0 + KST); STAGE(SA(1, 0), a0 + KST); STAGE(SB(1, 1), b0 + HTB + KST);
    WAIT_V(6); BAR;
    int tn = help ? KSPLIT + 4 : 5;
    for (int t = tb; t < te; t += 2) {
      if (wid == 0 && tn < te - 3) {
        if (t >= tn) {
          const u32 c = lds_poll(LDS_BYTES - 32 + (lane & 1) * 4);
          STAMP_AT(16 + round * 16 + (tn / 7) * 2);
          if (__builtin_amdgcn_ballot_w64((int)(c - 56u * call) < 0) != 0) wait_ready(cp, 56u * call, t0, bad);
          STAMP_AT(17 + round * 16 + (tn / 7) * 2);
          tn += 7;
        } else if (t + 2 >= tn) {
          cp += 32;
          if (lane < 2) __builtin_amdgcn_global_load_lds(cp, (LAS u32*)(lds + LDS_BYTES - 32), 4, 0, 17);
        }
      }
      const bool last = (t == te - 2);
      const char* a1 = cA + (size_t)(t + 1) * KST;
      const char* a2 = cA + (size_t)(last ? t : t + 2) * KST; const char* b2 = cB + (size_t)(last ? t : t + 2) * KST;
      const char* a3 = a2 + KST; const char* b3 = b2 + KST;
      LDB(B0, 0, 0); SCHED; LDA(At, 0, 0); STAGE(SA(1, 1), a1 + HTB);
      WAIT_L(8); BAR; WAIT_L(0); MMA(0, 0, At, B0); BAR; SCHED;
      LDB(B1, 0, 1); STAGE(SB(0, 0), b2);
      BAR; WAIT_L(0); MMA(0, 1, At, B1); BAR;
      LDA(At, 0, 1); STAGE(SA(0, 0), a2);
      BAR; WAIT_L(0); MMA(1, 0, At, B0); BAR; SCHED;
      STAGE(SB(0, 1), b2 + HTB);
      WAIT_V(6); BAR; MMA(1, 1, At, B1); BAR;
      LDB(B0, 1, 0); SCHED; LDA(At, 1, 0); STAGE(SA(0, 1), a2 + HTB);
      WAIT_L(8); BAR; WAIT_L(0); MMA(0, 0, At, B0); BAR; SCHED;
      LDB(B1, 1, 1); STAGE(SB(1, 0), b3);
      BAR; WAIT_L(0); MMA(0, 1, At, B1); BAR;
      LDA(At, 1, 1); STAGE(SA(1, 0), a3);
      BAR; WAIT_L(0); MMA(1, 0, At, B0); BAR; SCHED;
      STAGE(SB(1, 1), b3 + HTB);
      WAIT_V(6); BAR; MMA(1, 1, At, B1); BAR;
    }
    if (wid == 0) lds_post(LDS_BYTES - 16, bad);
    asm volatile("s_waitcnt vmcnt(0) lgkmcnt(0)" ::: "memory");
    if (wr == 0) BAR;
    BAR;
#undef SA
#undef SB
#undef STAGE
#undef LDA
#undef LDB
#undef MMA
    int tz = threadIdx.x; asm volatile("" : "+v"(tz));
    lane = tz & 63;
    STAMP_AT(3 + round * 3);
    u32 miss = rfl(lds_poll(LDS_BYTES - 16));
    const rsrc_t rp = mkrsrc(wsl + WS_PART + ((pos & 15) * 8 + wid) * 32768u, 32768u);
    GAS u32* const pf = (GAS u32*)(wsl + WS_PFLAG + ((pos & 15) * 8 + wid) * 128);
    if (split && !help) {
      wait_ready(pf, call, t0, miss);
#pragma unroll
      for (int i0 = 0; i0 < 32; i0 += 16) {
        u32x4 p[16];
#pragma unroll
        for (int i = 0; i < 16; ++i) p[i] = __builtin_amdgcn_raw_buffer_load_b128(rp, lane * 16 + (i0 + i) * 1024, 0, 17);
#pragma unroll
        for (int i = i0; i < i0 + 16; ++i) acc[i >> 4][(i >> 3) & 1][(i >> 1) & 3][i & 1] += __builtin_bit_cast(f32x4, p[i - i0]);
        SCHED;
      }
    }
    if (miss) {
#pragma unroll
      for (int i = 0; i < 32; ++i) acc[i >> 4][(i >> 3) & 1][(i >> 1) & 3][i & 1] = (f32x4){__builtin_nanf(""), __builtin_nanf(""), __builtin_nanf(""), __builtin_nanf("")};
    }
    if (help) {
#pragma unroll
      for (int i = 0; i < 32; ++i) bst128(rp, lane * 16 + i * 1024, __builtin_bit_cast(u32x4, acc[i >> 4][(i >> 3) & 1][(i >> 1) & 3][i & 1]));
      waitv<0>();
      if (lane == 0) __hip_atomic_store(pf, call, __ATOMIC_RELAXED, __HIP_MEMORY_SCOPE_AGENT);
      break;
    }
    if (wr == 0) __builtin_amdgcn_s_setprio(2);
    pack_out(a, acc, tile * 8 + wid, call, lds, wid, lane);
    __builtin_amdgcn_s_setprio(0);
    STAMP_AT(4 + round * 3);
  }
}

DEV void loader_role(const Args& a, LAS char* lds, int peer, int lane, u64 t0) {
  const char* src = a.ws[peer] + WS_BLOB + lane * 16;
  u32 served = 0; int pend = -1; u32 pend_tag = 0;
  int done = 0;
  for (int left = NDEC; left > 0;) {
    const u32 r = lds_poll(L_REQ + (lane & 7) * 4);
    u32 key = ((lane & 7) < NDEC && (r & 255) != served) ? ((255 - (r & 255)) << 3 | (7 - (lane & 7))) + 1 : 0;
    key = umax(key, dpp0<0x111, 0xf>(key)); key = umax(key, dpp0<0x112, 0xf>(key)); key = umax(key, dpp0<0x114, 0xf>(key));
    key = (u32)__builtin_amdgcn_readlane((int)key, 7);
    if (key == 0) {
      if (pend >= 0) { waitv<0>(); lds_post(L_FULL + pend * 4, pend_tag); pend = -1; }
      else { if (now() - t0 > DEADLINE + 20000000ull) break; __builtin_amdgcn_s_sleep(2); }
      continue;
    }
    const int j = 7 - ((key - 1) & 7);
    const u32 rq = (u32)__builtin_amdgcn_readlane((int)r, j);
    if ((lane & 7) == j) served = rq & 255;
    if (rq >> 31) { --left; continue; }
    ++done;
    const u32 slot = umin(rq >> 18, NSLOT - 1), bytes = (((rq >> 8) & 1023) + 1) * 16;
    const char* s = src + (size_t)slot * 16384;
    LAS char* dst = lds + L_DEC + j * DREG + BOFF;
    int row = 0;
    for (u32 o = 0; o < bytes; o += 1024, ++row) {
      if (o + lane * 16 < bytes) __builtin_amdgcn_global_load_lds((const GAS u32*)(s + o), (LAS u32*)(dst + o), 16, 0, LD_AUX);
      waitv<WIN - 1>();
      if (row == WIN - 2 && pend >= 0) { lds_post(L_FULL + pend * 4, pend_tag); pend = -1; }
    }
    if (pend >= 0) { waitv<0>(); lds_post(L_FULL + pend * 4, pend_tag); }
    pend = j; pend_tag = rq & 255;
  }
  waitv<0>();
  if (pend >= 0) lds_post(L_FULL + pend * 4, pend_tag);
}

template <int NB> DEV void out_groups(u32 A, u32 B, u32 pl, u32 g0, u32 G, u32& k0, u32 cp, int lane) {
  u32x4 lo[NB]; u32 x[NB], e[NB][4], k[NB][5], sel[NB][4], d0[NB][4], d1[NB][4];
#pragma unroll
  for (int b = 0; b < NB; ++b) { x[b] = *(LAS u16*)lp(pl + 2 * (g0 + lane) + 128 * b); lo[b] = lds_r128(lp(B + 1024 + 16 * (g0 + lane) + 1024 * b)); }
  SCHED;
#pragma unroll
  for (int b = 0; b < NB; ++b) {
    x[b] = (x[b] << 16) | (x[b] >> 4); k[b][0] = 0;
#pragma unroll
    for (int i = 0; i < 4; ++i) { e[b][i] = ~((x[b] >> i) | lo[b][i]) & 0x01010101u; k[b][i + 1] = k[b][i] + __builtin_popcount(e[b][i]); }
  }
  u32 k1[NB];
#pragma unroll
  for (int b = 0; b < NB; b += 2) {
    const u32 c = NB > 1 ? k[b][4] | (k[(b + 1) % NB][4] << 16) : k[b][4], inc = wave_scan(c), tot = (u32)__builtin_amdgcn_readlane((int)inc, 63);
    k1[b] = k0 + ((inc - c) & 0xFFFF); k0 += tot & 0xFFFF;
    if (NB > 1) { k1[(b + 1) % NB] = k0 + ((inc - c) >> 16); k0 += tot >> 16; }
  }
  auto issue = [&](int b) {
#pragma unroll
    for (int i = 0; i < 4; ++i) {
      k[b][i] += k1[b];
      sel[b][i] = *(LAS u32*)lp(__builtin_amdgcn_udot4(e[b][i], 0x20100804u, L_LUT2, false));
      const LAS u32* q = (const LAS u32*)lp(k[b][i] & ~3u); d0[b][i] = q[0]; d1[b][i] = q[1];
    }
  };
  issue(0);
#pragma unroll
  for (int b = 0; b < NB; ++b) {
    if (b + 1 < NB) issue(b + 1);
    SCHED;
    u32 v[8];
#pragma unroll
    for (int i = 0; i < 4; ++i) {
      const u32 h = perm((lo[b][i] & 0x01010101u) | ((i ? x[b] >> (i - 1) : x[b] << 1) & 0x02020202u), __builtin_amdgcn_alignbyte(d1[b][i], d0[b][i], k[b][i]), sel[b][i]);
      v[2 * i] = pk_shr1(pk_sub(cp, perm(h, lo[b][i], 0x05010400u))); v[2 * i + 1] = pk_shr1(pk_sub(cp, perm(h, lo[b][i], 0x07030602u)));
    }
    const u32 g = umin(g0 + lane + 64 * b, G);
    lds_w128(lp(A + 32 * g), (u32x4){v[0], v[1], v[2], v[3]}); lds_w128(lp(A + 32 * g + 16), (u32x4){v[4], v[5], v[6], v[7]});
    SCHED;
  }
}
DEV void decoder_role(const Args& a, int peer, int c, int j, int lane, u32 call, u64 t0) {
  char* const wsl = a.wsl;
  const u32 A = L_DEC + j * DREG, B = A + BOFF;
  const int q = (peer - a.rank) & 7;
  bool fail = false, posted = false, more = true;
  u32 tag = 1;
  const char* const queue = wsl + WS_QUEUE + peer * 128;
  const char* const dsco = wsl + WS_DESCO + (size_t)peer * 32768;
  const u32 qbase = (call - 1) * QSTEP;
  const char* dsc = wsl + WS_DESC + (size_t)((peer * 16 + (c & 15)) * NIN + j) * 16;
  const char* ndsc = dsc + NDEC * 16;
  u32 da = 0, db = 0, nda = 0, ndb = 0;
  int m = j;
  if (c >= 16) {
    m = NIN;
    const u32 u0 = collect<0>(draw(queue, true)) - qbase, u1 = collect<0>(draw(queue, u0 < 2048)) - qbase;
    dsc = dsco + (u0 & 2047) * 16; ndsc = dsco + (u1 & 2047) * 16; more = u1 < 2048;
    if (u0 >= 2048) m = -1;
  }
  auto post = [&](int m, u32 da, u32 db, u32 tag) {
    u32 slot, bytes;
    if (m < NIN) {
      const int kj = m < 16 ? m >> 1 : m & 7, t = m < 16 ? c >> 3 : (m >> 3) & 1, s = m < 16 ? (m & 1) * 4 + ((c >> 1) & 3) : 8 + (c >> 1);
      slot = (t ? SLOT_W + a.rank * 256 : 0) + s * 16 + kj * 2 + (c & 1);
      bytes = in_bytes(umin(da >> 16, IN_UCAP));
    } else {
      const u32 G = (umin(da & 0xFFFF, 8192) + 15) >> 4;
      slot = SLOT_O + ((db >> 16) & 2047);
      bytes = out_bytes(G, umin(db & 0xFFFF, OUT_UCAP));
    }
    if (db >> 31) bytes = 16384;
    asm volatile("s_waitcnt lgkmcnt(0)" ::: "memory");
    lds_post(L_REQ + j * 4, (slot << 18) | ((bytes / 16 - 1) << 8) | tag);
  };
  auto peek = [&](u32x4 d) {
    u32 c0, c1;
    asm volatile("s_waitcnt vmcnt(0)\n\tv_readfirstlane_b32 %0, %4\n\tv_readfirstlane_b32 %1, %5\n\tv_readfirstlane_b32 %2, %6\n\tv_readfirstlane_b32 %3, %7"
                 : "=s"(c0), "=s"(nda), "=s"(ndb), "=s"(c1) : "v"(d.x), "v"(d.y), "v"(d.z), "v"(d.w) : "memory");
    return more && c0 == call && c1 == call;
  };
  for (; m >= 0; m += NDEC) {
    if (!posted) {
      u32x4 d;
      for (;;) {
        d = ld_sys128(dsc);
        if (rfl(d.x) == call && rfl(d.w) == call) break;
        if (now() - t0 > DEADLINE) { fail = true; break; }
        __builtin_amdgcn_s_sleep(4);
      }
      if (fail) break;
      da = rfl(d.y); db = rfl(d.z);
      if (m >= NPUSH) post(m, da, db, tag);
    }
    if (tag <= 11) STAMP_AT(12 + tag * 4);
    u32x4 nd; asm volatile("global_load_dwordx4 %0, %1, %2 sc0 sc1" : "=&v"(nd) : "v"(0), "s"(ndsc) : "memory");
    const bool drawn = more && m + 2 * NDEC >= NIN;
    u32 u = 0;
    const u32 ntag = tag == 255 ? 1 : tag + 1;
    const bool raw = db >> 31;
    const int kj = m < 16 ? m >> 1 : m & 7, t = m < 16 ? c >> 3 : (m >> 3) & 1, s = m < 16 ? (m & 1) * 4 + ((c >> 1) & 3) : 8 + (c >> 1);
    const u32 U = m < NIN ? umin(da >> 16, IN_UCAP) : umin(db & 0xFFFF, OUT_UCAP), G = (umin(da & 0xFFFF, 8192) + 15) >> 4;
    if (m < NPUSH) {
      const rsrc_t rl = mkrsrc(wsl + WS_BLOB + (SLOT_W + a.rank * 256 + peer * 32 + c * 2 + m) * 16384u, raw ? 16384u : in_bytes(U));
      u32x4 in[16];
#pragma unroll
      for (int i = 0; i < 16; ++i) in[i] = __builtin_amdgcn_raw_buffer_load_b128(rl, lane * 16 + i * 1024, 0, 17);
#pragma unroll
      for (int i = 0; i < 16; ++i) lds_w128(lp(B + i * 1024 + lane * 16), in[i]);
      cbar();
    }
    while (m >= NPUSH) {
      if (rfl(lds_poll(L_FULL + j * 4)) == tag) break;
      if (now() - t0 > DEADLINE) { fail = true; break; }
      __builtin_amdgcn_s_sleep(2);
    }
    if (fail) { peek(nd); break; }
    posted = false;
    if (tag <= 11) STAMP_AT(13 + tag * 4);
    if (m < NIN) {
      const u32 cp = (da & 0xFFFF) * 0x10001u;
      u32 H[32];
      if (raw) {
#pragma unroll
        for (int r = 0; r < 8; ++r) { const u32x4 t = lds_r128(lp(B + (8 + r) * 1024 + lane * 16)); H[4 * r] = t.x; H[4 * r + 1] = t.y; H[4 * r + 2] = t.z; H[4 * r + 3] = t.w; }
      } else {
        const u32x4 p0 = lds_r128(lp(B + 8192 + lane * 16)), p1 = lds_r128(lp(B + 9216 + lane * 16));
        read_unary(B + 10240, U, A, lane);
        const u32 P[8] = {p0.x, p0.y, p0.z, p0.w, p1.x, p1.y, p1.z, p1.w};
        u32 E[8], cnt = 0, k[33], sel[32], d0[32], d1[32];
#pragma unroll
        for (int r = 0; r < 8; ++r) { E[r] = ~(P[r] | (P[r] >> 1)) & 0x55555555u; cnt += __builtin_popcount(E[r]); }
        k[0] = A + (wave_scan(cnt) - cnt);
        auto issue = [&](int r) {
#pragma unroll
          for (int i = 4 * r; i < 4 * r + 4; ++i) {
            const u32 e = (E[r] >> (i & 2) * 2) & (0x01010101u << (i & 1) * 2);
            sel[i] = *(LAS u32*)lp(__builtin_amdgcn_udot4(e, (i & 1) ? 0x08040201u : 0x20100804u, L_LUT2, false));
            const LAS u32* w = (const LAS u32*)lp(k[i] & ~3u); d0[i] = w[0]; d1[i] = w[1];
            k[i + 1] = k[i] + __builtin_popcount(e);
          }
        };
        issue(0);
#pragma unroll
        for (int r = 0; r < 8; ++r) {
          if (r < 7) issue(r + 1);
          SCHED;
#pragma unroll
          for (int i = 4 * r; i < 4 * r + 4; ++i) H[i] = perm((P[r] >> (i & 3) * 2) & 0x03030303u, __builtin_amdgcn_alignbyte(d1[i], d0[i], k[i]), sel[i]);
          SCHED;
        }
      }
      u32x4 lo[8];
#pragma unroll
      for (int r = 0; r < 8; ++r) lo[r] = lds_r128(lp(B + r * 1024 + lane * 16));
      if (peek(nd)) { post(m + NDEC, nda, ndb, ntag); posted = true; }
      const u32 tk = draw(queue, drawn);
      const u32 so = t * IMG_BYTES + (u32)(s * 64 + 7 + kj * 7 + q) * 32768u + (c & 1) * (t ? 8192 : 16384);
      const rsrc_t rimg[2] = {mkrsrc(wsl + WS_IMG + so, 8192), mkrsrc(wsl + WS_IMG + so + (t ? 16384 : 8192), 8192)};
#pragma unroll
      for (int r = 0; r < 8; ++r) {
        u32 v[8];
#pragma unroll
        for (int k = 0; k < 4; ++k) {
          const u32 u0 = pk_sub(cp, perm(H[4 * r + k], lo[r][k], 0x05010400u)), u1 = pk_sub(cp, perm(H[4 * r + k], lo[r][k], 0x07030602u));
          v[2 * k] = __builtin_amdgcn_alignbit(u0, u0, 1); v[2 * k + 1] = __builtin_amdgcn_alignbit(u1, u1, 1);
        }
        ist128(rimg[r >> 2], lane * 16 + ((2 * r) & 7) * 1024, (u32x4){v[0], v[1], v[2], v[3]});
        ist128(rimg[r >> 2], lane * 16 + ((2 * r + 1) & 7) * 1024, (u32x4){v[4], v[5], v[6], v[7]});
      }
      if (tag <= 11) STAMP_AT(14 + tag * 4);
      u = collect<0>(tk);
      if (tag <= 11) STAMP_AT(15 + tag * 4);
      if (lane == 0) __hip_atomic_fetch_add((GAS u32*)(wsl + WS_READY + ((t * 4 + (s >> 2)) * 9 + 1 + kj) * 128), 1u, __ATOMIC_RELAXED, __HIP_MEMORY_SCOPE_AGENT);
    } else {
      const u32 blob = (db >> 16) & 2047, tile = blob >> 3, w8 = blob & 7;
      const u32 so = ((tile >> 4) * 256 + (w8 & 3) * 32) * 65536u + (peer * 4096 + (tile & 15) * 256 + (w8 >> 2) * 128) * 2u;
      const u32 vo = (lane >> 2) * 65536u + (lane & 3) * 16;
      const rsrc_t rout = mkrsrc(a.out + so, OUT_BYTES - so);
      bool nok;
      if (raw) {
        nok = peek(nd);
        const u32 tk = draw(queue, drawn);
#pragma unroll
        for (int ch = 0; ch < 16; ++ch)
          __builtin_amdgcn_raw_buffer_store_b128(lds_r128(lp(B + ch * 1024 + lane * 16)), rout, vo + (ch & 3) * 64 + ((ch >> 3) * 128 + ((ch >> 2) & 1) * 16) * 65536u, 0, OUT_AUX);
        u = collect<16>(tk);
      } else {
        const u32 cp = (da >> 16) * 0x10001u;
        const u32x4 bits = lds_r128(lp(B + lane * 16));
        const u32 pl = B + 1024 + 16 * G;
        u32 k0 = B + out_bytes(G, U), g0 = 0;
        read_unary(pl + ((2 * G + 3) & ~3u), U, k0, lane);
        for (; g0 + 128 < G; g0 += 256) out_groups<4>(A, B, pl, g0, G, k0, cp, lane);
        for (; g0 < G; g0 += 64) out_groups<1>(A, B, pl, g0, G, k0, cp, lane);
        nok = peek(nd);
        if (nok && 32 * G + 32 <= BOFF) { post(m + NDEC, nda, ndb, ntag); posted = true; }
        const u32 tk = draw(queue, drawn);
        u32 st[4], pos[8][4], d[8][4][3]; u32x2 sel[8][4];
        list_starts(bits, st);
        auto issue = [&](int h) {
          const u32 n0 = (bits[h >> 1] >> 1) & 0x78787878u, n1 = (bits[h >> 1] << 3) & 0x78787878u;
          const u32 p0 = A + ((h & 1) ? st[h >> 1] >> 16 : st[h >> 1] & 0xFFFF);
          u32 cnt = 0;
#pragma unroll
          for (int i = 0; i < 4; ++i) {
            const u32 n8 = (((i & 1) ? n1 : n0) >> (24 - 16 * (h & 1) - 8 * (i >> 1))) & 255;
            pos[h][i] = p0 + 2 * cnt; cnt += __builtin_popcount(n8);
            sel[h][i] = *(LAS u32x2*)lp(L_LUT + n8);
            const LAS u32* w = (const LAS u32*)lp(pos[h][i] & ~3u);
            d[h][i][0] = w[0]; d[h][i][1] = w[1]; d[h][i][2] = w[2];
          }
        };
        issue(0);
#pragma unroll
        for (int h = 0; h < 8; ++h) {
          if (h < 7) issue(h + 1);
          SCHED;
#pragma unroll
          for (int i = 0; i < 2; ++i) {
            u32x4 ov; const int ch = 2 * h + i;
#pragma unroll
            for (int n = 2 * i; n < 2 * i + 2; ++n) {
              const u32 e0 = __builtin_amdgcn_alignbyte(d[h][n][1], d[h][n][0], pos[h][n]), e1 = __builtin_amdgcn_alignbyte(d[h][n][2], d[h][n][1], pos[h][n]);
              ov[2 * (n & 1)] = perm(e1, e0, sel[h][n].x); ov[2 * (n & 1) + 1] = perm(e1, e0, sel[h][n].y);
            }
            __builtin_amdgcn_raw_buffer_store_b128(ov, rout, vo + (ch & 3) * 64 + ((ch >> 3) * 128 + ((ch >> 2) & 1) * 16) * 65536u, 0, OUT_AUX);
          }
          SCHED;
        }
        u = collect<16>(tk);
      }
      if (nok && !posted) { post(m + NDEC, nda, ndb, ntag); posted = true; }
      if (tag <= 11) STAMP_AT(14 + tag * 4);
    }
    tag = ntag; da = nda; db = ndb;
    if (!more) break;
    dsc = ndsc; ndsc += NDEC * 16;
    if (drawn) { u -= qbase; more = u < 2048; ndsc = dsco + (u & 2047) * 16; }
  }
  if (fail) tag = tag == 255 ? 1 : tag + 1;
  lds_post(L_REQ + j * 4, 0x80000000u | tag);
  waitv<0>();
  if (fail) {
    __builtin_amdgcn_raw_buffer_store_b32(0x7FC07FC0u, mkrsrc(a.out, OUT_BYTES), lane * 4 + (blockIdx.x * 8 + j) * 65536u, 0, 0);
    if (lane == 0) __hip_atomic_store((GAS u32*)(wsl + WS_TMO), call, __ATOMIC_RELAXED, __HIP_MEMORY_SCOPE_AGENT);
  }
}

__global__ void __launch_bounds__(512, 2) fused_gemm_ar(const Args a) {
  extern __shared__ __attribute__((aligned(16))) char smem[];
  LAS char* lds = (LAS char*)smem;
  const u64 t0 = now();
  const int wid = __builtin_amdgcn_readfirstlane(threadIdx.x >> 6), lane = threadIdx.x & 63;
  const int xcd = vbid() & 7, li = vbid() >> 3;
  Head hd = {a.x, a.w, a.wsl, a.rank, a.flags};
  asm volatile("" : "+s"(hd.x), "+s"(hd.w), "+s"(hd.wsl), "+s"(hd.rank), "+s"(hd.flags));
  char* const wsl = hd.wsl;
  GAS u32* callp = (GAS u32*)(wsl + WS_CALL) + blockIdx.x * 8 + wid;
  const u32 callv = __hip_atomic_load(callp, __ATOMIC_RELAXED, __HIP_MEMORY_SCOPE_AGENT);
  char* const wsp = a.ws[lane & 7];
  STAMP_AT(0);
  const bool is_gemm = li < 16;
  const int pos = xcd * 16 + (li & 15);
  auto tables = [&]() {
    if (wid == 0) {
      if (lane < 32) *(LAS u32*)(lds + L_REQ + lane * 4) = 0;
      if (lane < 16) {
        u32 k = 0, sel[2] = {0, 0};
#pragma unroll
        for (int i = 0; i < 4; ++i) {
          const bool on = (lane >> (3 - i)) & 1;
          sel[i >> 1] |= (on ? (2 * k) | ((2 * k + 1) << 8) : 0x0C0Cu) << (16 * (i & 1));
          k += on;
        }
        *(LAS u32x2*)(lds + L_LUT + lane * 8) = (u32x2){sel[0], sel[1]};
        u32 s2 = 0; k = 0;
#pragma unroll
        for (int i = 0; i < 4; ++i) { const bool on = (lane >> i) & 1; s2 |= (on ? k : 4 + i) << (8 * i); k += on; }
        *(LAS u32*)(lds + L_LUT2 + lane * 4) = s2;
      }
    }
    asm volatile("s_waitcnt lgkmcnt(0)" ::: "memory");
  };
  const int q = li - 16;
  const bool is_help = !is_gemm && (q & 7) == 0;
  if (!is_gemm && !is_help) tables();
  {
    int e = -1, n = 1;
    const bool early = is_gemm && (xcd & 1);
    if (early) { if (wid < 4) e = 2176 + wid * 64 + (xcd >> 1) * 16 + li; }
    else if (is_gemm) { e = 512 + wid * 64 + (xcd >> 1) * 16 + li; n = 3; }
    else if (is_help) { e = 1024 + (xcd * 2 + (q >> 3)) * 8 + wid; n = 2; }
    else if (wid >= 3) { e = 128 + (wid == 7 ? 2 : wid == 5 ? 3 : wid - 3) * 112 + (xcd * 2 + (q >> 3)) * 7 + (q & 7) - 1; if (e >= 512 || wid == 6) e = -1; }
    else if (wid) e = (wid - 1) * 64 + ((q ^ (wid * 8)) & 8) + xcd;
    if (!is_gemm && wid < 5) __builtin_amdgcn_s_setprio(1);
    LAS char* scr = is_gemm || is_help ? lds + wid * EREG : wid == 7 ? lds + L_SCR7 : lds + L_DEC + (wid - 1) * DREG + BOFF;
    if (GDELAY && n > 1) for (int i = 0; i < GDELAY; ++i) __builtin_amdgcn_s_sleep(37);
    if (D2 && (is_gemm || wid > 2)) __builtin_amdgcn_s_sleep(D2);
    if (e < 0 && !is_gemm) __builtin_amdgcn_s_barrier();
    if (e >= 0)
      for (int i = 0; i < n && (e < 2176 || n == 1); ++i, e += 640) {
        bool isx; int j, s, dq, mode;
        if (e < 128) { j = 0; isx = (e & 8) == 0; s = (e >> 6) * 4 + ((e >> 1) & 3); dq = -q; mode = 6; }
        else if (e < 1024) { j = e >> 7; dq = (e >> 3) & 7; isx = dq == 0; s = ((e >> 6) & 1) * 4 + ((e >> 1) & 3); mode = 2; }
        else if (e < 1152) { j = (e >> 4) & 7; isx = true; s = 8 + ((e >> 1) & 7); dq = 0; mode = 3; }
        else if (e < 2176) { j = (e - 1152) >> 7; isx = false; s = 8 + ((e >> 1) & 7); dq = (e >> 4) & 7; mode = dq ? 2 : 1; }
        else { j = (e >> 4) & 7; isx = e < 2304; s = (e >> 1) & 7; dq = 0; mode = 1; }
        prep_piece(hd, !is_gemm && !is_help, mode, isx, s, j, e & 1, (hd.rank + dq) & 7, callv, wsp, scr, lane, i);
        if (is_help) STAMP_AT(16 + i);
      }
  }
  __builtin_amdgcn_s_setprio(0);
  const u32 call = rfl(callv) + 1;
  if (lane == 0) __hip_atomic_store(callp, call, __ATOMIC_RELAXED, __HIP_MEMORY_SCOPE_AGENT);
  STAMP_AT(1);
  if (is_gemm || is_help) gemm_role(a, lds, is_help ? 16 + xcd * 2 + (q >> 3) : pos, is_help, wid, call, t0);
  if (is_help) { tables(); __builtin_amdgcn_s_barrier(); }
  if (!is_gemm && (!is_help || (xcd && (q >> 3) < NHELP))) {
    const int peer = (a.rank + (is_help ? xcd : q & 7)) & 7, c = is_help ? 16 : xcd * 2 + (q >> 3);
    if (wid == 0) loader_role(a, lds, peer, lane, t0);
    else if (wid <= NDEC) decoder_role(a, peer, c, wid - 1, lane, call, t0);
  }
  STAMP_AT(8);
}

#define HIP_CHECK(expr) do { hipError_t _e = (expr); if (_e != hipSuccess) { \
    fprintf(stderr, "HIP error %s at %s:%d\n", hipGetErrorString(_e), __FILE__, __LINE__); \
    return; } } while (0)

#define DBG_FLAGS 0

extern "C" void kernel_launch(int rank, int nranks, ncclComm_t comm, hipStream_t stream,
                              void* const* d_in, const int* in_sizes, int n_in,
                              void* d_out, int out_size, const int* in_dtypes, int out_dtype,
                              void* const* d_out_all, void* const* d_ws_all, size_t ws_bytes) {
  if (nranks != NR || (unsigned)rank >= (unsigned)NR || n_in < 2 || ws_bytes < WS_USED || (size_t)out_size * 2 != OUT_BYTES ||
      in_sizes[0] != 4096 * 512 || in_sizes[1] != 512 * 32768) {
    fprintf(stderr, "unexpected problem shape\n");
    return;
  }
  static bool attr_set[NR] = {};
  if (!attr_set[rank]) {
    HIP_CHECK(hipFuncSetAttribute((const void*)fused_gemm_ar, hipFuncAttributeMaxDynamicSharedMemorySize, LDS_BYTES));
    attr_set[rank] = true;
  }
  Args a;
  a.x = (const char*)d_in[0]; a.w = (const char*)d_in[1]; a.out = (char*)d_out;
  for (int p = 0; p < NR; ++p) a.ws[p] = (char*)d_ws_all[p];
  a.wsl = a.ws[rank]; a.rank = rank; a.flags = DBG_FLAGS;
  static const int blk[8][2] = {{2, 0}, {0, 0}, {2, 1}, {1, 0}, {3, 0}, {0, 1}, {3, 1}, {1, 1}};
  for (int k = 0; k < 256; ++k) {
    const int b = (k >> 4) & 7, i = k & 15;
    a.tiles[k] = (unsigned char)((blk[b][0] * 4 + (i >> 2)) * 16 + (blk[b][1] + (k >> 7) * 2) * 4 + (i & 3));
  }
  hipLaunchKernelGGL(fused_gemm_ar, dim3(256), dim3(512), LDS_BYTES, stream, a);
  HIP_CHECK(hipGetLastError());
}
```
